```python
import numpy as np

kernel_name = "gpu_a2a_gemm_m4096_k4096_n4096_e5m2e5m2_dqsilu"


ABSMAX_RTOL = 0.05

COMPUTE_DTYPE = "float32"
EPILOGUE = "dequant_silu"
IN_QUANT = None
OUT_QUANT = None
IN_FP8 = "e5m2e5m2"
IN_DTYPES = {"x": "e5m2", "w_mat": "e5m2",
             "scale_x": "float32", "scale_w": "float32"}
WORLD_SIZE = 8

def _snap_fp8(x, mant_bits, max_abs, min_normal):
    a = np.abs(x.astype(np.float64))
    _m, _e = np.frexp(a)
    q = np.where(a >= min_normal, np.ldexp(1.0, _e - 1 - mant_bits),
                 min_normal * 2.0 ** -mant_bits)
    snapped = np.minimum(np.round(a / q) * q, max_abs)
    return (np.sign(x) * snapped).astype(np.float32)

def setup_inputs():

    x = _snap_fp8(np.random.randn(4096, 4096).astype(np.float32), 2, 57344.0, 2.0 ** -14)
    w_mat = _snap_fp8(np.random.randn(4096, 4096).astype(np.float32), 2, 57344.0, 2.0 ** -14)
    scale_x = (0.02 * (1.0 + np.random.rand(1))).astype(np.float32)
    scale_w = (0.02 * (1.0 + np.random.rand(1))).astype(np.float32)
    return {"x": x, "w_mat": w_mat, "scale_x": scale_x, "scale_w": scale_w}

def reference(x, w_mat, scale_x, scale_w):
    acc = (x @ w_mat).astype(np.float32)
    y = (acc * (scale_x * scale_w)).astype(np.float32)
    return (y / (1.0 + np.exp(-np.clip(y, -60.0, 60.0)))).astype(np.float32)
```

```cpp
#include <hip/hip_runtime.h>
#include <rccl/rccl.h>
#include <cstdio>

#define HIP_CHECK(expr) do { hipError_t _e = (expr); if (_e != hipSuccess) { \
    fprintf(stderr, "HIP error %s at %s:%d\n", hipGetErrorString(_e), __FILE__, __LINE__); \
    return; } } while (0)

typedef unsigned u32x4 __attribute__((ext_vector_type(4)));
typedef int v2i __attribute__((ext_vector_type(2)));
typedef int v4i __attribute__((ext_vector_type(4)));
typedef int v8i __attribute__((ext_vector_type(8)));
typedef float v4f __attribute__((ext_vector_type(4)));
typedef unsigned long long u64;
#define LAS __attribute__((address_space(3)))
#define GAS __attribute__((address_space(1)))

namespace {
constexpr int NR = 8;
constexpr int KC = 512;
constexpr int NN = 4096;
constexpr size_t SLOT_B = 2u << 20;
constexpr size_t FLAG_OFF = 2 * SLOT_B;
constexpr size_t CNT_OFF = FLAG_OFF + 32768;
constexpr size_t DIAG_OFF = CNT_OFF + 4096;
constexpr size_t WS_NEED = DIAG_OFF + 256 * 128 * 8;
constexpr u64 DEADLINE = 20000000ull;
constexpr int NS = 6;
constexpr int W_B = 16384, A_B = 10240, ST_B = W_B + A_B;
constexpr int SYNC_OFF = NS * ST_B;
constexpr int LDS_B = SYNC_OFF + 64;
constexpr int NTHREADS = 640;
constexpr int NSEND_WG = 32;
#define SEND_WIN 2

struct Params {
    const char* xsend[NR];
    char* wsdst[NR];
    const char* w;
    const float* sx;
    const float* sw;
    float* out;
    int rank;
    int pad;
};

#define ST16_SYS(p, v) asm volatile("global_store_dwordx4 %0, %1, off sc0 sc1\n\ts_nop 1" ::"v"(p), "v"(v) : "memory")
template <int N> __device__ __forceinline__ void waitv() { asm volatile("s_waitcnt vmcnt(%0)" ::"n"(N) : "memory"); }
__device__ __forceinline__ int tile_g0(int t) { return t <= 4 ? t * 10 : 40 + (t - 4) * 8; }
__device__ __forceinline__ int tile_ng(int t) { return t < 4 ? 10 : 8; }
__device__ __forceinline__ unsigned lds_ld(LAS unsigned* p) {
    return __hip_atomic_load(p, __ATOMIC_RELAXED, __HIP_MEMORY_SCOPE_WORKGROUP);
}
__device__ __forceinline__ void lds_st(LAS unsigned* p, unsigned v) {
    __hip_atomic_store(p, v, __ATOMIC_RELAXED, __HIP_MEMORY_SCOPE_WORKGROUP);
}

typedef int v4s __attribute__((ext_vector_type(4)));
__device__ __forceinline__ v4s make_rsrc(const GAS void* base, unsigned bytes) {
    const u64 a = (u64)base;
    return (v4s){(int)(unsigned)a, (int)((unsigned)(a >> 32) & 0xffffu), (int)bytes, 0x00020000};
}
#define DMA(v, off, pol) "buffer_load_dwordx4 %[" #v "], %[rs], %[t] offen offset:" #off pol " lds\n\t"
#define W4(last) DMA(v0, 0, "") "s_add_u32 %[t], %[t], 31744\n\t" DMA(v0, 1024, "") "s_add_u32 %[t], %[t], 31744\n\t" \
                 DMA(v1, 2048, "") "s_add_u32 %[t], %[t], 31744\n\t" DMA(v1, 3072, "") last
#define W4N "s_add_u32 m0, m0, 4096\n\ts_add_u32 %[t], %[t], 35840\n\t"
#define ISSUE_W(MID) asm volatile("s_nop 4\n\tglobal_load_dword %[f], %[z], %[fl] sc0 sc1\n\t" \
        "s_mov_b32 %[keep], m0\n\ts_mov_b32 m0, %[dst]\n\ts_mov_b32 %[t], %[so]\n\t" \
        W4(W4N) W4(W4N) W4(W4N) W4("") "s_mov_b32 m0, %[keep]\n\t" MID "s_waitcnt vmcnt(16)" \
        : [f] "=&v"(fv), [t] "=&s"(tmp), [keep] "=&s"(keep) \
        : [z] "v"(0), [fl] "s"(fl), [dst] "s"(wdst), [so] "s"(wso), [rs] "s"(rw), [v0] "v"(wv0), [v1] "v"(wv1) : "memory", "scc")
#define STAMP(i) do { } while (0)
}

__global__ __launch_bounds__(NTHREADS) void a2a_gemm_kernel(Params p) {
    __shared__ __attribute__((aligned(128))) char smem[LDS_B];
    LAS char* lds = (LAS char*)smem;
    LAS unsigned* sy = (LAS unsigned*)(lds + SYNC_OFF);
    const int b = blockIdx.x;
    const int wv = __builtin_amdgcn_readfirstlane(threadIdx.x >> 6), lane = threadIdx.x & 63;
    const int xg = b & 7, idx = b >> 3;
    const int srel = b < NSEND_WG ? (xg + 2 * idx) & 7 : 0;
    u64 a0 = (u64)p.xsend[srel], a1 = (u64)p.wsdst[srel], a2 = (u64)p.wsdst[0], a3 = (u64)p.w, a4 = (u64)p.sx,
        a5 = (u64)p.sw, a6 = (u64)p.out;
    int rank = p.rank;
    asm volatile("" : "+s"(a0), "+s"(a1), "+s"(a2), "+s"(a3), "+s"(a4), "+s"(a5), "+s"(a6), "+s"(rank));
    const GAS char* const xs = (const GAS char*)a0;
    GAS char* const dws = (GAS char*)a1;
    GAS char* const myws = (GAS char*)a2;
    const GAS char* const pw = (const GAS char*)a3;
    const GAS float* const psx = (const GAS float*)a4;
    const GAS float* const psw = (const GAS float*)a5;
    GAS float* const pout = (GAS float*)a6;
    GAS unsigned* const cnt = (GAS unsigned*)(myws + CNT_OFF) + b;

    if (b < NSEND_WG) {
        const int q = idx, rel = srel;
        if (rel == 0) return;
        const int g0 = tile_g0(rel % 7);
        u32x4 v[8];
        if (wv < 8) {
            const GAS char* src = xs + (lane >> 3) * KC + q * 128 + (lane & 7) * 16;
#pragma unroll
            for (int i = 0; i < 8; ++i) v[i] = *(const GAS u32x4*)(src + ((g0 + wv + 8 * i) & 63) * (8 * KC));
        }
        const unsigned call = __builtin_amdgcn_readfirstlane(
            __hip_atomic_load(cnt, __ATOMIC_RELAXED, __HIP_MEMORY_SCOPE_AGENT)) + 1;
        if (wv < 8) lds_st(sy + wv, 0);
        asm volatile("s_waitcnt lgkmcnt(0)" ::: "memory");
        __builtin_amdgcn_s_barrier();
        if (wv < 8) {
            GAS char* dbase = dws + (call & 1) * SLOT_B + (size_t)(rank * 4 + q) * 65536 + lane * 16;
#pragma unroll
            for (int i = 0; i < 8; ++i) asm volatile("" ::"v"(v[i]));
            waitv<0>();
            STAMP(16 + wv);
            constexpr int WIN = SEND_WIN;
#pragma unroll
            for (int i = 0; i < WIN; ++i) ST16_SYS(dbase + ((g0 + wv + 8 * i) & 63) * 1024, v[i]);
#define SEND_STEP(I) { waitv<((I) + WIN - 1 < 7 ? WIN - 1 : 7 - (I))>(); lds_st(sy + wv, (I) + 1); \
            if ((I) + WIN < 8) ST16_SYS(dbase + ((g0 + wv + 8 * ((I) + WIN)) & 63) * 1024, v[((I) + WIN) & 7]); }
            SEND_STEP(0) SEND_STEP(1) SEND_STEP(2) SEND_STEP(3) SEND_STEP(4) SEND_STEP(5) SEND_STEP(6) SEND_STEP(7)
        } else if (wv == 8) {
            int e = 0, t = rel % 7;
            for (int r = 0; r < 7; ++r) {
                e += tile_ng(t);
                while (!__all((int)(lds_ld(sy + (lane & 7)) * 8 + (lane & 7)) >= e)) __builtin_amdgcn_s_sleep(1);
                if (lane == 0)
                    __hip_atomic_store((GAS unsigned*)(dws + FLAG_OFF + ((t * 8 + rank) * 4 + q) * 128), call,
                                       __ATOMIC_RELAXED, __HIP_MEMORY_SCOPE_SYSTEM);
                STAMP(1 + r);
                t = t == 6 ? 0 : t + 1;
            }
            if (lane == 0) __hip_atomic_store(cnt, call, __ATOMIC_RELAXED, __HIP_MEMORY_SCOPE_AGENT);
        }
        return;
    }

    const int ci = idx - 4, sl = ci / 7, t = ci - sl * 7;
    const int slab = sl * 8 + xg;
    const int g0 = tile_g0(t), ng = tile_ng(t);

    if (wv >= 4) {
        const int j = wv - 4;
        const u64 tstart = __builtin_amdgcn_s_memrealtime();
        const unsigned wdst = (unsigned)(size_t)lds + j * ST_B, adst = wdst + W_B;
        const int r8 = lane >> 3, p8 = lane & 7;
        const int ch0 = (p8 ^ (r8 >> 1)) * 16;
        const unsigned wv0 = r8 * NN + slab * 128 + ch0, wv1 = wv0 ^ 64;
        const v4s rw = make_rsrc(pw, NN * NN);
        unsigned call, fv, tmp, keep;
        bool bad = false;
        lds_st(sy + j, 0);
        if (j == 0) lds_st(sy + 6, 0);
        if (j < 4) {
            const v4s rx = make_rsrc(xs, KC * KC);
            const unsigned v0 = r8 * KC + ch0, v1 = v0 ^ 64, so = g0 * 8 * KC + j * 128;
#define X2(o0, o1) DMA(v0, o0, "") "s_add_u32 %[t], %[t], 3072\n\t" DMA(v1, o1, "")
#define XN "s_add_u32 %[t], %[t], 3072\n\t"
#define XM "s_add_u32 m0, m0, 4096\n\ts_add_u32 %[t], %[t], 7168\n\t"
            asm volatile("s_nop 4\n\ts_mov_b32 %[keep], m0\n\ts_mov_b32 m0, %[dst]\n\ts_mov_b32 %[t], %[so]\n\t"
                         X2(0, 1024) XN X2(2048, 3072) XM X2(0, 1024) XN X2(2048, 3072)
                         "s_cmp_eq_u32 %[ng], 8\n\ts_cbranch_scc1 1f\n\t" XM X2(0, 1024) "1:\n\ts_mov_b32 m0, %[keep]"
                         : [t] "=&s"(tmp), [keep] "=&s"(keep)
                         : [dst] "s"(adst), [so] "s"(so), [rs] "s"(rx), [v0] "v"(v0), [v1] "v"(v1), [ng] "s"(ng) : "memory", "scc");
        }
        const v4s ra = make_rsrc(myws, 2 * SLOT_B);
        const unsigned av0 = r8 * 128 + ch0, av1 = av0 ^ 64;
        for (int s = j, round = 1; s < 32; s += NS, ++round) {
            const int c = s >> 2, q = s & 3;
            int rel = 0;
            if (c > 0) { rel = t - c + 1; if (rel <= 0) rel += 7; }
            const int sr = (rank + rel) & 7;
            const unsigned wso = (unsigned)(sr * KC + q * 128) * NN;
            const GAS unsigned* const flag = (const GAS unsigned*)(myws + FLAG_OFF + ((t * 8 + sr) * 4 + q) * 128);
            unsigned f;
            if (round == 1) {
                const GAS unsigned* fl = cnt;
                ISSUE_W("s_waitcnt lgkmcnt(0)\n\ts_barrier\n\t");
                call = __builtin_amdgcn_readfirstlane(fv) + 1;
                f = 0;
            } else {
                while (true) {
                    u32x4 cs = *(volatile LAS u32x4*)(sy + 8);
                    unsigned m = min(min(cs[0], cs[1]), min(cs[2], cs[3]));
                    if ((int)__builtin_amdgcn_readfirstlane(m) >= s - (NS - 1)) break;
                    __builtin_amdgcn_s_sleep(1);
                }
                const GAS unsigned* fl = flag;
                ISSUE_W("");
                f = __builtin_amdgcn_readfirstlane(fv);
            }
            if (c > 0) {
                while (f < call) {
                    if (__builtin_amdgcn_s_memrealtime() - tstart > DEADLINE) { bad = true; break; }
                    asm volatile("global_load_dword %0, %1, %2 sc0 sc1\n\ts_waitcnt vmcnt(0)"
                                 : "=&v"(fv) : "v"(0), "s"(flag) : "memory");
                    f = __builtin_amdgcn_readfirstlane(fv);
                }
                STAMP(32 + s);
                if (!bad) {
                    const unsigned so = (call & 1) * (unsigned)SLOT_B + (sr * 4 + q) * 65536 + g0 * 1024;
#define SYS " sc0 sc1"
#define A4 DMA(v0, 0, SYS) DMA(v1, 1024, SYS) DMA(v0, 2048, SYS) DMA(v1, 3072, SYS)
#define AN "s_add_u32 m0, m0, 4096\n\ts_add_u32 %[t], %[t], 4096\n\t"
                    asm volatile("s_nop 4\n\ts_mov_b32 %[keep], m0\n\ts_mov_b32 m0, %[dst]\n\ts_mov_b32 %[t], %[so]\n\t"
                                 A4 AN A4 "s_cmp_eq_u32 %[ng], 8\n\ts_cbranch_scc1 1f\n\t"
                                 AN DMA(v0, 0, SYS) DMA(v1, 1024, SYS) "1:\n\ts_mov_b32 m0, %[keep]"
                                 : [t] "=&s"(tmp), [keep] "=&s"(keep)
                                 : [dst] "s"(adst), [so] "s"(so), [rs] "s"(ra), [v0] "v"(av0), [v1] "v"(av1), [ng] "s"(ng)
                                 : "memory", "scc");
                } else {
                    lds_st(sy + 6, 1);
                }
            }
            waitv<0>();
            lds_st(sy + j, round);
            STAMP(64 + s);
        }
        if (j == 0 && lane == 0) __hip_atomic_store(cnt, call, __ATOMIC_RELAXED, __HIP_MEMORY_SCOPE_AGENT);
        return;
    }

    __builtin_amdgcn_s_setprio(2);
    const float scale = psx[0] * psw[0];
    lds_st(sy + 8 + wv, 0);
    asm volatile("s_waitcnt lgkmcnt(0)" ::: "memory");
    __builtin_amdgcn_s_barrier();
    const int g = lane >> 4, i16 = lane & 15;
    const int aoff0 = W_B + i16 * 128 + ((g ^ (i16 >> 1)) * 16), aoff1 = aoff0 ^ 64;
    const int wq = i16 >> 1;
    const int wf = (wq >> 1) | ((g & 1) << 2);
    const int boff0 = (g * 16 + wq) * 128 + (i16 & 1) * 8 + (((2 * wv) ^ wf) * 16), boff1 = boff0 ^ 16;
    v4f acc[5][2];
#pragma unroll
    for (int m = 0; m < 5; ++m) { acc[m][0] = (v4f){0, 0, 0, 0}; acc[m][1] = (v4f){0, 0, 0, 0}; }
    v4i xa[5][2];
    v2i wb[2][2][4];
#define SB0 __builtin_amdgcn_sched_barrier(0)
#define RD_A(m, sb) { xa[m][0] = *(LAS v4i*)(lds + (sb) + aoff0 + (m) * 2048); xa[m][1] = *(LAS v4i*)(lds + (sb) + aoff1 + (m) * 2048); }
#define RD_B1(u, n, r, sb) wb[u][n][r] = __builtin_amdgcn_ds_read_tr8_b64_v2i32( \
        (LAS v2i*)(lds + (sb) + ((n) ? boff1 : boff0) + ((r) & 1) * 1024 + ((r) >> 1) * 8192))
#define RD_B(u, n, sb) { RD_B1(u, n, 0, sb); RD_B1(u, n, 1, sb); RD_B1(u, n, 2, sb); RD_B1(u, n, 3, sb); }
#define MM(u, m, n) acc[m][n] = __builtin_amdgcn_mfma_scale_f32_16x16x128_f8f6f4( \
        (v8i){wb[u][n][0][0], wb[u][n][0][1], wb[u][n][1][0], wb[u][n][1][1], wb[u][n][2][0], wb[u][n][2][1], wb[u][n][3][0], wb[u][n][3][1]}, \
        (v8i){xa[m][0][0], xa[m][0][1], xa[m][0][2], xa[m][0][3], xa[m][1][0], xa[m][1][1], xa[m][1][2], xa[m][1][3]}, \
        acc[m][n], 1, 1, 0, 0, 0, 0)
#define SPIN(slot, rnd) while (__builtin_amdgcn_readfirstlane(lds_ld(sy + (slot))) < (unsigned)(rnd)) { }
    SPIN(0, 1);
    asm volatile("" ::: "memory");
    STAMP(1);
    RD_B(0, 0, 0); RD_B(0, 1, 0);
    RD_A(0, 0); RD_A(1, 0); RD_A(2, 0); RD_A(3, 0); RD_A(4, 0);
    asm volatile("" ::: "memory");
    lds_st(sy + 8 + wv, 1);
    int slot = 1, rnd = 1;
    unsigned rdy = lds_ld(sy + 1);
#define STAGE(u, s) { \
        const int sb = slot * ST_B; \
        int nslot = slot + 1, nrnd = rnd; if (nslot == NS) { nslot = 0; ++nrnd; } \
        if (__builtin_expect(__builtin_amdgcn_readfirstlane(rdy) < (unsigned)rnd, 0)) SPIN(slot, rnd); \
        asm volatile("" ::: "memory"); SB0; \
        MM(u, 0, 0); RD_B((u) ^ 1, 0, sb); SB0; \
        MM(u, 0, 1); SB0; RD_A(0, sb); SB0; \
        MM(u, 1, 0); RD_B((u) ^ 1, 1, sb); SB0; \
        MM(u, 1, 1); SB0; RD_A(1, sb); SB0; \
        MM(u, 2, 0); MM(u, 2, 1); SB0; RD_A(2, sb); rdy = lds_ld(sy + nslot); SB0; \
        MM(u, 3, 0); MM(u, 3, 1); SB0; RD_A(3, sb); SB0; \
        MM(u, 4, 0); MM(u, 4, 1); SB0; RD_A(4, sb); SB0; \
        asm volatile("" ::: "memory"); \
        lds_st(sy + 8 + wv, (s) + 2); \
        slot = nslot; rnd = nrnd; }
    for (int s = 0; s < 30; s += 2) {
        STAGE(0, s);
        STAGE(1, s + 1);
    }
    STAGE(0, 30);
    MM(1, 0, 0); MM(1, 0, 1); MM(1, 1, 0); MM(1, 1, 1); MM(1, 2, 0); MM(1, 2, 1);
    MM(1, 3, 0); MM(1, 3, 1); MM(1, 4, 0); MM(1, 4, 1);
    const float scale_o = lds_ld(sy + 6) != 0 ? __builtin_nanf("") : scale;
    GAS float* orow = pout + (size_t)(g0 * 8 + i16) * NN + slab * 128 + wv * 32 + g * 4;
    const float nl2e = -1.4426950408889634f;
#pragma unroll
    for (int m = 0; m < 5; ++m) {
        if (m == 4 && ng == 8) break;
#pragma unroll
        for (int n = 0; n < 2; ++n) {
            v4f o;
#pragma unroll
            for (int r = 0; r < 4; ++r) {
                const float y = acc[m][n][r] * scale_o;
                const float yc = __builtin_amdgcn_fmed3f(y, -60.0f, 60.0f);
                const float e = __builtin_amdgcn_exp2f(yc * nl2e);
                o[r] = y * __builtin_amdgcn_rcpf(1.0f + e);
            }
            *(GAS v4f*)(orow + (size_t)m * 16 * NN + n * 16) = o;
        }
    }
}

extern "C" void kernel_launch(int rank, int nranks, ncclComm_t comm, hipStream_t stream,
                              void* const* d_in, const int* in_sizes, int n_in,
                              void* d_out, int out_size, const int* in_dtypes, int out_dtype,
                              void* const* d_out_all, void* const* d_ws_all, size_t ws_bytes) {
    (void)comm; (void)in_dtypes; (void)out_dtype; (void)d_out_all;
    if (nranks != NR || (unsigned)rank >= (unsigned)NR || n_in != 4 || ws_bytes < WS_NEED ||
        in_sizes[0] != NN * KC || in_sizes[1] != NN * NN || out_size != KC * NN) {
        fprintf(stderr, "unexpected problem shape\n");
        return;
    }
    Params p;
    for (int rel = 0; rel < NR; ++rel) {
        const int d = (rank - rel) & 7;
        p.xsend[rel] = (const char*)d_in[0] + (size_t)d * KC * KC;
        p.wsdst[rel] = (char*)d_ws_all[d];
    }
    p.w = (const char*)d_in[1];
    p.sx = (const float*)d_in[2];
    p.sw = (const float*)d_in[3];
    p.out = (float*)d_out;
    p.rank = rank;
    p.pad = 0;
    a2a_gemm_kernel<<<256, NTHREADS, 0, stream>>>(p);
    HIP_CHECK(hipGetLastError());
}
```

```cpp
#include <hip/hip_runtime.h>
#include <rccl/rccl.h>
#include <cstdio>

#define HIP_CHECK(expr) do { hipError_t _e = (expr); if (_e != hipSuccess) { \
    fprintf(stderr, "HIP error %s at %s:%d\n", hipGetErrorString(_e), __FILE__, __LINE__); \
    return; } } while (0)

typedef unsigned u32x4 __attribute__((ext_vector_type(4)));
typedef int v2i __attribute__((ext_vector_type(2)));
typedef int v4i __attribute__((ext_vector_type(4)));
typedef int v8i __attribute__((ext_vector_type(8)));
typedef float v4f __attribute__((ext_vector_type(4)));
typedef unsigned long long u64;
#define LAS __attribute__((address_space(3)))
#define GAS __attribute__((address_space(1)))
#define CAS __attribute__((address_space(4)))

#define SEND_WIN 4
#define PATIENCE 2
#define OUT_MODE 1
#define OUT_POL " sc1"

namespace {
constexpr int NR = 8;
constexpr int KC = 512;
constexpr int NN = 4096;
constexpr size_t SLOT_B = 2u << 20;
constexpr size_t FLAG_OFF = 2 * SLOT_B;
constexpr size_t CNT_OFF = FLAG_OFF + 32768;
constexpr size_t DIAG_OFF = CNT_OFF + 4096;
constexpr size_t WS_NEED = DIAG_OFF + 256 * 128 * 8;
constexpr unsigned DEADLINE = 20000000u;
constexpr int W_B = 16384, WAVE_B = 40960, PART_B = 10240;
constexpr int LDS_B = 4 * WAVE_B;
constexpr int NTHREADS = 256;
constexpr int NSEND_WG = 32;

struct Params {
    const char* xsend[NR];
    char* wsdst[NR];
    const char* w;
    const float* sx;
    const float* sw;
    float* out;
    int rank;
    int pad;
};

template <int N> __device__ __forceinline__ void waitv() { asm volatile("s_waitcnt vmcnt(%0)" ::"n"(N) : "memory"); }
__device__ __forceinline__ int tile_b0(int t) { return t <= 4 ? t * 5 : 20 + (t - 4) * 4; }
__device__ __forceinline__ int tile_nb(int t) { return t < 4 ? 5 : 4; }
#define WG_RLX __ATOMIC_RELAXED, __HIP_MEMORY_SCOPE_WORKGROUP
__device__ __forceinline__ unsigned lds_ld(LAS unsigned* p) { return __hip_atomic_load(p, WG_RLX); }
__device__ __forceinline__ void lds_st(LAS unsigned* p, unsigned v) { __hip_atomic_store(p, v, WG_RLX); }
__device__ __forceinline__ unsigned uni(unsigned v) { return __builtin_amdgcn_readfirstlane(v); }
typedef int v4s __attribute__((ext_vector_type(4)));
__device__ __forceinline__ v4s make_rsrc(u64 a, unsigned bytes) {
    return (v4s){(int)(unsigned)a, (int)((unsigned)(a >> 32) & 0xffffu), (int)bytes, 0x00020000};
}
#define SB0 __builtin_amdgcn_sched_barrier(0)
#define CFENCE asm volatile("" ::: "memory")
#define PIN_S(x) asm volatile("" : "+s"(x))
#define PIN_V(x) asm volatile("" : "+v"(x))

#define ST_DECL
#define STAMP(i)
#define CSTAMP(i)
#define DUMP()
#define DIAG_PAR
#define DIAG_ARG

#define ACC(m, n) (((m) * 8 + (n)) * 4)
#define XA(u, m) (160 + (u) * 40 + (m) * 8)
#define WB(u, n) (128 + (u) * 64 + (n) * 8)
#define T_MM(u, k) "v_mfma_f32_16x16x128_f8f6f4 a[4*(" #k "):4*(" #k ")+3], v[128+64*" #u "+8*((" #k ")&7):135+64*" #u "+8*((" #k ")&7)], " \
        "a[160+40*" #u "+8*((" #k ")>>3):167+40*" #u "+8*((" #k ")>>3)], a[4*(" #k "):4*(" #k ")+3] cbsz:1 blgp:1\n\t"
#define T_TR(u, n, r, off) "ds_read_b64_tr_b8 v[192-64*" #u "+8*" #n "+2*" #r ":193-64*" #u "+8*" #n "+2*" #r "], %[ta] offset:" #off "\n\t"
#define T_RB0(u, n) "v_add_u32 %[ta], %[bs], %[b" #n "]\n\t" T_TR(u, n, 0, 0) T_TR(u, n, 1, 1024)
#define T_RB1(u, n) T_TR(u, n, 2, 8192) T_TR(u, n, 3, 9216)
#define T_LA(u, j, s) "buffer_load_dwordx4 a[200-40*" #u "+4*" #j ":203-40*" #u "+4*" #j "], %[av], %[ra], %[" #s "] offen offset:1024*(" #j "&3) sc0 sc1\n\t"
#define T_POLL "global_load_dword a240, %[fo], %[fb] sc0 sc1\n\t"
#define T_DMA(v, off) "buffer_load_dwordx4 %[" #v "], %[rw], %[t] offen offset:" #off " lds\n\t"
#define T_ST "s_add_u32 %[t], %[t], 31744\n\t"
#define T_SM "s_add_u32 m0, m0, 4096\n\ts_add_u32 %[t], %[t], 35840\n\t"
#define T_LW16 "s_mov_b32 m0, %[d]\n\ts_mov_b32 %[t], %[s]\n\ts_mov_b32 %[k], 4\n\ts_nop 2\n" \
        "1:\n\t" T_DMA(v0, 0) T_ST T_DMA(v0, 1024) T_ST T_DMA(v1, 2048) T_ST T_DMA(v1, 3072) T_SM \
        "s_sub_u32 %[k], %[k], 1\n\ts_cmp_lg_u32 %[k], 0\n\ts_cbranch_scc1 1b\n\t"
template <int NB>
__device__ __forceinline__ void gemm_wave(LAS char* lds, u64 xs, u64 myws, u64 pw, GAS unsigned* cnt, GAS float* otile,
                                          u64 psx, u64 psw, int rank, int t, int slab, int wv, int lane DIAG_PAR) {
    ST_DECL;
    const int b0 = tile_b0(t);
    const int g = lane >> 4, i16 = lane & 15;
    const unsigned lbase = (unsigned)(size_t)lds + wv * WAVE_B;
    const int r8 = lane >> 3, ch0 = ((lane & 7) ^ (r8 >> 1)) * 16;
    const unsigned wv0 = r8 * NN + slab * 128 + ch0, wv1 = wv0 ^ 64;
    const v4s rw = make_rsrc(pw, NN * NN);
    unsigned tmp, tmp2;
#define LW16(dst, so) asm volatile(T_LW16 : [t] "=&s"(tmp), [k] "=&s"(tmp2) \
        : [d] "s"(dst), [s] "s"(so), [rw] "s"(rw), [v0] "v"(wv0), [v1] "v"(wv1) : "memory", "scc")
    const unsigned wso0 = wv * 128 * NN;
#define WSO(src) (wso0 + ((unsigned)(src) << 21))
    LW16(lbase, WSO(rank));
    {
        const v4s rx = make_rsrc(xs, KC * KC);
        const unsigned v0 = r8 * KC + ch0, v1 = v0 ^ 64, so = b0 * 16 * KC + wv * 128, vo = i16 * KC + g * 16;
#define T_X4 T_DMA(v0, 0) "s_add_u32 %[t], %[t], 3072\n\t" T_DMA(v1, 1024) "s_add_u32 %[t], %[t], 3072\n\t" \
             T_DMA(v0, 2048) "s_add_u32 %[t], %[t], 3072\n\t" T_DMA(v1, 3072) "s_add_u32 m0, m0, 4096\n\ts_add_u32 %[t], %[t], 7168\n\t"
        asm volatile("s_mov_b32 m0, %[d]\n\ts_mov_b32 %[t], %[s]\n\t" T_X4 T_X4
                     ".if %c[nb] == 5\n\tbuffer_load_dwordx4 a[192:195], %[vo], %[rw], %[t] offen\n\t"
                     "buffer_load_dwordx4 a[196:199], %[vo], %[rw], %[t] offen offset:64\n\t.endif"
                     : [t] "=&s"(tmp) : [d] "s"(lbase + 2 * W_B), [s] "s"(so), [rw] "s"(rx), [v0] "v"(v0), [v1] "v"(v1),
                       [vo] "v"(vo), [nb] "n"(NB) : "memory", "scc");
    }
    unsigned srcs = 0;
#pragma unroll
    for (int r = 0; r < 7; ++r) srcs |= (unsigned)((rank + t - r + (t - r <= 0 ? 7 : 0)) & 7) << (4 * r);
#define SRC(r) ((srcs >> (4 * (r))) & 7)
    const float scale = *(const CAS float*)psx * *(const CAS float*)psw;
    const int wq = i16 >> 1, wf = (wq >> 1) | ((g & 1) << 2);
    unsigned boff[8];
#pragma unroll
    for (int n = 0; n < 8; ++n) boff[n] = (g * 16 + wq) * 128 + (i16 & 1) * 8 + ((((n + 2 * wv + 2) & 7) ^ wf) * 16);
    unsigned ta, cur = lbase;
    const unsigned flip = lbase ^ (lbase + W_B);
#define IN_RB [bs] "s"(cur), [b0] "v"(boff[0]), [b1] "v"(boff[1]), [b2] "v"(boff[2]), [b3] "v"(boff[3]), \
              [b4] "v"(boff[4]), [b5] "v"(boff[5]), [b6] "v"(boff[6]), [b7] "v"(boff[7])
#define T_RB(u, n) T_RB0(u, n) T_RB1(u, n)
#define RB_ALL(u) asm volatile(T_RB(u, 0) T_RB(u, 1) T_RB(u, 2) T_RB(u, 3) T_RB(u, 4) T_RB(u, 5) T_RB(u, 6) T_RB(u, 7) \
        "s_waitcnt lgkmcnt(0)" : [ta] "=&v"(ta) : IN_RB : "memory")
#define MM(u, m, n) asm volatile("v_mfma_f32_16x16x128_f8f6f4 a[%c0:%c0+3], v[%c1:%c1+7], a[%c2:%c2+7], a[%c0:%c0+3] cbsz:1 blgp:1" \
        ::"n"(ACC(m, n)), "n"(WB(u, n)), "n"(XA(u, m)))
    unsigned fv;
    {
        const unsigned x0 = lbase + 2 * W_B + i16 * 128 + ((g ^ (i16 >> 1)) * 16), x1 = x0 ^ 64;
        asm volatile("global_load_dword %[f], %[z], %[cnt] sc0 sc1\n\t" T_LW16
                     ".set k_, 0\n\t.rept %c[na]\n\tv_accvgpr_write_b32 a[k_], 0\n\t.set k_, k_ + 1\n\t.endr\n\t"
                     "v_accvgpr_write_b32 a240, 0\n\ts_waitcnt vmcnt(16)\n\t"
                     ".set k_, 0\n\t.rept 4\n\tds_read_b128 a[160+8*k_:163+8*k_], %[x0] offset:2048*k_\n\t"
                     "ds_read_b128 a[164+8*k_:167+8*k_], %[x1] offset:2048*k_\n\t.set k_, k_ + 1\n\t.endr"
                     : [f] "=&v"(fv), [t] "=&s"(tmp), [k] "=&s"(tmp2)
                     : [z] "v"(0), [cnt] "s"(cnt), [d] "s"(lbase + W_B), [s] "s"(WSO(SRC(0))), [rw] "s"(rw), [v0] "v"(wv0),
                       [v1] "v"(wv1), [na] "n"(NB * 32), [x0] "v"(x0), [x1] "v"(x1)
                     : "memory", "scc", "a0", "a255", "v128", "v255");
    }
    RB_ALL(1);
    STAMP(1); CSTAMP(101);
    LW16(lbase, WSO(SRC(1)));
    const unsigned call = uni(fv) + 1;
    const unsigned tstart = (unsigned)__builtin_amdgcn_s_memrealtime();
    const v4s ra = make_rsrc(myws, 2 * SLOT_B);
    const unsigned av16 = lane * 16;
    const unsigned aso0 = (call & 1) * (unsigned)SLOT_B + wv * 65536 + b0 * 2048;
    const u64 fbase = myws + FLAG_OFF + t * 4096 + wv * 128;
    const unsigned foff = SRC(lane < 7 ? lane : 0) * 512;
#define IN_POLL [fo] "v"(foff), [fb] "s"(fbase)
#define IN_LA [av] "v"(av16), [ra] "s"(ra), [s0] "s"(aso), [s1] "s"(aso + 4096), [s2] "s"(aso + 8192)
#define NOTE() { asm volatile("v_accvgpr_read_b32 %0, a240" : "=v"(fv)::"memory"); av |= (unsigned)__ballot(fv >= call) & 0x7f; }
    int ua = 0, ub = 1, nxt;
    unsigned rest = 0x7c, av = 0, late = 0;
    unsigned aso, wso, more;
#define T_N ""
#define T_A(u, L0, L1, L2, L3, L4, L5, L6, L7, L8, L9) \
        T_MM(u, 3) L0 T_RB0(u, 0) T_MM(u, 4) L1 T_RB1(u, 0) T_MM(u, 5) L2 T_RB0(u, 1) T_MM(u, 6) L3 T_RB1(u, 1) \
        T_MM(u, 7) L4 T_RB0(u, 2) T_MM(u, 8) L5 T_RB1(u, 2) T_MM(u, 9) L6 T_RB0(u, 3) T_MM(u, 10) L7 T_RB1(u, 3) \
        T_MM(u, 11) L8 T_RB0(u, 4) T_MM(u, 12) L9 T_RB1(u, 4) T_MM(u, 13) T_RB0(u, 5) T_MM(u, 14) T_RB1(u, 5) \
        T_MM(u, 15) T_RB0(u, 6) T_MM(u, 16) T_RB1(u, 6) T_MM(u, 17) T_RB0(u, 7) T_MM(u, 18) T_RB1(u, 7)
#define T_IF5(x) ".if %c[nb] == 5\n\t" x ".endif\n\t"
#define T_IF4(x) ".if %c[nb] == 4\n\t" x ".endif\n\t"
#define T_C4(u, k) T_MM(u, k) T_DMA(v0, 0) T_ST T_MM(u, k + 1) T_DMA(v0, 1024) T_ST T_MM(u, k + 2) T_DMA(v1, 2048) T_ST \
                   T_MM(u, k + 3) T_DMA(v1, 3072) T_SM
#define T_C2(u, k) T_MM(u, k) T_DMA(v0, 0) T_ST T_DMA(v0, 1024) T_ST T_MM(u, k + 1) T_DMA(v1, 2048) T_ST T_DMA(v1, 3072) T_SM
#define T_M4(u, k) T_MM(u, k) T_MM(u, k + 1) T_MM(u, k + 2) T_MM(u, k + 3)
#define UNIT(u) { \
        SB0; MM(u, 0, 0); SB0; \
          \
        unsigned sw = (av >> ub) & ~(av >> ua) & 1; \
        nxt = sw ? ub : ua; ua = sw ? ua : ub; cur ^= sw ? 0 : flip; PIN_S(nxt); PIN_S(ua); PIN_S(cur); \
        SB0; MM(u, 0, 1); SB0; \
        aso = aso0 + (SRC(nxt) << 18); PIN_S(aso); \
        if (__builtin_expect(sw, 0)) waitv<0>(); \
        SB0; MM(u, 0, 2); SB0; \
        if (__builtin_expect((av >> nxt) & 1, 1)) { \
            asm volatile(T_A(u, T_LA(u, 0, s0), T_LA(u, 1, s0), T_LA(u, 2, s0), T_LA(u, 3, s0), T_LA(u, 4, s1), T_LA(u, 5, s1), \
                             T_LA(u, 6, s1), T_LA(u, 7, s1), T_IF5(T_LA(u, 8, s2)), T_IF5(T_LA(u, 9, s2))) T_MM(u, 19) T_MM(u, 20) \
                         : [ta] "=&v"(ta) : IN_RB, IN_LA, [nb] "n"(NB) : "memory"); \
        } else { \
            asm volatile(T_A(u, T_POLL, T_N, T_N, T_N, T_N, T_N, T_N, T_N, T_N, T_N) "s_waitcnt vmcnt(0)" \
                         : [ta] "=&v"(ta) : IN_RB, IN_POLL : "memory"); \
            NOTE(); \
            if ((av >> nxt) & 1) \
                asm volatile(T_MM(u, 19) T_LA(u, 0, s0) T_LA(u, 1, s0) T_LA(u, 2, s0) T_LA(u, 3, s0) T_LA(u, 4, s1) T_MM(u, 20) \
                             T_LA(u, 5, s1) T_LA(u, 6, s1) T_LA(u, 7, s1) T_IF5(T_LA(u, 8, s2) T_LA(u, 9, s2)) \
                             ::IN_LA, [nb] "n"(NB) : "memory"); \
            else asm volatile(T_MM(u, 19) T_MM(u, 20)); \
        } \
        unsigned went = (av >> nxt) & 1; PIN_S(went); \
        more = rest; \
        unsigned pick = rest & av ? rest & av : rest & ~late ? rest & ~late : rest | 0x80; PIN_S(pick); \
        SB0; MM(u, 2, 5); SB0; \
        ub = __builtin_ctz(pick); rest &= ~(1u << ub); wso = WSO(SRC(ub)); PIN_S(rest); PIN_S(wso); \
        SB0; MM(u, 2, 6); SB0; \
        if (more) asm volatile(T_MM(u, 23) T_POLL "s_mov_b32 m0, %[d]\n\ts_mov_b32 %[t], %[s]\n\ts_waitcnt lgkmcnt(0)\n\t" \
                               T_IF5(T_C4(u, 24) T_C4(u, 28) T_C4(u, 32) T_C4(u, 36)) \
                               T_IF4(T_C2(u, 24) T_C2(u, 26) T_C2(u, 28) T_C2(u, 30)) "s_waitcnt vmcnt(16)" \
                               : [t] "=&s"(tmp) : IN_POLL, [d] "s"(cur), [s] "s"(wso), [rw] "s"(rw), [v0] "v"(wv0), [v1] "v"(wv1), \
                                 [nb] "n"(NB) : "memory", "scc"); \
        else asm volatile(T_MM(u, 23) T_POLL T_M4(u, 24) T_M4(u, 28) T_IF5(T_M4(u, 32) T_M4(u, 36)) \
                          "s_waitcnt vmcnt(0) lgkmcnt(0)" ::IN_POLL, [nb] "n"(NB) : "memory"); \
        NOTE(); \
        if (__builtin_expect(!went, 0)) STALL(u); }
#define STALL(u) { int pat = 0; \
        while (!((av >> nxt) & 1)) { \
            const unsigned ina = (av >> ua) & 1, inb = (av >> ub) & 1; \
            if ((ina | inb | (av & rest)) && ++pat >= PATIENCE) { \
                int ld; \
                unsigned to = cur; \
                if (ina | inb) { \
                    waitv<0>(); \
                    if (ina) { cur ^= flip; const int x_ = ua; ua = ub; ub = x_; } \
                    RB_ALL(u); \
                    late |= 1u << nxt; \
                    ld = nxt; nxt = ub; ub = ld; to = cur; \
                    if (ua == 7) { ua = ub; ub = 7; to = cur ^ flip; } \
                } else { \
                    ld = __builtin_ctz(av & rest); \
                    rest = (rest | (1u << ub)) & ~(1u << ld); late |= 1u << ub; ub = ld; \
                } \
                LW16(to, WSO(SRC(ld))); \
                continue; \
            } \
            if ((unsigned)__builtin_amdgcn_s_memrealtime() - tstart > DEADLINE) break; \
            asm volatile(T_POLL "s_waitcnt vmcnt(0)" ::IN_POLL : "memory"); \
            NOTE(); \
        } \
        STAMP(36 + nxt * 4 + wv); \
        aso = aso0 + (SRC(nxt) << 18); \
        if ((av >> nxt) & 1) { \
            asm volatile(T_LA(u, 0, s0) T_LA(u, 1, s0) T_LA(u, 2, s0) T_LA(u, 3, s0) T_LA(u, 4, s1) T_LA(u, 5, s1) T_LA(u, 6, s1) \
                         T_LA(u, 7, s1) T_IF5(T_LA(u, 8, s2) T_LA(u, 9, s2)) T_POLL "s_waitcnt vmcnt(0)" \
                         ::IN_LA, IN_POLL, [nb] "n"(NB) : "memory"); \
            NOTE(); \
        } else {      \
            asm volatile(".set k_, %c1\n\t.rept %c2\n\tv_accvgpr_write_b32 a[k_], %0\n\t.set k_, k_ + 8\n\t.endr\n\ts_waitcnt vmcnt(0)\n\ts_nop 1" \
                         ::"v"(0x7f7f7f7f), "n"(XA(1 - (u), 0)), "n"(NB) : "memory"); \
        } }
    waitv<16>();
#pragma unroll 1
    for (int it = 0;; ++it) {
        UNIT(0);
        STAMP(64 + it * 8 + wv);
        if (it == 3) break;
        UNIT(1);
        STAMP(68 + it * 8 + wv);
    }
    const unsigned mine = lbase + lane * 16;
#define PUT(d, m, n) if constexpr ((d) + 0 * NB > 0) asm volatile("ds_write_b128 %0, a[%c1:%c1+3] offset:%c2" \
        ::"v"(mine), "n"(ACC(m, (2 * (d) - 2 + (n)) & 7)), "n"((((d) - 1) & 3) * PART_B + (m) * 2048 + (n) * 1024) : "memory")
#define LAST1(d, m) if constexpr ((m) < NB) { MM(1, m, 2 * (d)); PUT(d, m, 0); MM(1, m, 2 * (d) + 1); PUT(d, m, 1); }
#define LAST(d) LAST1(d, 0) LAST1(d, 1) LAST1(d, 2) LAST1(d, 3) LAST1(d, 4)
    LAST(0) LAST(1) LAST(2) LAST(3)
    asm volatile("s_waitcnt lgkmcnt(0)\n\ts_barrier\n\ts_nop 9" ::: "memory");
    if (wv == 0 && lane == 0) __hip_atomic_store(cnt, call, __ATOMIC_RELAXED, __HIP_MEMORY_SCOPE_AGENT);
    GAS float* orow = otile + (size_t)i16 * NN + wv * 32 + g * 4;
    const float l2e = 1.4426950408889634f, se = -scale * l2e, lim = 60.0f * l2e;
    const LAS char* const p0 = lds + ((wv + 3) & 3) * WAVE_B + lane * 16;
    const LAS char* const p1 = lds + ((wv + 2) & 3) * WAVE_B + PART_B + lane * 16;
    const LAS char* const p2 = lds + ((wv + 1) & 3) * WAVE_B + 2 * PART_B + lane * 16;
#define OUT1(m, n) if constexpr ((m) < NB) { \
        constexpr int o = (m) * 2048 + (n) * 1024; \
        float c0, c1, c2, c3; \
        asm volatile("v_accvgpr_read_b32 %0, a[%c4]\n\tv_accvgpr_read_b32 %1, a[%c4+1]\n\tv_accvgpr_read_b32 %2, a[%c4+2]\n\t" \
                     "v_accvgpr_read_b32 %3, a[%c4+3]" : "=v"(c0), "=v"(c1), "=v"(c2), "=v"(c3) : "n"(ACC(m, 6 + (n)))); \
        const v4f a = ((v4f){c0, c1, c2, c3} + *(const LAS v4f*)(p0 + o)) + (*(const LAS v4f*)(p1 + o) + *(const LAS v4f*)(p2 + o)); \
          \
        const v4f e = a * se; \
        v4f dn; \
        _Pragma("unroll") for (int r = 0; r < 4; ++r) dn[r] = __builtin_amdgcn_exp2f(__builtin_amdgcn_fmed3f(e[r], -lim, lim)); \
        dn = dn + 1.0f; \
        _Pragma("unroll") for (int r = 0; r < 4; ++r) dn[r] = __builtin_amdgcn_rcpf(dn[r]); \
        const v4f y = (a * scale) * dn; \
        asm volatile("global_store_dwordx4 %0, %1, off offset:%2" OUT_POL "\n\ts_nop 1" \
                     ::"v"(orow + (size_t)(m) * 16 * NN), "v"(y), "n"((n) * 64)); }
    OUT1(0, 0) OUT1(0, 1) OUT1(1, 0) OUT1(1, 1) OUT1(2, 0) OUT1(2, 1) OUT1(3, 0) OUT1(3, 1) OUT1(4, 0) OUT1(4, 1)
}
}

__global__ __launch_bounds__(NTHREADS) void a2a_gemm_kernel(Params p) {
    __shared__ __attribute__((aligned(128))) char smem[LDS_B];
    LAS char* lds = (LAS char*)smem;
    const int b = blockIdx.x;
    const int wv = __builtin_amdgcn_readfirstlane(threadIdx.x >> 6), lane = threadIdx.x & 63;
    const int xg = b & 7, idx = b >> 3;
    const int srel = b < NSEND_WG ? (xg + 2 * idx) & 7 : 0;
    u64 xs = (u64)p.xsend[srel], dws = (u64)p.wsdst[srel], myws = (u64)p.wsdst[0], pw = (u64)p.w, psx = (u64)p.sx,
        psw = (u64)p.sw, pout = (u64)p.out;
    int rank = p.rank;
    asm volatile("" : "+s"(xs), "+s"(dws), "+s"(myws), "+s"(pw), "+s"(psx), "+s"(psw), "+s"(pout), "+s"(rank));
    GAS unsigned* const cnt = (GAS unsigned*)(myws + CNT_OFF) + b;

    if (__builtin_expect(b < NSEND_WG, 1)) {
        ST_DECL;
        const int q = idx, rel = srel;
        if (rel == 0) return;
        const int t0 = rel == 7 ? 0 : rel;
        LAS unsigned* const prog = (LAS unsigned*)lds;
        if (__builtin_expect(wv < 3, 1)) {
            constexpr int WIN = SEND_WIN;
            static_assert(WIN % 2 == 0 && WIN >= 2 && WIN <= 8, "");
            const int r8 = lane >> 3, ch0 = ((lane & 7) ^ (r8 >> 1)) * 16, g = lane >> 4, i16 = lane & 15;
            const unsigned v0 = r8 * KC + q * 128 + ch0, v1 = (v0 ^ 64) + 8 * KC - 1024, gw = tile_b0(t0) + wv;
            const unsigned lb = (unsigned)(size_t)lds + 1024 + wv * 22528;
            const unsigned x0 = lb + i16 * 128 + ((g ^ (i16 >> 1)) * 16), x1 = x0 ^ 64;
            const v4s rx = make_rsrc(xs, KC * KC);
            const u64 sd = dws + (size_t)(rank * 4 + q) * 65536, sdl = wv == 2 ? myws + (size_t)(rank * 4 + q) * 65536 : sd;
            unsigned vd = lane * 16, vt, st, vc;
            u32x4 v[22];
#define GRP(i, sh) "s_add_u32 %[st], %[g], " #i "*3\n\ts_and_b32 %[st], %[st], 31\n\ts_lshl_b32 %[st], %[st], " #sh "\n\t"
#define XLD(i) "s_add_u32 m0, %[lb], 2048*" #i "\n\t" GRP(i, 13) "buffer_load_dwordx4 %[v0], %[rx], %[st] offen lds\n\t" \
               "buffer_load_dwordx4 %[v1], %[rx], %[st] offen offset:1024 lds\n\t"
#define XRD(i, a, c) "ds_read_b128 %[" #a "], %[x0] offset:2048*" #i "\n\tds_read_b128 %[" #c "], %[x1] offset:2048*" #i "\n\t"
#define XST(i, a, c) ".if %c[win] > 2*" #i "\n\ts_waitcnt vmcnt(20)\n\t" XRD(i, a, c) GRP(i, 11) "v_add_u32 %[vt], %[st], %[vd]\n\t" \
                     "s_waitcnt lgkmcnt(0)\n\tglobal_store_dwordx4 %[vt], %[" #a "], %[sd] sc0 sc1\n\t" \
                     "global_store_dwordx4 %[vt], %[" #c "], %[sd] offset:1024 sc0 sc1\n\t.endif\n\t"
#define XRL(i, a, c) ".if %c[win] <= 2*" #i "\n\t" XRD(i, a, c) ".endif\n\t"
            asm volatile("global_load_dword %[vc], %[z], %[cnt] sc0 sc1\n\t"
                         XLD(0) XLD(1) XLD(2) XLD(3) XLD(4) XLD(5) XLD(6) XLD(7) XLD(8) XLD(9) XLD(10)
                         "s_waitcnt vmcnt(22)\n\tv_not_b32 %[vc], %[vc]\n\tv_and_b32 %[vc], 1, %[vc]\n\t"
                         "v_lshl_add_u32 %[vd], %[vc], 21, %[vd]\n\t"
                         XST(0, d0, d1) XST(1, d2, d3) XST(2, d4, d5) XST(3, d6, d7)
                         "s_barrier\n\ts_waitcnt vmcnt(%c[win])\n\t"
                         XRL(1, d2, d3) XRL(2, d4, d5) XRL(3, d6, d7) "s_waitcnt lgkmcnt(0)"
                         : [d0] "=&v"(v[0]), [d1] "=&v"(v[1]), [d2] "=&v"(v[2]), [d3] "=&v"(v[3]), [d4] "=&v"(v[4]),
                           [d5] "=&v"(v[5]), [d6] "=&v"(v[6]), [d7] "=&v"(v[7]), [vc] "=&v"(vc), [vt] "=&v"(vt),
                           [st] "=&s"(st), [vd] "+v"(vd)
                         : [z] "v"(0), [cnt] "s"(cnt), [v0] "v"(v0), [v1] "v"(v1), [rx] "s"(rx), [lb] "s"(lb), [x0] "v"(x0),
                           [x1] "v"(x1), [sd] "s"(sd), [g] "s"(gw), [win] "n"(WIN) : "memory", "scc");
#pragma unroll
            for (int i = 4; i < 11; ++i) {
                v[2 * i] = *(const LAS u32x4*)(size_t)(x0 + 2048 * i);
                v[2 * i + 1] = *(const LAS u32x4*)(size_t)(x1 + 2048 * i);
            }
            static_assert(SLOT_B == 1u << 21, "");
            STAMP(16 + wv);
#define SEND_STEP(I) { waitv<(2 * (I) + WIN < 22 ? WIN - 2 : 20 - 2 * (I))>(); if (lane == 0) lds_st(prog + wv, (I) + 1); \
            if (2 * (I) + WIN < 22) asm volatile("global_store_dwordx4 %0, %1, %3 sc0 sc1\n\t" \
                "global_store_dwordx4 %0, %2, %3 offset:1024 sc0 sc1\n\ts_nop 1" \
                ::"v"(vd + ((gw + 3 * ((I) + WIN / 2)) & 31) * 2048), "v"(v[(2 * (I) + WIN) % 22]), \
                  "v"(v[(2 * (I) + WIN + 1) % 22]), "s"((I) + WIN / 2 == 10 ? sdl : sd) : "memory"); }
            SEND_STEP(0) SEND_STEP(1) SEND_STEP(2) SEND_STEP(3) SEND_STEP(4) SEND_STEP(5)
            SEND_STEP(6) SEND_STEP(7) SEND_STEP(8) SEND_STEP(9) SEND_STEP(10)
        } else {
            if (lane < 3) lds_st(prog + lane, 0);
            const unsigned call = uni(__hip_atomic_load(cnt, __ATOMIC_RELAXED, __HIP_MEMORY_SCOPE_AGENT)) + 1;
            asm volatile("s_waitcnt lgkmcnt(0)" ::: "memory");
            __builtin_amdgcn_s_barrier();
            const int l3 = lane < 3 ? lane : 0;
            int e = 0, t = t0;
#pragma unroll 1
            for (int r = 0; r < 7; ++r) {
                e += tile_nb(t);
                while (!__all((int)(lds_ld(prog + l3) * 3 + l3) >= e)) __builtin_amdgcn_s_sleep(1);
                if (lane == 0)
                    __hip_atomic_store((GAS unsigned*)(dws + FLAG_OFF + ((t * 8 + rank) * 4 + q) * 128), call,
                                       __ATOMIC_RELAXED, __HIP_MEMORY_SCOPE_SYSTEM);
                STAMP(1 + r);
                t = t == 6 ? 0 : t + 1;
            }
            if (lane == 0) __hip_atomic_store(cnt, call, __ATOMIC_RELAXED, __HIP_MEMORY_SCOPE_AGENT);
        }
        DUMP();
        return;
    }

    const int ci = idx - 4, sl = ci / 7, t = ci - sl * 7;
    const int slab = sl * 8 + xg;
    GAS float* const otile = (GAS float*)pout + (size_t)(tile_b0(t) * 16) * NN + slab * 128;
    if (t < 4) gemm_wave<5>(lds, xs, myws, pw, cnt, otile, psx, psw, rank, t, slab, wv, lane DIAG_ARG);
    else gemm_wave<4>(lds, xs, myws, pw, cnt, otile, psx, psw, rank, t, slab, wv, lane DIAG_ARG);
}

extern "C" void kernel_launch(int rank, int nranks, ncclComm_t comm, hipStream_t stream,
                              void* const* d_in, const int* in_sizes, int n_in,
                              void* d_out, int out_size, const int* in_dtypes, int out_dtype,
                              void* const* d_out_all, void* const* d_ws_all, size_t ws_bytes) {
    (void)comm; (void)in_dtypes; (void)out_dtype; (void)d_out_all;
    if (nranks != NR || (unsigned)rank >= (unsigned)NR || n_in != 4 || ws_bytes < WS_NEED ||
        in_sizes[0] != NN * KC || in_sizes[1] != NN * NN || out_size != KC * NN) {
        fprintf(stderr, "unexpected problem shape\n");
        return;
    }
    Params p;
    for (int rel = 0; rel < NR; ++rel) {
        const int d = (rank - rel) & 7;
        p.xsend[rel] = (const char*)d_in[0] + (size_t)d * KC * KC;
        p.wsdst[rel] = (char*)d_ws_all[d];
    }
    p.w = (const char*)d_in[1];
    p.sx = (const float*)d_in[2];
    p.sw = (const float*)d_in[3];
    p.out = (float*)d_out;
    p.rank = rank;
    p.pad = 0;
    a2a_gemm_kernel<<<256, NTHREADS, 0, stream>>>(p);
    HIP_CHECK(hipGetLastError());
}
```

```cpp
#include <hip/hip_runtime.h>
#include <rccl/rccl.h>
#include <cstdio>

#define HIP_CHECK(expr) do { hipError_t _e = (expr); if (_e != hipSuccess) { \
    fprintf(stderr, "HIP error %s at %s:%d\n", hipGetErrorString(_e), __FILE__, __LINE__); \
    return; } } while (0)

typedef unsigned u32x4 __attribute__((ext_vector_type(4)));
typedef int v2i __attribute__((ext_vector_type(2)));
typedef int v4i __attribute__((ext_vector_type(4)));
typedef int v8i __attribute__((ext_vector_type(8)));
typedef float v4f __attribute__((ext_vector_type(4)));
typedef unsigned long long u64;
#define LAS __attribute__((address_space(3)))
#define GAS __attribute__((address_space(1)))
#define CAS __attribute__((address_space(4)))

#define SEND_WIN 4
#define PATIENCE 2
#define SLABMAP 1
#define VMAP 0
#define UNIT_V(m, c) (7 * (c) + (m))
#define UNIT_M(t, q) (t)
#define EPIV 0
#define OUT_MODE 1
#define OUT_POL " sc1"

namespace {
constexpr int NR = 8;
constexpr int KC = 512;
constexpr int NN = 4096;
constexpr size_t SLOT_B = 2u << 20;
constexpr size_t FLAG_OFF = 2 * SLOT_B;
constexpr size_t CNT_OFF = FLAG_OFF + 32768;
constexpr size_t DIAG_OFF = CNT_OFF + 4096;
constexpr size_t WS_NEED = DIAG_OFF + 256 * 128 * 8;
constexpr unsigned DEADLINE = 20000000u;
constexpr int W_B = 16384, WAVE_B = 2 * W_B, XS_OFF = 4 * WAVE_B, XS_B = 8192;
constexpr int LDS_B = XS_OFF + 4 * XS_B;
constexpr int NTHREADS = 256;
constexpr int NSEND_WG = 32;

struct Params {
    const char* xsend[NR];
    char* wsdst[NR];
    const char* w;
    const float* sx;
    const float* sw;
    float* out;
    int rank;
    int pad;
};

template <int N> __device__ __forceinline__ void waitv() { asm volatile("s_waitcnt vmcnt(%0)" ::"n"(N) : "memory"); }
constexpr __host__ __device__ int tile_b0(int t) { return t <= 3 ? t * 4 : t * 5 - 3; }
constexpr __host__ __device__ int tile_nb(int t) { return t < 3 ? 4 : 5; }
struct SendTab { unsigned char v[NSEND_WG][4][16]; };
constexpr SendTab make_tab() {
    SendTab s = {};
    for (int b = 0; b < NSEND_WG; ++b) {
        const int c = b >> 3, rel = ((b & 7) + 2 * c) & 7;
        int p = 0;
        for (int r = 0; r < 7; ++r) {
            const int V = UNIT_V((rel + r) % 7, c), q = V / 7, t = V % 7;
            for (int k = 0; k < tile_nb(t); ++k, ++p) s.v[b][p % 3][p / 3] = (unsigned char)(32 * q + tile_b0(t) + k);
            s.v[b][3][r] = (unsigned char)(4 * t + q);
            s.v[b][3][8 + r] = (unsigned char)p;
        }
    }
    return s;
}
__device__ const __attribute__((aligned(16))) SendTab kSend = make_tab();
#define WG_RLX __ATOMIC_RELAXED, __HIP_MEMORY_SCOPE_WORKGROUP
__device__ __forceinline__ unsigned lds_ld(LAS unsigned* p) { return __hip_atomic_load(p, WG_RLX); }
__device__ __forceinline__ void lds_st(LAS unsigned* p, unsigned v) { __hip_atomic_store(p, v, WG_RLX); }
__device__ __forceinline__ unsigned uni(unsigned v) { return __builtin_amdgcn_readfirstlane(v); }
typedef int v4s __attribute__((ext_vector_type(4)));
__device__ __forceinline__ v4s make_rsrc(u64 a, unsigned bytes) {
    return (v4s){(int)(unsigned)a, (int)((unsigned)(a >> 32) & 0xffffu), (int)bytes, 0x00020000};
}
#define SB0 __builtin_amdgcn_sched_barrier(0)
#define CFENCE asm volatile("" ::: "memory")
#define PIN_S(x) asm volatile("" : "+s"(x))
#define PIN_V(x) asm volatile("" : "+v"(x))

#define ST_DECL
#define STAMP(i)
#define CSTAMP(i)
#define DUMP()
#define DIAG_T
#define DIAG_TO
#define DIAG_PAR
#define DIAG_ARG

#define ACC(m, n) (((m) * 8 + (n)) * 4)
#define XA(u, m) (160 + (u) * 40 + (m) * 8)
#define WB(u, n) (128 + (u) * 64 + (n) * 8)
#define V10(x) "v" #x "0", "v" #x "1", "v" #x "2", "v" #x "3", "v" #x "4", "v" #x "5", "v" #x "6", "v" #x "7", "v" #x "8", "v" #x "9"
#define CLOB_WB "v128", "v129", V10(13), V10(14), V10(15), V10(16), V10(17), V10(18), V10(19), V10(20), V10(21), V10(22), V10(23), \
        V10(24), "v250", "v251", "v252", "v253", "v254", "v255"
#define T_MM(u, k) "v_mfma_f32_16x16x128_f8f6f4 a[4*(" #k "):4*(" #k ")+3], v[128+64*" #u "+8*((" #k ")&7):135+64*" #u "+8*((" #k ")&7)], " \
        "a[160+40*" #u "+8*((" #k ")>>3):167+40*" #u "+8*((" #k ")>>3)], " T_C(k) " cbsz:1 blgp:1\n\t"
#define T_C(k) "a[4*(" #k "):4*(" #k ")+3]"
#define T_TR(u, n, r, off) "ds_read_b64_tr_b8 v[192-64*" #u "+8*" #n "+2*" #r ":193-64*" #u "+8*" #n "+2*" #r "], %[ta] offset:" #off "\n\t"
#define T_RB0(u, n) "v_add_u32 %[ta], %[bs], %[b" #n "]\n\t" T_TR(u, n, 0, 0) T_TR(u, n, 1, 1024)
#define T_RB1(u, n) T_TR(u, n, 2, 8192) T_TR(u, n, 3, 9216)
#define T_LA(u, j, s) "buffer_load_dwordx4 a[200-40*" #u "+4*" #j ":203-40*" #u "+4*" #j "], %[av], %[ra], %[" #s "] offen offset:1024*(" #j "&3) sc0 sc1\n\t"
#define T_POLL "global_load_dword a240, %[fo], %[fb] sc0 sc1\n\t"
#define T_DMA(v, off) "buffer_load_dwordx4 %[" #v "], %[rw], %[t] offen offset:" #off " lds\n\t"
#define T_ST "s_add_u32 %[t], %[t], 31744\n\t"
#define T_GRP(j) "s_xor_b32 m0, %[d], 4096*" #j "\n\ts_xor_b32 %[t], %[s], 131072*" #j "\n\t"
#define T_G4 T_DMA(v0, 0) T_ST T_DMA(v0, 1024) T_ST T_DMA(v1, 2048) T_ST T_DMA(v1, 3072)
template <int NB>
__device__ __forceinline__ void gemm_wave(LAS char* lds, u64 xs, u64 myws, u64 pw, GAS unsigned* cnt, GAS float* otile,
                                          u64 psx, u64 psw, int rank, int t, int slab, int sl, int wv, int lane DIAG_PAR) {
    ST_DECL;
    const int b0 = tile_b0(t);
    const int g = lane >> 4, i16 = lane & 15;
    const unsigned lbase = (unsigned)(size_t)lds + wv * WAVE_B, xbase = (unsigned)(size_t)lds + XS_OFF + wv * XS_B;
    const int r8 = lane >> 3, ch0 = ((lane & 7) ^ (r8 >> 1)) * 16;
    const unsigned wv0 = r8 * NN + slab * 128 + ch0, wv1 = wv0 ^ 64;
    const v4s rw = make_rsrc(pw, NN * NN);
    unsigned tmp, tmp2;
#define LW16(dst, so) asm volatile(T_GRP(0) T_G4 T_GRP(1) T_G4 T_GRP(2) T_G4 T_GRP(3) T_G4 : [t] "=&s"(tmp) \
        : [d] "s"(dst), [s] "s"(so), [rw] "s"(rw), [v0] "v"(wv0), [v1] "v"(wv1) : "memory", "scc")
    const int g0 = wv ^ (t & 3);
    const unsigned wso0 = wv * 128 * NN + (g0 << 17), fdst = W_B + (g0 << 12);
#define WSO(src) (wso0 + ((unsigned)(src) << 21))
    const int gm = UNIT_M(t, wv);
    unsigned srcs = 0;
#pragma unroll
    for (int r = 0; r < 7; ++r) srcs |= (unsigned)((rank + gm - r + (gm - r <= 0 ? 7 : 0)) & 7) << (4 * r);
#define SRC(r) ((srcs >> (4 * (r))) & 7)
    {
        const v4s rx = make_rsrc(xs, KC * KC);
        const unsigned x0 = r8 * KC + ch0, x1 = (x0 ^ 64) + 8 * KC - 1024, xo = b0 * 16 * KC + wv * 128, vo = i16 * KC + g * 16;
#define T_XB(j) "s_add_u32 %[k], %[xr], " #j "\n\ts_and_b32 %[k], %[k], 3\n\ts_lshl_b32 %[t], %[k], 11\n\ts_add_u32 m0, %[xd], %[t]\n\t" \
        "s_lshl_b32 %[k], %[k], 13\n\ts_add_u32 %[t], %[xo], %[k]\n\tbuffer_load_dwordx4 %[x0], %[rx], %[t] offen lds\n\t" \
        "buffer_load_dwordx4 %[x1], %[rx], %[t] offen offset:1024 lds\n\t"
#define T_WO(j) T_GRP(j) T_G4
#define T_WU(j, v) "s_xor_b32 m0, %[du], 2048*" #j "\n\ts_xor_b32 %[t], %[su], 65536*" #j "\n\t" T_DMA(v, 0) T_ST T_DMA(v, 1024)
#define T_Z(j) ".set k_, 4*%c[nb]*" #j "\n\t.rept 4*%c[nb]\n\tv_accvgpr_write_b32 a[k_], 0\n\t.set k_, k_ + 1\n\t.endr\n\t"
        asm volatile("global_load_dword a240, %[z], %[cnt] sc0 sc1\n\t"
                     T_XB(0) T_WO(0)
                     ".if %c[nb] == 5\n\ts_add_u32 %[t], %[xo], 32768\n\tbuffer_load_dwordx4 a[192:195], %[vo], %[rx], %[t] offen\n\t"
                     "buffer_load_dwordx4 a[196:199], %[vo], %[rx], %[t] offen offset:64\n\t.endif\n\t"
                     T_WU(0, v0) T_WU(1, v1) T_Z(0) T_Z(1) T_Z(2) T_Z(3) T_Z(4) T_Z(5) T_Z(6) T_Z(7)
                     T_XB(1) T_XB(2) T_XB(3) T_WO(1) T_WO(2) T_WO(3) T_WU(2, v0) T_WU(3, v1) T_WU(4, v0) T_WU(5, v1) T_WU(6, v0) T_WU(7, v1)
                     : [t] "=&s"(tmp), [k] "=&s"(tmp2)
                     : [xd] "s"(xbase), [xo] "s"(xo), [xr] "s"(sl), [rx] "s"(rx), [x0] "v"(x0), [x1] "v"(x1), [vo] "v"(vo),
                       [d] "s"(lbase ^ (g0 << 12)), [s] "s"(WSO(rank)), [du] "s"(lbase ^ fdst),
                       [su] "s"(WSO(SRC(0))), [rw] "s"(rw), [v0] "v"(wv0),
                       [v1] "v"(wv1), [nb] "n"(NB), [z] "v"(0), [cnt] "s"(cnt) : "memory", "scc", "a0", "a255");
    }
    const float scale = *(const CAS float*)psx * *(const CAS float*)psw;
    const int wq = i16 >> 1, wf = (wq >> 1) | ((g & 1) << 2);
    unsigned boff[8];
#pragma unroll
    for (int n = 0; n < 8; ++n) boff[n] = (g * 16 + wq) * 128 + (i16 & 1) * 8 + ((((n + 2 * wv + 2) & 7) ^ wf) * 16);
    unsigned ta, sa = lbase;
    constexpr unsigned flip = W_B;
#define IN_RB [bs] "s"(sa), [b0] "v"(boff[0]), [b1] "v"(boff[1]), [b2] "v"(boff[2]), [b3] "v"(boff[3]), \
              [b4] "v"(boff[4]), [b5] "v"(boff[5]), [b6] "v"(boff[6]), [b7] "v"(boff[7])
#define T_RB(u, n) T_RB0(u, n) T_RB1(u, n)
#define RB_ALL(u) asm volatile(T_RB(u, 0) T_RB(u, 1) T_RB(u, 2) T_RB(u, 3) T_RB(u, 4) T_RB(u, 5) T_RB(u, 6) T_RB(u, 7) \
        "s_waitcnt lgkmcnt(0)" : [ta] "=&v"(ta) : IN_RB : "memory", CLOB_WB)
#define MM(u, m, n) asm volatile("v_mfma_f32_16x16x128_f8f6f4 a[%c0:%c0+3], v[%c1:%c1+7], a[%c2:%c2+7], a[%c0:%c0+3] cbsz:1 blgp:1" \
        ::"n"(ACC(m, n)), "n"(WB(u, n)), "n"(XA(u, m)))
    unsigned fv;
    {
        const unsigned x0 = xbase + i16 * 128 + ((g ^ (i16 >> 1)) * 16), x1 = x0 ^ 64;
        asm volatile("s_waitcnt vmcnt(12)\n\t"
                     ".set k_, 0\n\t.rept 4\n\tds_read_b128 a[160+8*k_:163+8*k_], %[x0] offset:2048*k_\n\t"
                     "ds_read_b128 a[164+8*k_:167+8*k_], %[x1] offset:2048*k_\n\t.set k_, k_ + 1\n\t.endr\n\t"
                     "v_accvgpr_read_b32 %[f], a240\n\tv_accvgpr_write_b32 a240, 0"
                     : [f] "=v"(fv) : [x0] "v"(x0), [x1] "v"(x1) : "memory");
    }
    RB_ALL(1);
    STAMP(1); CSTAMP(101);
    const unsigned call = uni(fv) + 1;
    const unsigned tstart = (unsigned)__builtin_amdgcn_s_memrealtime();
    const v4s ra = make_rsrc(myws, 2 * SLOT_B);
    const unsigned av16 = lane * 16;
    const unsigned aso0 = (call & 1) * (unsigned)SLOT_B + wv * 65536 + b0 * 2048;
    const u64 fbase = myws + FLAG_OFF + t * 4096 + wv * 128;
    const unsigned foff = SRC(lane < 7 ? lane : 0) * 512;
#define IN_POLL [fo] "v"(foff), [fb] "s"(fbase)
#define IN_LA [av] "v"(av16), [ra] "s"(ra), [s0] "s"(aso), [s1] "s"(aso + 4096), [s2] "s"(aso + 8192)
#define NOTE() { asm volatile("v_accvgpr_read_b32 %0, a240" : "=v"(fv)::"memory"); av |= (unsigned)__ballot(fv >= call) & 0x7f; }
    int ua = 0, ub = 7, nxt;
    unsigned rest = 0x7e, av = 0, late = 0, aso;
    sa ^= flip;
#define T_IF5(x) ".if %c[nb] == 5\n\t" x ".endif\n\t"
#define T_LA10(u) T_LA(u, 0, s0) T_LA(u, 1, s0) T_LA(u, 2, s0) T_LA(u, 3, s0) T_LA(u, 4, s1) T_LA(u, 5, s1) T_LA(u, 6, s1) \
                  T_LA(u, 7, s1) T_IF5(T_LA(u, 8, s2) T_LA(u, 9, s2))
#define T_M2(u, k) T_MM(u, k) T_MM(u, k + 1)
#define T_M4(u, k) T_M2(u, k) T_M2(u, k + 2)
#define T_H2(u, D0, D1, D2, D3, G1, G2, G3) ".set h_, 8*%c[nb]-17\n\t" \
        T_MM(u, h_) D0 T_RB0(u, 0) T_MM(u, h_ + 1) D1 T_RB1(u, 0) T_MM(u, h_ + 2) D2 T_RB0(u, 1) T_MM(u, h_ + 3) D3 G1 T_RB1(u, 1) \
        T_MM(u, h_ + 4) D0 T_RB0(u, 2) T_MM(u, h_ + 5) D1 T_RB1(u, 2) T_MM(u, h_ + 6) D2 T_RB0(u, 3) T_MM(u, h_ + 7) D3 G2 T_RB1(u, 3) \
        T_MM(u, h_ + 8) D0 T_RB0(u, 4) T_MM(u, h_ + 9) D1 T_RB1(u, 4) T_MM(u, h_ + 10) D2 T_RB0(u, 5) T_MM(u, h_ + 11) D3 G3 T_RB1(u, 5) \
        T_MM(u, h_ + 12) D0 T_RB0(u, 6) T_MM(u, h_ + 13) D1 T_RB1(u, 6) T_MM(u, h_ + 14) D2 T_RB0(u, 7) T_MM(u, h_ + 15) D3 T_RB1(u, 7) \
        T_MM(u, h_ + 16)
#define CHOOSE() { const unsigned sw = (av >> ub) & ~(av >> ua) & 1; \
        nxt = sw ? ub : ua; ub = sw ? ua : ub; ua = nxt; sa ^= sw ? flip : 0; aso = aso0 + (SRC(nxt) << 18); }
#define PICK() __builtin_ctz(rest & av ? rest & av : rest & ~late ? rest & ~late : rest | 0x80)
#define UNIT(u) { \
        SB0; MM(u, 0, 0); SB0; \
        CHOOSE(); PIN_S(aso); \
        SB0; MM(u, 0, 1); SB0; \
        if (__builtin_expect((av >> nxt) & 1, 1)) { \
            asm volatile(T_MM(u, 2) T_LA(u, 0, s0) T_MM(u, 3) T_LA(u, 1, s0) T_MM(u, 4) T_LA(u, 2, s0) T_MM(u, 5) T_LA(u, 3, s0) \
                         T_MM(u, 6) T_LA(u, 4, s1) T_MM(u, 7) T_LA(u, 5, s1) T_MM(u, 8) T_LA(u, 6, s1) T_MM(u, 9) T_LA(u, 7, s1) \
                         T_MM(u, 10) T_IF5(T_LA(u, 8, s2)) T_MM(u, 11) T_IF5(T_LA(u, 9, s2)) T_MM(u, 12) \
                         T_IF5(T_M4(u, 13) T_M4(u, 17)) "s_waitcnt vmcnt(2*%c[nb])" ::IN_LA, [nb] "n"(NB) : "memory"); \
        } else { \
            asm volatile(T_POLL T_M4(u, 2) T_M4(u, 6) T_MM(u, 10) T_IF5(T_M4(u, 11) T_M4(u, 15)) "s_waitcnt vmcnt(0)" \
                         ::IN_POLL, [nb] "n"(NB) : "memory"); \
            NOTE(); \
            CHOOSE(); \
            if ((av >> nxt) & 1) asm volatile(T_MM(u, 8*%c[nb]-21) T_LA10(u) T_MM(u, 8*%c[nb]-20) ::IN_LA, [nb] "n"(NB) : "memory"); \
            else asm volatile(T_M2(u, 8*%c[nb]-21) ::[nb] "n"(NB)); \
        } \
        unsigned went = (av >> nxt) & 1, more = ub == 7 ? rest : 0; PIN_S(went); PIN_S(more); \
        SB0; MM(u, NB - 3, 5); SB0; \
        if (more) { \
            ub = PICK(); rest &= ~(1u << ub); \
            asm volatile(T_MM(u, 8*%c[nb]-18) T_POLL T_GRP(0) \
                         T_H2(u, T_DMA(v0, 0) T_ST, T_DMA(v0, 1024) T_ST, T_DMA(v1, 2048) T_ST, T_DMA(v1, 3072), T_GRP(1), T_GRP(2), T_GRP(3)) \
                         "s_waitcnt vmcnt(16) lgkmcnt(0)" \
                         : [t] "=&s"(tmp), [ta] "=&v"(ta) : IN_POLL, IN_RB, [d] "s"(sa ^ fdst), [s] "s"(WSO(SRC(ub))), [rw] "s"(rw), \
                           [v0] "v"(wv0), [v1] "v"(wv1), [nb] "n"(NB) : "memory", "scc", CLOB_WB); \
        } else asm volatile(T_MM(u, 8*%c[nb]-18) T_POLL T_H2(u, "", "", "", "", "", "", "") "s_waitcnt vmcnt(0) lgkmcnt(0)" \
                            : [ta] "=&v"(ta) : IN_POLL, IN_RB, [nb] "n"(NB) : "memory", CLOB_WB); \
        ua = ub; ub = 7; sa ^= flip; \
        NOTE(); \
        if (__builtin_expect(!went, 0)) STALL(u); }
#define STALL(u) { int pat = 0; \
        while (!((av >> nxt) & 1)) { \
            const unsigned ina = (av >> ua) & 1, inb = (av >> ub) & 1; \
            int ld = 7; \
            if (ub == 7 && rest) { \
                ld = PICK(); rest &= ~(1u << ld); \
            } else if ((ina | inb | (av & rest)) && ++pat >= PATIENCE) { \
                if (ina | inb) { \
                    waitv<0>(); \
                    if (!ina) { sa ^= flip; const int x_ = ua; ua = ub; ub = x_; } \
                    RB_ALL(u); \
                    late |= 1u << nxt; \
                    ld = nxt; nxt = ua; ua = ub; sa ^= flip; \
                } else { \
                    ld = __builtin_ctz(av & rest); \
                    rest = (rest | (1u << ub)) & ~(1u << ld); late |= 1u << ub; \
                } \
            } \
            if (ld != 7) { \
                LW16(sa ^ fdst, WSO(SRC(ld))); \
                ub = ld; \
                if (ua == 7) { ua = ub; ub = 7; sa ^= flip; } \
                continue; \
            } \
            if ((unsigned)__builtin_amdgcn_s_memrealtime() - tstart > DEADLINE) break; \
            asm volatile(T_POLL "s_waitcnt vmcnt(0)" ::IN_POLL : "memory"); \
            NOTE(); \
        } \
        STAMP(36 + nxt * 4 + wv); \
        aso = aso0 + (SRC(nxt) << 18); \
        if ((av >> nxt) & 1) { \
            asm volatile(T_LA10(u) T_POLL "s_waitcnt vmcnt(0)" ::IN_LA, IN_POLL, [nb] "n"(NB) : "memory"); \
            NOTE(); \
        } else {      \
            asm volatile(".set k_, %c1\n\t.rept %c2\n\tv_accvgpr_write_b32 a[k_], %0\n\t.set k_, k_ + 8\n\t.endr\n\ts_waitcnt vmcnt(0)\n\ts_nop 1" \
                         ::"v"(0x7f7f7f7f), "n"(XA(1 - (u), 0)), "n"(NB) : "memory"); \
        } }
#pragma unroll 1
    for (int it = 0;; ++it) {
        UNIT(0);
        STAMP(64 + it * 8 + wv);
        if (it == 3) break;
        UNIT(1);
        STAMP(68 + it * 8 + wv);
    }
    {
        const float l2e = 1.4426950408889634f;
        const u64 sc2 = (u64)__builtin_bit_cast(unsigned, scale) * 0x100000001ull;
        const u64 se2 = (u64)__builtin_bit_cast(unsigned, -scale * l2e) * 0x100000001ull;
        const unsigned wo = i16 * 128 + ((g ^ (i16 & 7)) * 16), ro = (unsigned)(size_t)lds + r8 * 128 + (((lane ^ r8) & 7) * 16);
        unsigned p[4], x[4];
#pragma unroll
        for (int d = 0; d < 4; ++d) {
            p[d] = ro + ((wv + 3 - d) & 3) * WAVE_B + d * 8192;
            x[d] = ro + XS_OFF + ((wv + 3 - d) & 3) * XS_B + d * 2048;
        }
        unsigned vo = (r8 * NN + wv * 32 + (lane & 7) * 4) * 4;
#define T_PUT(k) ".if (" #k ") < 32\n\t.if (" #k ") & 1\n\tds_write_b128 %[w1], a[4*(" #k "):4*(" #k ")+3] offset:4096*((" #k ")&6)+2048*((" #k ")>>3)\n\t" \
        ".else\n\tds_write_b128 %[w0], a[4*(" #k "):4*(" #k ")+3] offset:4096*((" #k ")&6)+2048*((" #k ")>>3)\n\t.endif\n\t" \
        ".else\n\t.if (" #k ") & 1\n\tds_write_b128 %[y1], a[4*(" #k "):4*(" #k ")+3] offset:1024*((" #k ")&6)\n\t" \
        ".else\n\tds_write_b128 %[y0], a[4*(" #k "):4*(" #k ")+3] offset:1024*((" #k ")&6)\n\t.endif\n\t.endif\n\t"
#define T_G1(m, s, e, rp, rx) ".if " #m " < 4\n\tds_read_b128 v[128+32*" #s "+4*" #e ":131+32*" #s "+4*" #e "], %[" #rp "] offset:2048*" #m "+1024*(" #e "&1)\n\t" \
        ".elseif " #m " < %c[nb]\n\tds_read_b128 v[128+32*" #s "+4*" #e ":131+32*" #s "+4*" #e "], %[" #rx "] offset:1024*(" #e "&1)\n\t.endif\n\t"
#define T_P1(m, n) ".if " #m " < %c[nb]\n\t" T_PUT(8*m+6+n) ".endif\n\t"
#define T_PRE ""
#define W2 2
#define T_FIN(s, w, m1, s1, m2) T_FIN_(s, w, m1, s1, m2)
#define T_R(n, k0, step, x) ".set k_, " #k0 "\n\t.rept " #n "\n\t" x "\n\t.set k_, k_+" #step "\n\t.endr\n\t"
#define T_A1 "v_pk_add_f32 v[o_+k_:o_+k_+1], v[o_+k_:o_+k_+1], v[o_+16+k_:o_+17+k_]"
#define T_CL "v_med3_f32 v[o_+8+k_], v[o_+8+k_], -%[lim], %[lim]"
#define T_EX "v_exp_f32 v[o_+8+k_], v[o_+8+k_]"
#define T_RC "v_rcp_f32 v[o_+8+k_], v[o_+8+k_]"
#define T_MU "v_pk_mul_f32 v[o_+k_:o_+k_+1], v[o_+k_:o_+k_+1], v[o_+8+k_:o_+9+k_]"
#define T_FIN_(s, w, m1, s1, m2) ".set o_, 128+32*" #s "\n\ts_waitcnt lgkmcnt(" #w ")\n\t" \
        T_G1(m1, s1, 0, p0, x0) T_R(4, 0, 2, T_A1) T_G1(m1, s1, 1, p0, x0) T_R(4, 8, 2, T_A1) T_G1(m1, s1, 2, p1, x1) \
        T_R(4, 0, 2, "v_pk_add_f32 v[o_+k_:o_+k_+1], v[o_+k_:o_+k_+1], v[o_+8+k_:o_+9+k_]") T_G1(m1, s1, 3, p1, x1) \
        T_R(4, 0, 2, "v_pk_mul_f32 v[o_+8+k_:o_+9+k_], v[o_+k_:o_+k_+1], %[se]") T_G1(m1, s1, 4, p2, x2) \
        T_R(4, 0, 1, T_CL) T_G1(m1, s1, 5, p2, x2) T_R(4, 4, 1, T_CL) T_G1(m1, s1, 6, p3, x3) \
        T_R(3, 0, 1, T_EX) T_G1(m1, s1, 7, p3, x3) T_R(3, 3, 1, T_EX) T_P1(m2, 0) T_R(2, 6, 1, T_EX) \
        T_R(4, 0, 2, "v_pk_mul_f32 v[o_+k_:o_+k_+1], v[o_+k_:o_+k_+1], %[sc]") T_P1(m2, 1) \
        T_R(4, 0, 2, "v_pk_add_f32 v[o_+8+k_:o_+9+k_], v[o_+8+k_:o_+9+k_], 1.0 op_sel_hi:[1,0]") \
        T_R(5, 0, 1, T_RC) T_R(2, 0, 2, T_MU) "global_store_dwordx4 %[vo], v[o_:o_+3], %[ot]" OUT_POL "\n\t" \
        T_R(3, 5, 1, T_RC) "s_nop 0\n\t" T_R(2, 4, 2, T_MU) "global_store_dwordx4 %[vo], v[o_+4:o_+7], %[ou]" OUT_POL "\n\t" \
        "v_add_u32 %[vo], 0x40000, %[vo]\n\t"
#define EPI_A
#define EPI_B
#define T_PM(v, j) ".set " #v ", 8*(((" #j ")%%(2*%c[nb]))>>1)+2*((" #j ")/(2*%c[nb]))+((" #j ")&1)\n\t"
        asm volatile(".set j_, 0\n\t.rept 7\n\t" T_PM(a_, j_) T_MM(1, a_) ".set j_, j_+1\n\t.endr\n\ts_nop 5\n\t" T_PUT(0)
                     ".rept 8*%c[nb]-7\n\t" T_PM(a_, j_) T_PM(b_, j_-6) T_MM(1, a_)
                     ".if j_-6 < 6*%c[nb]+2\n\t" T_PUT(b_) ".endif\n\t.set j_, j_+1\n\t.endr\n\t"
                     "s_waitcnt lgkmcnt(2)\n\ts_barrier\n\t"
                     T_PRE EPI_A DIAG_T
                     T_G1(0, 0, 0, p0, x0) T_G1(0, 0, 4, p2, x2) T_G1(0, 0, 1, p0, x0) T_G1(0, 0, 5, p2, x2)
                     T_G1(0, 0, 2, p1, x1) T_G1(0, 0, 6, p3, x3) T_G1(0, 0, 3, p1, x1) T_G1(0, 0, 7, p3, x3) T_P1(1, 0) T_P1(1, 1)
                     T_FIN(0, W2, 1, 1, 2) T_FIN(1, W2, 2, 2, 3) T_FIN(2, W2, 3, 3, 4) T_FIN(3, 0, 4, 0, 5) T_IF5(T_FIN(0, 0, 5, 0, 5)) EPI_B
                     : [vo] "+v"(vo) DIAG_TO
                     : [w0] "v"(lbase + wo), [w1] "v"(lbase + (wo ^ 64)), [y0] "v"(xbase + wo), [y1] "v"(xbase + (wo ^ 64)),
                       [p0] "v"(p[0]), [p1] "v"(p[1]), [p2] "v"(p[2]), [p3] "v"(p[3]), [x0] "v"(x[0]), [x1] "v"(x[1]), [x2] "v"(x[2]),
                       [x3] "v"(x[3]), [ot] "s"(otile), [ou] "s"(otile + 8 * NN), [sc] "v"(sc2), [se] "v"(se2),
                       [lim] "v"(60.0f * l2e), [nb] "n"(NB) : "memory", "scc", CLOB_WB);
    }
    if (wv == 0 && lane == 0) __hip_atomic_store(cnt, call, __ATOMIC_RELAXED, __HIP_MEMORY_SCOPE_AGENT);
}
}

__global__ __launch_bounds__(NTHREADS) void a2a_gemm_kernel(Params p) {
    __shared__ __attribute__((aligned(16384))) char smem[LDS_B];
    LAS char* lds = (LAS char*)smem;
    const int b = blockIdx.x;
    const int wv = __builtin_amdgcn_readfirstlane(threadIdx.x >> 6), lane = threadIdx.x & 63;
    const int xg = b & 7, idx = b >> 3;
    const int srel = b < NSEND_WG ? (xg + 2 * idx) & 7 : 0;
    u64 xs = (u64)p.xsend[srel], dws = (u64)p.wsdst[srel], myws = (u64)p.wsdst[0], pw = (u64)p.w, psx = (u64)p.sx,
        psw = (u64)p.sw, pout = (u64)p.out;
    int rank = p.rank;
    const u32x4 tab = *(const CAS u32x4*)((u64)&kSend + ((b & (NSEND_WG - 1)) * 4 + wv) * 16);
    unsigned tb0 = tab[0], tb1 = tab[1], tb2 = tab[2], tb3 = tab[3];
    asm volatile("" : "+s"(tb0), "+s"(tb1), "+s"(tb2), "+s"(tb3), "+s"(xs), "+s"(dws), "+s"(myws), "+s"(pw), "+s"(psx), "+s"(psw), "+s"(pout), "+s"(rank));
    GAS unsigned* const cnt = (GAS unsigned*)(myws + CNT_OFF) + b;

    if (__builtin_expect(b < NSEND_WG, 1)) {
        ST_DECL;
        const int rel = srel;
        if (rel == 0) return;
        LAS unsigned* const prog = (LAS unsigned*)lds;
        if (__builtin_expect(wv < 3, 1)) {
            constexpr int WIN = SEND_WIN;
            static_assert(WIN % 2 == 0 && WIN >= 2 && WIN <= 6, "");
            const int r8 = lane >> 3, ch0 = ((lane & 7) ^ (r8 >> 1)) * 16, g = lane >> 4, i16 = lane & 15;
            const unsigned v0 = r8 * KC + ch0, v1 = (v0 ^ 64) + 8 * KC - 1024;
            const unsigned lb = (unsigned)(size_t)lds + 1024 + wv * 22528;
            const unsigned x0 = lb + i16 * 128 + ((g ^ (i16 >> 1)) * 16), x1 = x0 ^ 64;
            const v4s rx = make_rsrc(xs, KC * KC);
            const u64 sd = dws + (size_t)rank * 262144, sdl = wv == 2 ? myws + (size_t)rank * 262144 : sd;
            unsigned vd = lane * 16, vt, st, s2, vc;
            u32x4 v[22];
#define XLD(i, w) "s_add_u32 m0, %[lb], 2048*" #i "\n\ts_bfe_u32 %[st], %[" #w "], 0x50000+8*(" #i "&3)\n\t" \
               "s_bfe_u32 %[s2], %[" #w "], 0x20005+8*(" #i "&3)\n\ts_lshl_b32 %[st], %[st], 13\n\ts_lshl_b32 %[s2], %[s2], 7\n\t" \
               "s_or_b32 %[st], %[st], %[s2]\n\tbuffer_load_dwordx4 %[v0], %[rx], %[st] offen lds\n\t" \
               "buffer_load_dwordx4 %[v1], %[rx], %[st] offen offset:1024 lds\n\t"
#define XRD(i, a, c) "ds_read_b128 %[" #a "], %[x0] offset:2048*" #i "\n\tds_read_b128 %[" #c "], %[x1] offset:2048*" #i "\n\t"
#define XLA(i, w) ".if %c[win] > " #i "\n\t" XLD(i, w) ".endif\n\t"
#define XLB(i, w) ".if %c[win] <= " #i "\n\t" XLD(i, w) ".endif\n\t"
#define XST(i, a, c) ".if %c[win] > 2*" #i "\n\ts_waitcnt vmcnt(2*%c[win]-2)\n\t" XRD(i, a, c) \
                     "s_bfe_u32 %[st], %[t0], 0x70000+8*" #i "\n\tv_lshl_add_u32 %[vt], %[st], 11, %[vd]\n\t" \
                     "s_waitcnt lgkmcnt(0)\n\tglobal_store_dwordx4 %[vt], %[" #a "], %[sd] sc0 sc1\n\t" \
                     "global_store_dwordx4 %[vt], %[" #c "], %[sd] offset:1024 sc0 sc1\n\t.endif\n\t"
            asm volatile("global_load_dword %[vc], %[z], %[cnt] sc0 sc1\n\t"
                         XLA(0, t0) XLA(1, t0) XLA(2, t0) XLA(3, t0) XLA(4, t1) XLA(5, t1)
                         "s_waitcnt vmcnt(2*%c[win])\n\tv_not_b32 %[vc], %[vc]\n\tv_and_b32 %[vc], 1, %[vc]\n\t"
                         "v_lshl_add_u32 %[vd], %[vc], 21, %[vd]\n\t"
                         XST(0, d0, d1) XST(1, d2, d3) XST(2, d4, d5)
                         XLB(2, t0) XLB(3, t0) XLB(4, t1) XLB(5, t1) XLD(6, t1) XLD(7, t1) XLD(8, t2) XLD(9, t2) XLD(10, t2)
                         "s_barrier\n\ts_waitcnt vmcnt(22-%c[win])"
                         : [d0] "=&v"(v[0]), [d1] "=&v"(v[1]), [d2] "=&v"(v[2]), [d3] "=&v"(v[3]), [d4] "=&v"(v[4]),
                           [d5] "=&v"(v[5]), [vc] "=&v"(vc), [vt] "=&v"(vt), [st] "=&s"(st), [s2] "=&s"(s2), [vd] "+v"(vd)
                         : [z] "v"(0), [cnt] "s"(cnt), [v0] "v"(v0), [v1] "v"(v1), [rx] "s"(rx), [lb] "s"(lb), [x0] "v"(x0),
                           [x1] "v"(x1), [sd] "s"(sd), [t0] "s"(tb0), [t1] "s"(tb1), [t2] "s"(tb2), [win] "n"(WIN) : "memory", "scc");
            static_assert(SLOT_B == 1u << 21, "");
#define XRC(lo, hi) _Pragma("unroll") for (int i = (lo); i < (hi); ++i) { \
                v[2 * i] = *(const LAS u32x4*)(size_t)(x0 + 2048 * i); v[2 * i + 1] = *(const LAS u32x4*)(size_t)(x1 + 2048 * i); }
            XRC(WIN / 2, WIN);
            STAMP(16 + wv);
#define POS(i) ((((i) < 4 ? tb0 : (i) < 8 ? tb1 : tb2) >> (8 * ((i) & 3))) & 127)
#define SEND_STEP(I) { waitv<(2 * (I) < WIN ? 20 - WIN : 2 * (I) + WIN < 22 ? WIN - 2 : 20 - 2 * (I))>(); \
            if (2 * (I) == WIN) XRC(WIN, 11); \
            if (lane == 0) lds_st(prog + wv, (I) + 1); \
            if (2 * (I) + WIN < 22) asm volatile("global_store_dwordx4 %0, %1, %3 sc0 sc1\n\t" \
                "global_store_dwordx4 %0, %2, %3 offset:1024 sc0 sc1\n\ts_nop 1" \
                ::"v"(vd + POS((I) + WIN / 2) * 2048), "v"(v[(2 * (I) + WIN) % 22]), \
                  "v"(v[(2 * (I) + WIN + 1) % 22]), "s"((I) + WIN / 2 == 10 ? sdl : sd) : "memory"); }
            SEND_STEP(0) SEND_STEP(1) SEND_STEP(2) SEND_STEP(3) SEND_STEP(4) SEND_STEP(5)
            SEND_STEP(6) SEND_STEP(7) SEND_STEP(8) SEND_STEP(9) SEND_STEP(10)
        } else {
            if (lane < 3) lds_st(prog + lane, 0);
            const unsigned call = uni(__hip_atomic_load(cnt, __ATOMIC_RELAXED, __HIP_MEMORY_SCOPE_AGENT)) + 1;
            asm volatile("s_waitcnt lgkmcnt(0)" ::: "memory");
            __builtin_amdgcn_s_barrier();
            const int l3 = lane < 3 ? lane : 0;
            u64 tq = (u64)tb1 << 32 | tb0, te = (u64)tb3 << 32 | tb2;
#pragma unroll 1
            for (int r = 0; r < 7; ++r, tq >>= 8, te >>= 8) {
                const int e = (int)te & 255, t = ((int)tq >> 2) & 7, q = (int)tq & 3;
                while (!__all((int)(lds_ld(prog + l3) * 3 + l3) >= e)) __builtin_amdgcn_s_sleep(1);
                if (lane == 0)
                    __hip_atomic_store((GAS unsigned*)(dws + FLAG_OFF + ((t * 8 + rank) * 4 + q) * 128), call,
                                       __ATOMIC_RELAXED, __HIP_MEMORY_SCOPE_SYSTEM);
                STAMP(1 + r);
            }
            if (lane == 0) __hip_atomic_store(cnt, call, __ATOMIC_RELAXED, __HIP_MEMORY_SCOPE_AGENT);
        }
        DUMP();
        return;
    }

    const int ci = idx - 4, sl = ci / 7, t = ci - sl * 7;
    const int slab = SLABMAP == 0 ? xg * 4 + sl : SLABMAP == 1 ? sl * 8 + xg : (sl >> 1) * 16 + xg * 2 + (sl & 1);
    GAS float* const otile = (GAS float*)pout + (size_t)(tile_b0(t) * 16) * NN + slab * 128;
    if (t >= 3) gemm_wave<5>(lds, xs, myws, pw, cnt, otile, psx, psw, rank, t, slab, sl, wv, lane DIAG_ARG);
    else gemm_wave<4>(lds, xs, myws, pw, cnt, otile, psx, psw, rank, t, slab, sl, wv, lane DIAG_ARG);
}

extern "C" void kernel_launch(int rank, int nranks, ncclComm_t comm, hipStream_t stream,
                              void* const* d_in, const int* in_sizes, int n_in,
                              void* d_out, int out_size, const int* in_dtypes, int out_dtype,
                              void* const* d_out_all, void* const* d_ws_all, size_t ws_bytes) {
    (void)comm; (void)in_dtypes; (void)out_dtype; (void)d_out_all;
    if (nranks != NR || (unsigned)rank >= (unsigned)NR || n_in != 4 || ws_bytes < WS_NEED ||
        in_sizes[0] != NN * KC || in_sizes[1] != NN * NN || out_size != KC * NN) {
        fprintf(stderr, "unexpected problem shape\n");
        return;
    }
    Params p;
    for (int rel = 0; rel < NR; ++rel) {
        const int d = (rank - rel) & 7;
        p.xsend[rel] = (const char*)d_in[0] + (size_t)d * KC * KC;
        p.wsdst[rel] = (char*)d_ws_all[d];
    }
    p.w = (const char*)d_in[1];
    p.sx = (const float*)d_in[2];
    p.sw = (const float*)d_in[3];
    p.out = (float*)d_out;
    p.rank = rank;
    p.pad = 0;
    a2a_gemm_kernel<<<256, NTHREADS, 0, stream>>>(p);
    HIP_CHECK(hipGetLastError());
}
```

```cpp
#include <hip/hip_runtime.h>
#include <rccl/rccl.h>
#include <cstdio>

#define HIP_CHECK(expr) do { hipError_t _e = (expr); if (_e != hipSuccess) { \
    fprintf(stderr, "HIP error %s at %s:%d\n", hipGetErrorString(_e), __FILE__, __LINE__); \
    return; } } while (0)

typedef unsigned long long u64;
#define GAS __attribute__((address_space(1)))
#define LAS __attribute__((address_space(3)))

namespace {
constexpr int NR = 8;
constexpr int KC = 512;
constexpr int NN = 4096;
constexpr size_t SLOT_B = 2u << 20;
constexpr size_t FLAG_OFF = 2 * SLOT_B;
constexpr size_t CNT_OFF = FLAG_OFF + 32768;
constexpr size_t DIAG_OFF = CNT_OFF + 4096;
constexpr size_t WS_NEED = DIAG_OFF + 256 * 128 * 8;
constexpr int LDS_B = 163840;
constexpr int NSEND_WG = 32;

struct Params {
    unsigned rw[4], rx[4], ra[4];
    unsigned rank, pad;
    u64 out, sx, sw;
    u64 peer[NR][2];
    unsigned srcs[8];
    unsigned tab[NR][4][2];
};
static_assert(sizeof(Params) == 0xf0 + NR * 32, "");
constexpr int kSend[7][7] = {{1, 2, 3, 4, 5, 6, 0}, {2, 3, 4, 5, 6, 0, 1}, {3, 4, 5, 6, 0, 1, 2}, {4, 5, 6, 0, 1, 2, 3}, {5, 6, 0, 1, 2, 3, 4}, {6, 0, 1, 2, 3, 4, 5}, {0, 1, 2, 3, 4, 5, 6}};
constexpr int kTake[7][7] = {{7, 6, 5, 4, 3, 2, 1}, {1, 7, 6, 5, 4, 3, 2}, {2, 1, 7, 6, 5, 4, 3}, {3, 2, 1, 7, 6, 5, 4}, {4, 3, 2, 1, 7, 6, 5}, {5, 4, 3, 2, 1, 7, 6}, {6, 5, 4, 3, 2, 1, 7}};
}

#define KERNEL_BODY \
    "s_load_dwordx16 s[8:23], %[ka], 0x0\n" \
    "s_lshr_b32 s46, %[bid], 3\n" \
    "s_and_b32 s47, s46, 3\n" \
    "s_lshl1_add_u32 s55, s47, %[bid]\n" \
    "s_and_b32 s50, s55, 7\n" \
    "s_lshl_b32 s55, s50, 4\n" \
    "s_load_dwordx4 s[28:31], %[ka], s55 offset:0x50\n" \
    "v_readfirstlane_b32 s44, %[tid]\n" \
    "s_cmp_lt_u32 %[bid], 32\n" \
    "s_cbranch_scc0 a2a_gemm\n" \
    "s_getpc_b64 s[36:37]\n" \
      \
    "a2a_pc0:\n" \
    "v_mul_u32_u24 v23, 96, %[tid]\n" \
    "v_min_u32 v23, a2a_end-a2a_pc0-4, v23\n" \
    "global_load_dword v20, v23, s[36:37]\n" \
    "s_lshr_b32 s56, s44, 3\n" \
    "s_lshl1_add_u32 s55, s55, s56\n" \
    "s_load_dwordx2 s[42:43], %[ka], s55 offset:0xf0\n" \
    "s_cmp_eq_u32 s50, 0\n" \
    "s_cbranch_scc1 a2a_end\n" \
    "v_and_b32 v1, 63, %[tid]\n" \
    "s_lshr_b32 s44, s44, 6\n" \
    "v_lshrrev_b32 v2, 3, v1\n" \
    "v_lshrrev_b32 v16, 4, v1\n" \
    "v_xor_b32 v3, v1, v16\n" \
    "v_and_b32 v3, 7, v3\n" \
    "v_lshlrev_b32 v3, 4, v3\n" \
    "v_lshl_or_b32 v6, v2, 9, v3\n" \
    "v_xor_b32 v7, 64, v6\n" \
    "v_add_u32 v7, 3072, v7\n" \
    "s_lshl2_add_u32 s55, %[bid], 4227072\n" \
    "v_mov_b32 v9, s55\n" \
    "s_cmp_eq_u32 s44, 3\n" \
    "s_cbranch_scc1 a2a_flagger\n" \
    "s_mul_i32 s51, s44, 22528\n" \
    "s_add_u32 s51, s51, %[lds]\n" \
    "s_lshl_b32 s82, s46, 7\n" \
    "s_waitcnt lgkmcnt(0)\n" \
    "s_add_u32 s12, s28, s82\n" \
    "s_addc_u32 s13, s29, 0\n" \
    "s_sub_u32 s14, s14, s82\n" \
    "global_load_dword v10, v9, s[16:17] sc0 sc1\n" \
    "s_add_u32 m0, s51, 0\n" \
    "s_bfe_u32 s58, s42, 327680\n" \
    "s_lshl_b32 s58, s58, 13\n" \
    "buffer_load_dwordx4 v6, s[12:15], s58 offen lds\n" \
    "buffer_load_dwordx4 v7, s[12:15], s58 offen offset:1024 lds\n" \
    "s_add_u32 m0, s51, 2048\n" \
    "s_bfe_u32 s58, s42, 327685\n" \
    "s_lshl_b32 s58, s58, 13\n" \
    "buffer_load_dwordx4 v6, s[12:15], s58 offen lds\n" \
    "buffer_load_dwordx4 v7, s[12:15], s58 offen offset:1024 lds\n" \
    "s_add_u32 m0, s51, 4096\n" \
    "s_bfe_u32 s58, s42, 327690\n" \
    "s_lshl_b32 s58, s58, 13\n" \
    "buffer_load_dwordx4 v6, s[12:15], s58 offen lds\n" \
    "buffer_load_dwordx4 v7, s[12:15], s58 offen offset:1024 lds\n" \
    "s_add_u32 m0, s51, 6144\n" \
    "s_bfe_u32 s58, s42, 327695\n" \
    "s_lshl_b32 s58, s58, 13\n" \
    "buffer_load_dwordx4 v6, s[12:15], s58 offen lds\n" \
    "buffer_load_dwordx4 v7, s[12:15], s58 offen offset:1024 lds\n" \
    "s_add_u32 m0, s51, 8192\n" \
    "s_bfe_u32 s58, s42, 327700\n" \
    "s_lshl_b32 s58, s58, 13\n" \
    "buffer_load_dwordx4 v6, s[12:15], s58 offen lds\n" \
    "buffer_load_dwordx4 v7, s[12:15], s58 offen offset:1024 lds\n" \
    "v_and_b32 v16, 15, v1\n" \
    "v_lshrrev_b32 v17, 1, v16\n" \
    "v_lshrrev_b32 v21, 4, v1\n" \
    "v_xor_b32 v21, v21, v17\n" \
    "v_lshlrev_b32 v21, 4, v21\n" \
    "v_lshl_or_b32 v21, v16, 7, v21\n" \
    "v_add_u32 v21, s51, v21\n" \
    "v_xor_b32 v22, 64, v21\n" \
    "s_lshl2_add_u32 s55, s20, s46\n" \
    "s_lshl_b32 s55, s55, 16\n" \
    "v_lshl_add_u32 v24, v1, 4, s55\n" \
    "s_lshl_b32 s55, s44, 8\n" \
    "s_add_u32 s55, s55, %[lds]\n" \
    "s_add_u32 s55, s55, 69632\n" \
    "v_lshl_add_u32 v23, v1, 2, s55\n" \
    "s_waitcnt vmcnt(10)\n" \
    "v_not_b32 v10, v10\n" \
    "v_and_b32 v10, 1, v10\n" \
    "v_lshl_add_u32 v24, v10, 21, v24\n" \
    "s_waitcnt vmcnt(8)\n" \
    "ds_read_b128 v[32:35], v21 offset:0\n" \
    "ds_read_b128 v[36:39], v22 offset:0\n" \
    "s_bfe_u32 s58, s42, 327680\n" \
    "s_lshl_b32 s58, s58, 11\n" \
    "v_add_u32 v16, s58, v24\n" \
    "s_waitcnt lgkmcnt(0)\n" \
    "global_store_dwordx4 v16, v[32:35], s[30:31] sc0 sc1\n" \
    "global_store_dwordx4 v16, v[36:39], s[30:31] offset:1024 sc0 sc1\n" \
    "s_waitcnt vmcnt(8)\n" \
    "ds_read_b128 v[40:43], v21 offset:2048\n" \
    "ds_read_b128 v[44:47], v22 offset:2048\n" \
    "s_bfe_u32 s58, s42, 327685\n" \
    "s_lshl_b32 s58, s58, 11\n" \
    "v_add_u32 v16, s58, v24\n" \
    "s_waitcnt lgkmcnt(0)\n" \
    "global_store_dwordx4 v16, v[40:43], s[30:31] sc0 sc1\n" \
    "global_store_dwordx4 v16, v[44:47], s[30:31] offset:1024 sc0 sc1\n" \
    "s_barrier\n" \
    "s_waitcnt vmcnt(4)\n" \
    "ds_read_b128 v[48:51], v21 offset:4096\n" \
    "ds_read_b128 v[52:55], v22 offset:4096\n" \
    "ds_read_b128 v[56:59], v21 offset:6144\n" \
    "ds_read_b128 v[60:63], v22 offset:6144\n" \
    "ds_read_b128 v[64:67], v21 offset:8192\n" \
    "ds_read_b128 v[68:71], v22 offset:8192\n" \
    "v_mov_b32 v17, 1\n" \
    "s_bfe_u32 s58, s42, 327690\n" \
    "s_lshl_b32 s58, s58, 11\n" \
    "v_add_u32 v16, s58, v24\n" \
    "s_waitcnt vmcnt(2)\n" \
    "s_waitcnt lgkmcnt(0)\n" \
    "global_store_dwordx4 v16, v[48:51], s[30:31] sc0 sc1\n" \
    "global_store_dwordx4 v16, v[52:55], s[30:31] offset:1024 sc0 sc1\n" \
    "ds_write_b32 v23, v17\n" \
    "s_add_u32 m0, s51, 10240\n" \
    "s_bfe_u32 s58, s42, 327705\n" \
    "s_lshl_b32 s58, s58, 13\n" \
    "buffer_load_dwordx4 v6, s[12:15], s58 offen lds\n" \
    "buffer_load_dwordx4 v7, s[12:15], s58 offen offset:1024 lds\n" \
    "v_mov_b32 v17, 2\n" \
    "s_bfe_u32 s58, s42, 327695\n" \
    "s_lshl_b32 s58, s58, 11\n" \
    "v_add_u32 v16, s58, v24\n" \
    "s_waitcnt vmcnt(4)\n" \
    "s_waitcnt lgkmcnt(0)\n" \
    "global_store_dwordx4 v16, v[56:59], s[30:31] sc0 sc1\n" \
    "global_store_dwordx4 v16, v[60:63], s[30:31] offset:1024 sc0 sc1\n" \
    "ds_write_b32 v23, v17\n" \
    "s_add_u32 m0, s51, 12288\n" \
    "s_bfe_u32 s58, s43, 327680\n" \
    "s_lshl_b32 s58, s58, 13\n" \
    "buffer_load_dwordx4 v6, s[12:15], s58 offen lds\n" \
    "buffer_load_dwordx4 v7, s[12:15], s58 offen offset:1024 lds\n" \
    "v_mov_b32 v17, 3\n" \
    "s_bfe_u32 s58, s42, 327700\n" \
    "s_lshl_b32 s58, s58, 11\n" \
    "v_add_u32 v16, s58, v24\n" \
    "s_waitcnt vmcnt(6)\n" \
    "s_waitcnt lgkmcnt(0)\n" \
    "global_store_dwordx4 v16, v[64:67], s[30:31] sc0 sc1\n" \
    "global_store_dwordx4 v16, v[68:71], s[30:31] offset:1024 sc0 sc1\n" \
    "ds_write_b32 v23, v17\n" \
    "s_add_u32 m0, s51, 14336\n" \
    "s_bfe_u32 s58, s43, 327685\n" \
    "s_lshl_b32 s58, s58, 13\n" \
    "buffer_load_dwordx4 v6, s[12:15], s58 offen lds\n" \
    "buffer_load_dwordx4 v7, s[12:15], s58 offen offset:1024 lds\n" \
    "v_mov_b32 v17, 4\n" \
    "s_bfe_u32 s58, s42, 327705\n" \
    "s_lshl_b32 s58, s58, 11\n" \
    "v_add_u32 v16, s58, v24\n" \
    "s_waitcnt vmcnt(6)\n" \
    "ds_read_b128 v[72:75], v21 offset:10240\n" \
    "ds_read_b128 v[76:79], v22 offset:10240\n" \
    "s_waitcnt lgkmcnt(0)\n" \
    "global_store_dwordx4 v16, v[72:75], s[30:31] sc0 sc1\n" \
    "global_store_dwordx4 v16, v[76:79], s[30:31] offset:1024 sc0 sc1\n" \
    "ds_write_b32 v23, v17\n" \
    "s_add_u32 m0, s51, 16384\n" \
    "s_bfe_u32 s58, s43, 327690\n" \
    "s_lshl_b32 s58, s58, 13\n" \
    "buffer_load_dwordx4 v6, s[12:15], s58 offen lds\n" \
    "buffer_load_dwordx4 v7, s[12:15], s58 offen offset:1024 lds\n" \
    "v_mov_b32 v17, 5\n" \
    "s_bfe_u32 s58, s43, 327680\n" \
    "s_lshl_b32 s58, s58, 11\n" \
    "v_add_u32 v16, s58, v24\n" \
    "s_waitcnt vmcnt(6)\n" \
    "ds_read_b128 v[80:83], v21 offset:12288\n" \
    "ds_read_b128 v[84:87], v22 offset:12288\n" \
    "s_waitcnt lgkmcnt(0)\n" \
    "global_store_dwordx4 v16, v[80:83], s[30:31] sc0 sc1\n" \
    "global_store_dwordx4 v16, v[84:87], s[30:31] offset:1024 sc0 sc1\n" \
    "ds_write_b32 v23, v17\n" \
    "s_add_u32 m0, s51, 18432\n" \
    "s_bfe_u32 s58, s43, 327695\n" \
    "s_lshl_b32 s58, s58, 13\n" \
    "buffer_load_dwordx4 v6, s[12:15], s58 offen lds\n" \
    "buffer_load_dwordx4 v7, s[12:15], s58 offen offset:1024 lds\n" \
    "v_mov_b32 v17, 6\n" \
    "s_bfe_u32 s58, s43, 327685\n" \
    "s_lshl_b32 s58, s58, 11\n" \
    "v_add_u32 v16, s58, v24\n" \
    "s_waitcnt vmcnt(6)\n" \
    "ds_read_b128 v[88:91], v21 offset:14336\n" \
    "ds_read_b128 v[92:95], v22 offset:14336\n" \
    "s_waitcnt lgkmcnt(0)\n" \
    "global_store_dwordx4 v16, v[88:91], s[30:31] sc0 sc1\n" \
    "global_store_dwordx4 v16, v[92:95], s[30:31] offset:1024 sc0 sc1\n" \
    "ds_write_b32 v23, v17\n" \
    "s_add_u32 m0, s51, 20480\n" \
    "s_bfe_u32 s58, s43, 327700\n" \
    "s_lshl_b32 s58, s58, 13\n" \
    "buffer_load_dwordx4 v6, s[12:15], s58 offen lds\n" \
    "buffer_load_dwordx4 v7, s[12:15], s58 offen offset:1024 lds\n" \
    "v_mov_b32 v17, 7\n" \
    "s_bfe_u32 s58, s43, 327690\n" \
    "s_lshl_b32 s58, s58, 11\n" \
    "v_add_u32 v16, s58, v24\n" \
    "s_waitcnt vmcnt(6)\n" \
    "ds_read_b128 v[96:99], v21 offset:16384\n" \
    "ds_read_b128 v[100:103], v22 offset:16384\n" \
    "s_waitcnt lgkmcnt(0)\n" \
    "global_store_dwordx4 v16, v[96:99], s[30:31] sc0 sc1\n" \
    "global_store_dwordx4 v16, v[100:103], s[30:31] offset:1024 sc0 sc1\n" \
    "ds_write_b32 v23, v17\n" \
    "v_mov_b32 v17, 8\n" \
    "s_bfe_u32 s58, s43, 327695\n" \
    "s_lshl_b32 s58, s58, 11\n" \
    "v_add_u32 v16, s58, v24\n" \
    "s_waitcnt vmcnt(4)\n" \
    "ds_read_b128 v[104:107], v21 offset:18432\n" \
    "ds_read_b128 v[108:111], v22 offset:18432\n" \
    "s_waitcnt lgkmcnt(0)\n" \
    "global_store_dwordx4 v16, v[104:107], s[30:31] sc0 sc1\n" \
    "global_store_dwordx4 v16, v[108:111], s[30:31] offset:1024 sc0 sc1\n" \
    "ds_write_b32 v23, v17\n" \
    "v_mov_b32 v17, 9\n" \
    "s_bfe_u32 s58, s43, 327700\n" \
    "s_lshl_b32 s58, s58, 11\n" \
    "v_add_u32 v16, s58, v24\n" \
    "s_waitcnt vmcnt(2)\n" \
    "ds_read_b128 v[112:115], v21 offset:20480\n" \
    "ds_read_b128 v[116:119], v22 offset:20480\n" \
    "s_waitcnt lgkmcnt(0)\n" \
    "s_cmp_eq_u32 s44, 2\n" \
    "s_cselect_b32 s30, s16, s30\n" \
    "s_cselect_b32 s31, s17, s31\n" \
    "global_store_dwordx4 v16, v[112:115], s[30:31] sc0 sc1\n" \
    "global_store_dwordx4 v16, v[116:119], s[30:31] offset:1024 sc0 sc1\n" \
    "ds_write_b32 v23, v17\n" \
    "v_mov_b32 v17, 10\n" \
    "s_waitcnt vmcnt(2)\n" \
    "ds_write_b32 v23, v17\n" \
    "v_mov_b32 v17, 11\n" \
    "s_waitcnt vmcnt(0)\n" \
    "ds_write_b32 v23, v17\n" \
    "s_branch a2a_end\n" \
      \
    "a2a_flagger:\n" \
    "v_min_u32 v16, 2, v1\n" \
    "v_lshlrev_b32 v23, 8, v16\n" \
    "v_add_u32 v23, %[lds], v23\n" \
    "v_add_u32 v23, 69632, v23\n" \
    "v_mov_b32 v17, 0\n" \
    "ds_write_b32 v23, v17\n" \
    "s_waitcnt lgkmcnt(0)\n" \
    "global_load_dword v10, v9, s[16:17] sc0 sc1\n" \
    "s_barrier\n" \
    "s_lshl_b32 s55, s20, 9\n" \
    "s_lshl_b32 s56, s46, 7\n" \
    "s_add_u32 s82, s55, s56\n" \
    "s_add_u32 s82, s82, 4194304\n" \
    "s_mov_b32 s73, 0\n" \
    "s_mov_b32 s67, 0\n" \
    "s_waitcnt vmcnt(0)\n" \
    "v_add_u32 v10, 1, v10\n" \
    "a2a_fl_round:\n" \
    "s_and_b32 s48, s42, 7\n" \
    "s_lshr_b32 s42, s42, 4\n" \
    "s_cmp_lt_u32 s48, 3\n" \
    "s_cselect_b32 s55, 4, 5\n" \
    "s_add_u32 s67, s67, s55\n" \
    "s_lshl_b32 s55, s48, 12\n" \
    "s_add_u32 s55, s55, s82\n" \
    "v_mov_b32 v15, s55\n" \
    "a2a_fl_spin:\n" \
    "ds_read_b32 v17, v23\n" \
    "s_waitcnt lgkmcnt(0)\n" \
    "v_mad_u32_u24 v17, v17, 3, v16\n" \
    "v_cmp_gt_u32 vcc, s67, v17\n" \
    "s_cmp_lg_u32 vcc_lo, 0\n" \
    "s_cbranch_scc1 a2a_fl_spin\n" \
    "s_mov_b64 exec, 1\n" \
    "global_store_dword v15, v10, s[30:31] sc0 sc1\n" \
    "s_mov_b64 exec, -1\n" \
    "s_add_u32 s73, s73, 1\n" \
    "s_cmp_lt_u32 s73, 7\n" \
    "s_cbranch_scc1 a2a_fl_round\n" \
    "s_mov_b64 exec, 1\n" \
    "global_store_dword v9, v10, s[16:17] sc1\n" \
    "s_mov_b64 exec, -1\n" \
    "s_lshl_b32 s55, s46, 15\n" \
    "v_lshl_add_u32 v16, v1, 17, s55\n" \
    "global_load_dword v20, v16, s[22:23]\n" \
    "s_waitcnt vmcnt(0)\n" \
    "s_branch a2a_end\n" \
      \
    "a2a_g_small:\n" \
    "s_lshl2_add_u32 s56, s48, s44\n" \
    "s_mul_i32 s56, s56, 43\n" \
    "v_min_u32 v21, 42, v1\n" \
    "v_add_u32 v21, s56, v21\n" \
    "v_min_u32 v21, 511, v21\n" \
    "s_lshl_b32 s56, s47, 7\n" \
    "v_lshl_add_u32 v21, v21, 9, s56\n" \
    "v_and_b32 v22, 3, v1\n" \
    "s_sub_u32 s55, %[bid], 160\n" \
    "s_lshl2_add_u32 s55, s55, s44\n" \
    "s_mul_i32 s55, s55, 2390\n" \
    "s_lshr_b32 s55, s55, 8\n" \
    "s_add_u32 s55, s55, 512\n" \
    "v_lshrrev_b32 v16, 2, v1\n" \
    "v_min_u32 v16, 9, v16\n" \
    "v_add_u32 v16, s55, v16\n" \
    "s_waitcnt lgkmcnt(0)\n" \
    "global_load_dword v20, v21, s[28:29]\n" \
    "s_lshl_b32 s55, s20, 9\n" \
    "v_add_u32 v16, s55, v16\n" \
    "v_and_b32 v16, 4095, v16\n" \
    "v_lshlrev_b32 v16, 12, v16\n" \
    "v_lshl_or_b32 v16, v22, 10, v16\n" \
    "s_sub_u32 s55, 384, s8\n" \
    "s_and_b32 s55, s55, 896\n" \
    "buffer_load_dword v20, v16, s[8:11], s55 offen\n" \
    "s_branch a2a_g_go\n" \
    ".p2align 7\n" \
      \
    "a2a_gemm:\n" \
    "v_and_b32 v1, 63, %[tid]\n" \
    "s_lshr_b32 s44, s44, 6\n" \
    "v_lshrrev_b32 v2, 3, v1\n" \
    "v_lshrrev_b32 v16, 4, v1\n" \
    "v_xor_b32 v3, v1, v16\n" \
    "v_and_b32 v3, 7, v3\n" \
    "v_lshlrev_b32 v3, 4, v3\n" \
    "v_lshl_or_b32 v6, v2, 9, v3\n" \
    "v_xor_b32 v7, 64, v6\n" \
    "v_add_u32 v7, 3072, v7\n" \
    "s_lshr_b32 s48, s46, 2\n" \
    "s_sub_u32 s48, 7, s48\n" \
    "s_lshl_b32 s55, s48, 2\n" \
    "s_add_u32 s56, s55, s48\n" \
    "s_sub_i32 s56, s56, 3\n" \
    "s_max_i32 s49, s55, s56\n" \
    "s_lshl_b32 s52, s44, 13\n" \
    "s_add_u32 s52, s52, %[lds]\n" \
    "s_add_u32 s52, s52, 131072\n" \
    "s_lshl_b32 s82, s49, 13\n" \
    "s_lshl_b32 s55, s44, 7\n" \
    "s_or_b32 s82, s82, s55\n" \
    "s_lshl_b32 s83, s47, 13\n" \
    "s_lshl_b32 s55, s47, 11\n" \
    "s_or_b32 s84, s52, s55\n" \
    "s_cmp_ge_u32 s48, 3\n" \
    "s_cselect_b32 s80, 1, 0\n" \
    "s_cbranch_scc0 a2a_g_small\n" \
    "s_waitcnt lgkmcnt(0)\n" \
    "a2a_g_go:\n" \
    "s_xor_b32 m0, s84, 0\n" \
    "s_xor_b32 s58, s83, 0\n" \
    "s_add_u32 s58, s58, s82\n" \
    "buffer_load_dwordx4 v6, s[12:15], s58 offen lds\n" \
    "buffer_load_dwordx4 v7, s[12:15], s58 offen offset:1024 lds\n" \
    "s_load_dwordx4 s[24:27], %[ka], 0x40\n" \
    "s_lshl_b32 s55, s48, 2\n" \
    "s_load_dword s61, %[ka], s55 offset:0xd0\n" \
    "s_lshl_b32 s51, s44, 15\n" \
    "s_add_u32 s51, s51, %[lds]\n" \
    "s_and_b32 s55, s48, 3\n" \
    "s_xor_b32 s55, s55, s44\n" \
    "s_lshl_b32 s56, s55, 12\n" \
    "s_or_b32 s54, s56, 16384\n" \
    "s_xor_b32 s76, s51, s56\n" \
    "s_lshl2_add_u32 s53, s44, s55\n" \
    "s_lshl_b32 s53, s53, 17\n" \
    "s_and_b32 s57, %[bid], 7\n" \
    "s_lshl2_add_u32 s57, s57, s47\n" \
    "s_lshl_b32 s57, s57, 7\n" \
    "v_lshl_or_b32 v4, v2, 12, v3\n" \
    "v_or_b32 v4, s57, v4\n" \
    "v_xor_b32 v5, 64, v4\n" \
    "s_lshl_b32 s77, s20, 21\n" \
    "s_add_u32 s77, s77, s53\n" \
    "s_xor_b32 m0, s76, 0\n" \
    "s_xor_b32 s58, s77, 0\n" \
    "buffer_load_dwordx4 v4, s[8:11], s58 offen offset:0 lds\n" \
    "s_add_u32 s58, s58, 31744\n" \
    "buffer_load_dwordx4 v4, s[8:11], s58 offen offset:1024 lds\n" \
    "s_add_u32 s58, s58, 31744\n" \
    "buffer_load_dwordx4 v5, s[8:11], s58 offen offset:2048 lds\n" \
    "s_add_u32 s58, s58, 31744\n" \
    "buffer_load_dwordx4 v5, s[8:11], s58 offen offset:3072 lds\n" \
    "s_lshl2_add_u32 s55, %[bid], 4227072\n" \
    "v_mov_b32 v9, s55\n" \
    "global_load_dword v10, v9, s[16:17] sc0 sc1\n" \
    "s_cmp_eq_u32 s80, 0\n" \
    "s_cbranch_scc1 a2a_g_x4\n" \
    "v_and_b32 v8, 15, v1\n" \
    "v_lshlrev_b32 v8, 9, v8\n" \
    "v_and_or_b32 v8, v1, 48, v8\n" \
    "s_add_u32 s58, s82, 32768\n" \
    "buffer_load_dwordx4 a[32:35], v8, s[12:15], s58 offen\n" \
    "buffer_load_dwordx4 a[36:39], v8, s[12:15], s58 offen offset:64\n" \
    "a2a_g_x4:\n" \
    "s_lshl_b32 s40, s57, 2\n" \
    "s_lshl_b32 s55, s49, 18\n" \
    "s_add_u32 s40, s40, s55\n" \
    "s_xor_b32 s79, s51, s54\n" \
    "s_xor_b32 m0, s84, 2048\n" \
    "s_xor_b32 s58, s83, 8192\n" \
    "s_add_u32 s58, s58, s82\n" \
    "buffer_load_dwordx4 v6, s[12:15], s58 offen lds\n" \
    "buffer_load_dwordx4 v7, s[12:15], s58 offen offset:1024 lds\n" \
    "s_xor_b32 m0, s84, 4096\n" \
    "s_xor_b32 s58, s83, 16384\n" \
    "s_add_u32 s58, s58, s82\n" \
    "buffer_load_dwordx4 v6, s[12:15], s58 offen lds\n" \
    "buffer_load_dwordx4 v7, s[12:15], s58 offen offset:1024 lds\n" \
    "s_waitcnt lgkmcnt(0)\n" \
    "s_load_dword s85, s[24:25], 0x0\n" \
    "s_load_dword s86, s[26:27], 0x0\n" \
    "s_xor_b32 m0, s84, 6144\n" \
    "s_xor_b32 s58, s83, 24576\n" \
    "s_add_u32 s58, s58, s82\n" \
    "buffer_load_dwordx4 v6, s[12:15], s58 offen lds\n" \
    "buffer_load_dwordx4 v7, s[12:15], s58 offen offset:1024 lds\n" \
    "s_xor_b32 m0, s76, 4096\n" \
    "s_xor_b32 s58, s77, 131072\n" \
    "buffer_load_dwordx4 v4, s[8:11], s58 offen offset:0 lds\n" \
    "s_add_u32 s58, s58, 31744\n" \
    "buffer_load_dwordx4 v4, s[8:11], s58 offen offset:1024 lds\n" \
    "s_add_u32 s58, s58, 31744\n" \
    "buffer_load_dwordx4 v5, s[8:11], s58 offen offset:2048 lds\n" \
    "s_add_u32 s58, s58, 31744\n" \
    "buffer_load_dwordx4 v5, s[8:11], s58 offen offset:3072 lds\n" \
    "s_xor_b32 m0, s76, 8192\n" \
    "s_xor_b32 s58, s77, 262144\n" \
    "buffer_load_dwordx4 v4, s[8:11], s58 offen offset:0 lds\n" \
    "s_add_u32 s58, s58, 31744\n" \
    "buffer_load_dwordx4 v4, s[8:11], s58 offen offset:1024 lds\n" \
    "s_add_u32 s58, s58, 31744\n" \
    "buffer_load_dwordx4 v5, s[8:11], s58 offen offset:2048 lds\n" \
    "s_add_u32 s58, s58, 31744\n" \
    "buffer_load_dwordx4 v5, s[8:11], s58 offen offset:3072 lds\n" \
    "s_xor_b32 m0, s76, 12288\n" \
    "s_xor_b32 s58, s77, 393216\n" \
    "buffer_load_dwordx4 v4, s[8:11], s58 offen offset:0 lds\n" \
    "s_add_u32 s58, s58, 31744\n" \
    "buffer_load_dwordx4 v4, s[8:11], s58 offen offset:1024 lds\n" \
    "s_add_u32 s58, s58, 31744\n" \
    "buffer_load_dwordx4 v5, s[8:11], s58 offen offset:2048 lds\n" \
    "s_add_u32 s58, s58, 31744\n" \
    "buffer_load_dwordx4 v5, s[8:11], s58 offen offset:3072 lds\n" \
    "s_mov_b32 s76, s79\n" \
    "s_and_b32 s77, s61, 7\n" \
    "s_lshl_b32 s77, s77, 21\n" \
    "s_add_u32 s77, s77, s53\n" \
    "s_xor_b32 m0, s76, 0\n" \
    "s_xor_b32 s58, s77, 0\n" \
    "buffer_load_dwordx4 v4, s[8:11], s58 offen lds\n" \
    "s_add_u32 s58, s58, 31744\n" \
    "buffer_load_dwordx4 v4, s[8:11], s58 offen offset:1024 lds\n" \
    "s_xor_b32 m0, s76, 2048\n" \
    "s_xor_b32 s58, s77, 65536\n" \
    "buffer_load_dwordx4 v5, s[8:11], s58 offen lds\n" \
    "s_add_u32 s58, s58, 31744\n" \
    "buffer_load_dwordx4 v5, s[8:11], s58 offen offset:1024 lds\n" \
    "s_xor_b32 m0, s76, 4096\n" \
    "s_xor_b32 s58, s77, 131072\n" \
    "buffer_load_dwordx4 v4, s[8:11], s58 offen lds\n" \
    "s_add_u32 s58, s58, 31744\n" \
    "buffer_load_dwordx4 v4, s[8:11], s58 offen offset:1024 lds\n" \
    "s_xor_b32 m0, s76, 6144\n" \
    "s_xor_b32 s58, s77, 196608\n" \
    "buffer_load_dwordx4 v5, s[8:11], s58 offen lds\n" \
    "s_add_u32 s58, s58, 31744\n" \
    "buffer_load_dwordx4 v5, s[8:11], s58 offen offset:1024 lds\n" \
    "s_xor_b32 m0, s76, 8192\n" \
    "s_xor_b32 s58, s77, 262144\n" \
    "buffer_load_dwordx4 v4, s[8:11], s58 offen lds\n" \
    "s_add_u32 s58, s58, 31744\n" \
    "buffer_load_dwordx4 v4, s[8:11], s58 offen offset:1024 lds\n" \
    "v_and_b32 v16, 15, v1\n" \
    "v_lshrrev_b32 v17, 1, v16\n" \
    "v_lshrrev_b32 v21, 4, v1\n" \
    "v_xor_b32 v21, v21, v17\n" \
    "v_lshlrev_b32 v21, 4, v21\n" \
    "v_lshl_or_b32 v21, v16, 7, v21\n" \
    "v_add_u32 v21, s52, v21\n" \
    "v_xor_b32 v22, 64, v21\n" \
    "v_and_b32 v16, 48, v1\n" \
    "v_or_b32 v13, v16, v17\n" \
    "v_lshlrev_b32 v13, 7, v13\n" \
    "v_and_b32 v16, 1, v1\n" \
    "v_lshl_or_b32 v13, v16, 3, v13\n" \
    "v_lshrrev_b32 v17, 1, v17\n" \
    "v_lshl_or_b32 v13, v17, 4, v13\n" \
    "v_and_b32 v16, 16, v1\n" \
    "v_lshl_or_b32 v13, v16, 2, v13\n" \
    "v_add_u32 v12, s51, v13\n" \
    "v_lshlrev_b32 v14, 4, v1\n" \
    "v_min_u32 v16, 6, v1\n" \
    "v_lshlrev_b32 v16, 2, v16\n" \
    "v_lshrrev_b32 v15, v16, s61\n" \
    "v_and_b32 v15, 7, v15\n" \
    "v_lshlrev_b32 v15, 9, v15\n" \
    "s_lshl_b32 s55, s48, 12\n" \
    "s_lshl_b32 s56, s44, 7\n" \
    "s_add_u32 s55, s55, s56\n" \
    "s_add_u32 s55, s55, 4194304\n" \
    "s_add_u32 s38, s16, s55\n" \
    "s_addc_u32 s39, s17, 0\n" \
    "s_add_u32 s22, s22, s40\n" \
    "s_addc_u32 s23, s23, 0\n" \
    "s_add_u32 s40, s22, 32768\n" \
    "s_addc_u32 s41, s23, 0\n" \
    "s_lshl_b32 s62, s44, 16\n" \
    "s_lshl_b32 s55, s49, 11\n" \
    "s_add_u32 s62, s62, s55\n" \
    "s_memrealtime s[36:37]\n" \
    "s_mov_b32 s66, 0\n" \
    "s_mov_b32 s68, 0\n" \
    "s_mov_b32 s67, 0x7c\n" \
    "s_mov_b32 s69, 1\n" \
    "s_mov_b32 s70, 7\n" \
    "s_mov_b32 s88, 7\n" \
    "s_mov_b32 s71, 0\n" \
    "s_mov_b32 s73, 0\n" \
    "s_mov_b32 s72, s51\n" \
    "s_mov_b32 s75, 0\n" \
    "s_mov_b32 s89, 0\n" \
    "s_waitcnt lgkmcnt(0)\n" \
    "s_mov_b32 s60, s36\n" \
    "v_mov_b32 v28, s85\n" \
    "v_mul_f32 v28, s86, v28\n" \
    "v_mov_b32 v29, v28\n" \
    "v_mul_f32 v30, 0xbfb8aa3b, v28\n" \
    "v_mov_b32 v31, v30\n" \
    "v_mov_b32 v25, 0x42ad1f97\n" \
    "s_waitcnt vmcnt(10)\n" \
    "ds_read_b128 a[0:3], v21 offset:0\n" \
    "ds_read_b128 a[4:7], v22 offset:0\n" \
    "ds_read_b128 a[8:11], v21 offset:2048\n" \
    "ds_read_b128 a[12:15], v22 offset:2048\n" \
    "ds_read_b128 a[16:19], v21 offset:4096\n" \
    "ds_read_b128 a[20:23], v22 offset:4096\n" \
    "ds_read_b128 a[24:27], v21 offset:6144\n" \
    "ds_read_b128 a[28:31], v22 offset:6144\n" \
    "v_readfirstlane_b32 s59, v10\n" \
    "v_mov_b32 v10, 0\n" \
    "v_xor_b32 v11, 0, v12\n" \
    "ds_read_b64_tr_b8 a[80:81], v11\n" \
    "ds_read_b64_tr_b8 a[82:83], v11 offset:1024\n" \
    "ds_read_b64_tr_b8 a[84:85], v11 offset:8192\n" \
    "ds_read_b64_tr_b8 a[86:87], v11 offset:9216\n" \
    "v_xor_b32 v11, 16, v12\n" \
    "ds_read_b64_tr_b8 a[88:89], v11\n" \
    "ds_read_b64_tr_b8 a[90:91], v11 offset:1024\n" \
    "ds_read_b64_tr_b8 a[92:93], v11 offset:8192\n" \
    "ds_read_b64_tr_b8 a[94:95], v11 offset:9216\n" \
    "v_xor_b32 v11, 32, v12\n" \
    "ds_read_b64_tr_b8 a[96:97], v11\n" \
    "ds_read_b64_tr_b8 a[98:99], v11 offset:1024\n" \
    "ds_read_b64_tr_b8 a[100:101], v11 offset:8192\n" \
    "ds_read_b64_tr_b8 a[102:103], v11 offset:9216\n" \
    "v_xor_b32 v11, 48, v12\n" \
    "ds_read_b64_tr_b8 a[104:105], v11\n" \
    "ds_read_b64_tr_b8 a[106:107], v11 offset:1024\n" \
    "ds_read_b64_tr_b8 a[108:109], v11 offset:8192\n" \
    "ds_read_b64_tr_b8 a[110:111], v11 offset:9216\n" \
    "v_xor_b32 v11, 64, v12\n" \
    "ds_read_b64_tr_b8 a[112:113], v11\n" \
    "ds_read_b64_tr_b8 a[114:115], v11 offset:1024\n" \
    "ds_read_b64_tr_b8 a[116:117], v11 offset:8192\n" \
    "ds_read_b64_tr_b8 a[118:119], v11 offset:9216\n" \
    "v_xor_b32 v11, 80, v12\n" \
    "ds_read_b64_tr_b8 a[120:121], v11\n" \
    "ds_read_b64_tr_b8 a[122:123], v11 offset:1024\n" \
    "ds_read_b64_tr_b8 a[124:125], v11 offset:8192\n" \
    "ds_read_b64_tr_b8 a[126:127], v11 offset:9216\n" \
    "v_xor_b32 v11, 96, v12\n" \
    "ds_read_b64_tr_b8 a[128:129], v11\n" \
    "ds_read_b64_tr_b8 a[130:131], v11 offset:1024\n" \
    "ds_read_b64_tr_b8 a[132:133], v11 offset:8192\n" \
    "ds_read_b64_tr_b8 a[134:135], v11 offset:9216\n" \
    "v_xor_b32 v11, 112, v12\n" \
    "ds_read_b64_tr_b8 a[136:137], v11\n" \
    "ds_read_b64_tr_b8 a[138:139], v11 offset:1024\n" \
    "ds_read_b64_tr_b8 a[140:141], v11 offset:8192\n" \
    "ds_read_b64_tr_b8 a[142:143], v11 offset:9216\n" \
    "s_add_u32 s59, s59, 1\n" \
    "s_and_b32 s55, s59, 1\n" \
    "s_lshl_b32 s55, s55, 21\n" \
    "s_add_u32 s62, s62, s55\n" \
    "s_lshl_b32 s63, s71, 2\n" \
    "s_lshr_b32 s63, s61, s63\n" \
    "s_and_b32 s63, s63, 7\n" \
    "s_lshl_b32 s63, s63, 18\n" \
    "s_add_u32 s63, s63, s62\n" \
    "s_add_u32 s64, s63, 4096\n" \
    "s_add_u32 s65, s63, 8192\n" \
    "global_load_dword v10, v15, s[38:39] sc0 sc1\n" \
    "s_waitcnt lgkmcnt(0)\n" \
    "v_mfma_f32_16x16x128_f8f6f4 v[96:99], a[0:7], a[80:87], 0 cbsz:1 blgp:1\n" \
    "v_mfma_f32_16x16x128_f8f6f4 v[100:103], a[0:7], a[88:95], 0 cbsz:1 blgp:1\n" \
    "v_mfma_f32_16x16x128_f8f6f4 v[104:107], a[0:7], a[96:103], 0 cbsz:1 blgp:1\n" \
    "v_mfma_f32_16x16x128_f8f6f4 v[108:111], a[0:7], a[104:111], 0 cbsz:1 blgp:1\n" \
    "s_xor_b32 m0, s76, 10240\n" \
    "s_xor_b32 s58, s77, 327680\n" \
    "buffer_load_dwordx4 v5, s[8:11], s58 offen lds\n" \
    "s_add_u32 s58, s58, 31744\n" \
    "buffer_load_dwordx4 v5, s[8:11], s58 offen offset:1024 lds\n" \
    "v_mfma_f32_16x16x128_f8f6f4 v[112:115], a[0:7], a[112:119], 0 cbsz:1 blgp:1\n" \
    "v_mfma_f32_16x16x128_f8f6f4 v[116:119], a[0:7], a[120:127], 0 cbsz:1 blgp:1\n" \
    "v_mfma_f32_16x16x128_f8f6f4 v[120:123], a[0:7], a[128:135], 0 cbsz:1 blgp:1\n" \
    "v_mfma_f32_16x16x128_f8f6f4 v[124:127], a[0:7], a[136:143], 0 cbsz:1 blgp:1\n" \
    "v_mfma_f32_16x16x128_f8f6f4 v[128:131], a[8:15], a[80:87], 0 cbsz:1 blgp:1\n" \
    "v_mfma_f32_16x16x128_f8f6f4 v[132:135], a[8:15], a[88:95], 0 cbsz:1 blgp:1\n" \
    "v_mfma_f32_16x16x128_f8f6f4 v[136:139], a[8:15], a[96:103], 0 cbsz:1 blgp:1\n" \
    "v_mfma_f32_16x16x128_f8f6f4 v[140:143], a[8:15], a[104:111], 0 cbsz:1 blgp:1\n" \
    "v_mfma_f32_16x16x128_f8f6f4 v[144:147], a[8:15], a[112:119], 0 cbsz:1 blgp:1\n" \
    "v_mfma_f32_16x16x128_f8f6f4 v[148:151], a[8:15], a[120:127], 0 cbsz:1 blgp:1\n" \
    "s_xor_b32 m0, s76, 12288\n" \
    "s_xor_b32 s58, s77, 393216\n" \
    "buffer_load_dwordx4 v4, s[8:11], s58 offen lds\n" \
    "s_add_u32 s58, s58, 31744\n" \
    "buffer_load_dwordx4 v4, s[8:11], s58 offen offset:1024 lds\n" \
    "v_mfma_f32_16x16x128_f8f6f4 v[152:155], a[8:15], a[128:135], 0 cbsz:1 blgp:1\n" \
    "v_mfma_f32_16x16x128_f8f6f4 v[156:159], a[8:15], a[136:143], 0 cbsz:1 blgp:1\n" \
    "v_mfma_f32_16x16x128_f8f6f4 v[160:163], a[16:23], a[80:87], 0 cbsz:1 blgp:1\n" \
    "v_mfma_f32_16x16x128_f8f6f4 v[164:167], a[16:23], a[88:95], 0 cbsz:1 blgp:1\n" \
    "v_mfma_f32_16x16x128_f8f6f4 v[168:171], a[16:23], a[96:103], 0 cbsz:1 blgp:1\n" \
    "v_mfma_f32_16x16x128_f8f6f4 v[172:175], a[16:23], a[104:111], 0 cbsz:1 blgp:1\n" \
    "v_mfma_f32_16x16x128_f8f6f4 v[176:179], a[16:23], a[112:119], 0 cbsz:1 blgp:1\n" \
    "v_mfma_f32_16x16x128_f8f6f4 v[180:183], a[16:23], a[120:127], 0 cbsz:1 blgp:1\n" \
    "v_mfma_f32_16x16x128_f8f6f4 v[184:187], a[16:23], a[128:135], 0 cbsz:1 blgp:1\n" \
    "v_mfma_f32_16x16x128_f8f6f4 v[188:191], a[16:23], a[136:143], 0 cbsz:1 blgp:1\n" \
    "s_xor_b32 m0, s76, 14336\n" \
    "s_xor_b32 s58, s77, 458752\n" \
    "buffer_load_dwordx4 v5, s[8:11], s58 offen lds\n" \
    "s_add_u32 s58, s58, 31744\n" \
    "buffer_load_dwordx4 v5, s[8:11], s58 offen offset:1024 lds\n" \
    "v_mfma_f32_16x16x128_f8f6f4 v[192:195], a[24:31], a[80:87], 0 cbsz:1 blgp:1\n" \
    "v_mfma_f32_16x16x128_f8f6f4 v[196:199], a[24:31], a[88:95], 0 cbsz:1 blgp:1\n" \
    "v_mfma_f32_16x16x128_f8f6f4 v[200:203], a[24:31], a[96:103], 0 cbsz:1 blgp:1\n" \
    "v_mfma_f32_16x16x128_f8f6f4 v[204:207], a[24:31], a[104:111], 0 cbsz:1 blgp:1\n" \
    "v_mfma_f32_16x16x128_f8f6f4 v[208:211], a[24:31], a[112:119], 0 cbsz:1 blgp:1\n" \
    "v_mfma_f32_16x16x128_f8f6f4 v[212:215], a[24:31], a[120:127], 0 cbsz:1 blgp:1\n" \
    "v_mfma_f32_16x16x128_f8f6f4 v[216:219], a[24:31], a[128:135], 0 cbsz:1 blgp:1\n" \
    "v_mfma_f32_16x16x128_f8f6f4 v[220:223], a[24:31], a[136:143], 0 cbsz:1 blgp:1\n" \
    "s_cmp_eq_u32 s80, 0\n" \
    "s_cbranch_scc1 a2a_own4\n" \
    "v_mfma_f32_16x16x128_f8f6f4 v[224:227], a[32:39], a[80:87], 0 cbsz:1 blgp:1\n" \
    "v_mfma_f32_16x16x128_f8f6f4 v[228:231], a[32:39], a[88:95], 0 cbsz:1 blgp:1\n" \
    "v_mfma_f32_16x16x128_f8f6f4 v[232:235], a[32:39], a[96:103], 0 cbsz:1 blgp:1\n" \
    "v_mfma_f32_16x16x128_f8f6f4 v[236:239], a[32:39], a[104:111], 0 cbsz:1 blgp:1\n" \
    "v_mfma_f32_16x16x128_f8f6f4 v[240:243], a[32:39], a[112:119], 0 cbsz:1 blgp:1\n" \
    "v_mfma_f32_16x16x128_f8f6f4 v[244:247], a[32:39], a[120:127], 0 cbsz:1 blgp:1\n" \
    "v_mfma_f32_16x16x128_f8f6f4 v[248:251], a[32:39], a[128:135], 0 cbsz:1 blgp:1\n" \
    "v_mfma_f32_16x16x128_f8f6f4 v[252:255], a[32:39], a[136:143], 0 cbsz:1 blgp:1\n" \
      \
    "a2a_own4:\n" \
    "s_waitcnt vmcnt(6)\n" \
    "v_cmp_le_u32 vcc, s59, v10\n" \
    "s_and_b32 s55, vcc_lo, 0x7f\n" \
    "s_or_b32 s66, s66, s55\n" \
    "s_mov_b32 s79, 1\n" \
    "s_lshl_b32 s77, s79, 2\n" \
    "s_lshr_b32 s77, s61, s77\n" \
    "s_and_b32 s77, s77, 7\n" \
    "s_lshl_b32 s77, s77, 21\n" \
    "s_add_u32 s77, s77, s53\n" \
    "s_xor_b32 s76, s76, 16384\n" \
    "v_xor_b32 v12, 16384, v12\n" \
    "s_and_b32 s74, s66, 1\n" \
    "s_cbranch_scc0 a2a_own_n\n" \
    "buffer_load_dwordx4 a[40:43], v14, s[16:19], s63 offen offset:0 sc0 sc1\n" \
    "buffer_load_dwordx4 a[44:47], v14, s[16:19], s63 offen offset:1024 sc0 sc1\n" \
    "buffer_load_dwordx4 a[48:51], v14, s[16:19], s63 offen offset:2048 sc0 sc1\n" \
    "buffer_load_dwordx4 a[52:55], v14, s[16:19], s63 offen offset:3072 sc0 sc1\n" \
    "buffer_load_dwordx4 a[56:59], v14, s[16:19], s64 offen offset:0 sc0 sc1\n" \
    "buffer_load_dwordx4 a[60:63], v14, s[16:19], s64 offen offset:1024 sc0 sc1\n" \
    "buffer_load_dwordx4 a[64:67], v14, s[16:19], s64 offen offset:2048 sc0 sc1\n" \
    "buffer_load_dwordx4 a[68:71], v14, s[16:19], s64 offen offset:3072 sc0 sc1\n" \
    "s_cmp_eq_u32 s80, 0\n" \
    "s_cbranch_scc1 a2a_own_n\n" \
    "buffer_load_dwordx4 a[72:75], v14, s[16:19], s65 offen offset:0 sc0 sc1\n" \
    "buffer_load_dwordx4 a[76:79], v14, s[16:19], s65 offen offset:1024 sc0 sc1\n" \
    "a2a_own_n:\n" \
    "global_load_dword v10, v15, s[38:39] sc0 sc1\n" \
    "s_xor_b32 m0, s76, 0\n" \
    "s_xor_b32 s58, s77, 0\n" \
    "buffer_load_dwordx4 v4, s[8:11], s58 offen offset:0 lds\n" \
    "s_add_u32 s58, s58, 31744\n" \
    "buffer_load_dwordx4 v4, s[8:11], s58 offen offset:1024 lds\n" \
    "s_add_u32 s58, s58, 31744\n" \
    "buffer_load_dwordx4 v5, s[8:11], s58 offen offset:2048 lds\n" \
    "s_add_u32 s58, s58, 31744\n" \
    "buffer_load_dwordx4 v5, s[8:11], s58 offen offset:3072 lds\n" \
    "s_xor_b32 m0, s76, 4096\n" \
    "s_xor_b32 s58, s77, 131072\n" \
    "buffer_load_dwordx4 v4, s[8:11], s58 offen offset:0 lds\n" \
    "s_add_u32 s58, s58, 31744\n" \
    "buffer_load_dwordx4 v4, s[8:11], s58 offen offset:1024 lds\n" \
    "s_add_u32 s58, s58, 31744\n" \
    "buffer_load_dwordx4 v5, s[8:11], s58 offen offset:2048 lds\n" \
    "s_add_u32 s58, s58, 31744\n" \
    "buffer_load_dwordx4 v5, s[8:11], s58 offen offset:3072 lds\n" \
    "s_waitcnt vmcnt(8)\n" \
    "v_xor_b32 v11, 0, v12\n" \
    "ds_read_b64_tr_b8 a[144:145], v11\n" \
    "ds_read_b64_tr_b8 a[146:147], v11 offset:1024\n" \
    "ds_read_b64_tr_b8 a[148:149], v11 offset:8192\n" \
    "ds_read_b64_tr_b8 a[150:151], v11 offset:9216\n" \
    "s_xor_b32 m0, s76, 8192\n" \
    "s_xor_b32 s58, s77, 262144\n" \
    "buffer_load_dwordx4 v4, s[8:11], s58 offen offset:0 lds\n" \
    "v_xor_b32 v11, 16, v12\n" \
    "ds_read_b64_tr_b8 a[152:153], v11\n" \
    "ds_read_b64_tr_b8 a[154:155], v11 offset:1024\n" \
    "ds_read_b64_tr_b8 a[156:157], v11 offset:8192\n" \
    "ds_read_b64_tr_b8 a[158:159], v11 offset:9216\n" \
    "s_add_u32 s58, s58, 31744\n" \
    "buffer_load_dwordx4 v4, s[8:11], s58 offen offset:1024 lds\n" \
    "v_xor_b32 v11, 32, v12\n" \
    "ds_read_b64_tr_b8 a[160:161], v11\n" \
    "ds_read_b64_tr_b8 a[162:163], v11 offset:1024\n" \
    "ds_read_b64_tr_b8 a[164:165], v11 offset:8192\n" \
    "ds_read_b64_tr_b8 a[166:167], v11 offset:9216\n" \
    "s_add_u32 s58, s58, 31744\n" \
    "buffer_load_dwordx4 v5, s[8:11], s58 offen offset:2048 lds\n" \
    "v_xor_b32 v11, 48, v12\n" \
    "ds_read_b64_tr_b8 a[168:169], v11\n" \
    "ds_read_b64_tr_b8 a[170:171], v11 offset:1024\n" \
    "ds_read_b64_tr_b8 a[172:173], v11 offset:8192\n" \
    "ds_read_b64_tr_b8 a[174:175], v11 offset:9216\n" \
    "s_add_u32 s58, s58, 31744\n" \
    "buffer_load_dwordx4 v5, s[8:11], s58 offen offset:3072 lds\n" \
    "v_xor_b32 v11, 64, v12\n" \
    "ds_read_b64_tr_b8 a[176:177], v11\n" \
    "ds_read_b64_tr_b8 a[178:179], v11 offset:1024\n" \
    "ds_read_b64_tr_b8 a[180:181], v11 offset:8192\n" \
    "ds_read_b64_tr_b8 a[182:183], v11 offset:9216\n" \
    "s_xor_b32 m0, s76, 12288\n" \
    "s_xor_b32 s58, s77, 393216\n" \
    "buffer_load_dwordx4 v4, s[8:11], s58 offen offset:0 lds\n" \
    "v_xor_b32 v11, 80, v12\n" \
    "ds_read_b64_tr_b8 a[184:185], v11\n" \
    "ds_read_b64_tr_b8 a[186:187], v11 offset:1024\n" \
    "ds_read_b64_tr_b8 a[188:189], v11 offset:8192\n" \
    "ds_read_b64_tr_b8 a[190:191], v11 offset:9216\n" \
    "s_add_u32 s58, s58, 31744\n" \
    "buffer_load_dwordx4 v4, s[8:11], s58 offen offset:1024 lds\n" \
    "v_xor_b32 v11, 96, v12\n" \
    "ds_read_b64_tr_b8 a[192:193], v11\n" \
    "ds_read_b64_tr_b8 a[194:195], v11 offset:1024\n" \
    "ds_read_b64_tr_b8 a[196:197], v11 offset:8192\n" \
    "ds_read_b64_tr_b8 a[198:199], v11 offset:9216\n" \
    "s_add_u32 s58, s58, 31744\n" \
    "buffer_load_dwordx4 v5, s[8:11], s58 offen offset:2048 lds\n" \
    "v_xor_b32 v11, 112, v12\n" \
    "ds_read_b64_tr_b8 a[200:201], v11\n" \
    "ds_read_b64_tr_b8 a[202:203], v11 offset:1024\n" \
    "ds_read_b64_tr_b8 a[204:205], v11 offset:8192\n" \
    "ds_read_b64_tr_b8 a[206:207], v11 offset:9216\n" \
    "s_add_u32 s58, s58, 31744\n" \
    "buffer_load_dwordx4 v5, s[8:11], s58 offen offset:3072 lds\n" \
    "s_waitcnt lgkmcnt(0)\n" \
    "s_cmp_eq_u32 s74, 0\n" \
    "s_cbranch_scc1 a2a_stall\n" \
    "s_branch a2a_c0\n" \
      \
    "a2a_u1_top:\n" \
    "v_mfma_f32_16x16x128_f8f6f4 v[96:99], a[40:47], a[144:151], v[96:99] cbsz:1 blgp:1\n" \
    "v_cmp_le_u32 vcc, s59, v10\n" \
    "s_and_b32 s55, vcc_lo, 0x7f\n" \
    "s_or_b32 s66, s66, s55\n" \
    "s_lshr_b32 s55, s66, s70\n" \
    "v_mfma_f32_16x16x128_f8f6f4 v[100:103], a[40:47], a[152:159], v[100:103] cbsz:1 blgp:1\n" \
    "s_lshr_b32 s56, s66, s69\n" \
    "s_andn2_b32 s55, s55, s56\n" \
    "s_bitcmp1_b32 s55, 0\n" \
    "s_cselect_b32 s71, s70, s69\n" \
    "v_mfma_f32_16x16x128_f8f6f4 v[104:107], a[40:47], a[160:167], v[104:107] cbsz:1 blgp:1\n" \
    "s_cselect_b32 s70, s69, s70\n" \
    "s_cselect_b32 s57, 16384, 0\n" \
    "s_mov_b32 s69, s71\n" \
    "s_xor_b32 s72, s72, s57\n" \
    "s_cmp_lt_u32 s88, 7\n" \
    "s_cbranch_scc1 a2a_u1_pk\n" \
    "a2a_u1_pkr:\n" \
    "v_mfma_f32_16x16x128_f8f6f4 v[108:111], a[40:47], a[168:175], v[108:111] cbsz:1 blgp:1\n" \
    "s_lshl_b32 s63, s71, 2\n" \
    "s_lshr_b32 s63, s61, s63\n" \
    "s_and_b32 s63, s63, 7\n" \
    "s_lshl_b32 s63, s63, 18\n" \
    "s_add_u32 s63, s63, s62\n" \
    "s_add_u32 s64, s63, 4096\n" \
    "s_add_u32 s65, s63, 8192\n" \
    "s_bitcmp1_b32 s66, s71\n" \
    "s_cbranch_scc0 a2a_u1_absent\n" \
    "v_mfma_f32_16x16x128_f8f6f4 v[112:115], a[40:47], a[176:183], v[112:115] cbsz:1 blgp:1\n" \
    "buffer_load_dwordx4 a[0:3], v14, s[16:19], s63 offen offset:0 sc0 sc1\n" \
    "s_andn2_b32 s55, s67, s68\n" \
    "s_cselect_b32 s55, s55, s67\n" \
    "s_and_b32 s56, s67, s66\n" \
    "v_mfma_f32_16x16x128_f8f6f4 v[116:119], a[40:47], a[184:191], v[116:119] cbsz:1 blgp:1\n" \
    "buffer_load_dwordx4 a[4:7], v14, s[16:19], s63 offen offset:1024 sc0 sc1\n" \
    "s_cselect_b32 s55, s56, s55\n" \
    "s_or_b32 s55, s55, 0x80\n" \
    "s_ff1_i32_b32 s79, s55\n" \
    "v_mfma_f32_16x16x128_f8f6f4 v[120:123], a[40:47], a[192:199], v[120:123] cbsz:1 blgp:1\n" \
    "buffer_load_dwordx4 a[8:11], v14, s[16:19], s63 offen offset:2048 sc0 sc1\n" \
    "s_lshl_b32 s77, s79, 2\n" \
    "s_lshr_b32 s77, s61, s77\n" \
    "s_and_b32 s77, s77, 7\n" \
    "v_mfma_f32_16x16x128_f8f6f4 v[124:127], a[40:47], a[200:207], v[124:127] cbsz:1 blgp:1\n" \
    "buffer_load_dwordx4 a[12:15], v14, s[16:19], s63 offen offset:3072 sc0 sc1\n" \
    "s_lshl_b32 s77, s77, 21\n" \
    "s_add_u32 s77, s77, s53\n" \
    "s_cmp_eq_u32 s66, 0x7f\n" \
    "v_mfma_f32_16x16x128_f8f6f4 v[128:131], a[48:55], a[144:151], v[128:131] cbsz:1 blgp:1\n" \
    "buffer_load_dwordx4 a[16:19], v14, s[16:19], s64 offen offset:0 sc0 sc1\n" \
    "s_cselect_b32 s55, 0, -1\n" \
    "v_and_b32 v15, s55, v15\n" \
    "s_mov_b32 s74, 1\n" \
    "v_mfma_f32_16x16x128_f8f6f4 v[132:135], a[48:55], a[152:159], v[132:135] cbsz:1 blgp:1\n" \
    "buffer_load_dwordx4 a[20:23], v14, s[16:19], s64 offen offset:1024 sc0 sc1\n" \
    "s_xor_b32 s76, s72, s54\n" \
    "v_add_u32 v12, s72, v13\n" \
    "s_cmp_eq_u32 s70, 7\n" \
    "v_mfma_f32_16x16x128_f8f6f4 v[136:139], a[48:55], a[160:167], v[136:139] cbsz:1 blgp:1\n" \
    "buffer_load_dwordx4 a[24:27], v14, s[16:19], s64 offen offset:2048 sc0 sc1\n" \
    "s_cselect_b32 s55, s67, 0\n" \
    "s_cmp_eq_u32 s55, 0\n" \
    "v_mfma_f32_16x16x128_f8f6f4 v[140:143], a[48:55], a[168:175], v[140:143] cbsz:1 blgp:1\n" \
    "buffer_load_dwordx4 a[28:31], v14, s[16:19], s64 offen offset:3072 sc0 sc1\n" \
    "s_cbranch_scc1 a2a_u1_nolw\n" \
    "s_bitset0_b32 s67, s79\n" \
    "s_mov_b32 s70, s79\n" \
    "s_cmp_eq_u32 s80, 0\n" \
    "s_cbranch_scc1 a2a_u1_alt4\n" \
    "v_mfma_f32_16x16x128_f8f6f4 v[224:227], a[72:79], a[144:151], v[224:227] cbsz:1 blgp:1\n" \
    "buffer_load_dwordx4 a[32:35], v14, s[16:19], s65 offen offset:0 sc0 sc1\n" \
    "v_mfma_f32_16x16x128_f8f6f4 v[228:231], a[72:79], a[152:159], v[228:231] cbsz:1 blgp:1\n" \
    "buffer_load_dwordx4 a[36:39], v14, s[16:19], s65 offen offset:1024 sc0 sc1\n" \
    "v_mfma_f32_16x16x128_f8f6f4 v[232:235], a[72:79], a[160:167], v[232:235] cbsz:1 blgp:1\n" \
    "s_xor_b32 m0, s76, 0\n" \
    "s_xor_b32 s58, s77, 0\n" \
    "buffer_load_dwordx4 v4, s[8:11], s58 offen offset:0 lds\n" \
    "v_mfma_f32_16x16x128_f8f6f4 v[236:239], a[72:79], a[168:175], v[236:239] cbsz:1 blgp:1\n" \
    "s_add_u32 s58, s58, 31744\n" \
    "buffer_load_dwordx4 v4, s[8:11], s58 offen offset:1024 lds\n" \
    "v_mfma_f32_16x16x128_f8f6f4 v[240:243], a[72:79], a[176:183], v[240:243] cbsz:1 blgp:1\n" \
    "s_add_u32 s58, s58, 31744\n" \
    "buffer_load_dwordx4 v5, s[8:11], s58 offen offset:2048 lds\n" \
    "v_mfma_f32_16x16x128_f8f6f4 v[244:247], a[72:79], a[184:191], v[244:247] cbsz:1 blgp:1\n" \
    "s_add_u32 s58, s58, 31744\n" \
    "buffer_load_dwordx4 v5, s[8:11], s58 offen offset:3072 lds\n" \
    "v_mfma_f32_16x16x128_f8f6f4 v[248:251], a[72:79], a[192:199], v[248:251] cbsz:1 blgp:1\n" \
    "s_xor_b32 m0, s76, 4096\n" \
    "s_xor_b32 s58, s77, 131072\n" \
    "buffer_load_dwordx4 v4, s[8:11], s58 offen offset:0 lds\n" \
    "v_mfma_f32_16x16x128_f8f6f4 v[252:255], a[72:79], a[200:207], v[252:255] cbsz:1 blgp:1\n" \
    "s_add_u32 s58, s58, 31744\n" \
    "buffer_load_dwordx4 v4, s[8:11], s58 offen offset:1024 lds\n" \
    "a2a_u1_a5:\n" \
    "v_mfma_f32_16x16x128_f8f6f4 v[144:147], a[48:55], a[176:183], v[144:147] cbsz:1 blgp:1\n" \
    "s_add_u32 s58, s58, 31744\n" \
    "buffer_load_dwordx4 v5, s[8:11], s58 offen offset:2048 lds\n" \
    "v_mfma_f32_16x16x128_f8f6f4 v[148:151], a[48:55], a[184:191], v[148:151] cbsz:1 blgp:1\n" \
    "s_add_u32 s58, s58, 31744\n" \
    "buffer_load_dwordx4 v5, s[8:11], s58 offen offset:3072 lds\n" \
    "v_mfma_f32_16x16x128_f8f6f4 v[152:155], a[48:55], a[192:199], v[152:155] cbsz:1 blgp:1\n" \
    "global_load_dword v10, v15, s[38:39] sc0 sc1\n" \
    "s_cmp_lg_u32 s89, 0\n" \
    "s_cbranch_scc1 a2a_u1_h2p\n" \
    "s_waitcnt vmcnt(17)\n" \
    "v_mfma_f32_16x16x128_f8f6f4 v[156:159], a[48:55], a[200:207], v[156:159] cbsz:1 blgp:1\n" \
    "v_xor_b32 v11, 0, v12\n" \
    "ds_read_b64_tr_b8 a[80:81], v11\n" \
    "ds_read_b64_tr_b8 a[82:83], v11 offset:1024\n" \
    "v_mfma_f32_16x16x128_f8f6f4 v[160:163], a[56:63], a[144:151], v[160:163] cbsz:1 blgp:1\n" \
    "s_xor_b32 m0, s76, 8192\n" \
    "s_xor_b32 s58, s77, 262144\n" \
    "buffer_load_dwordx4 v4, s[8:11], s58 offen offset:0 lds\n" \
    "ds_read_b64_tr_b8 a[84:85], v11 offset:8192\n" \
    "ds_read_b64_tr_b8 a[86:87], v11 offset:9216\n" \
    "v_mfma_f32_16x16x128_f8f6f4 v[164:167], a[56:63], a[152:159], v[164:167] cbsz:1 blgp:1\n" \
    "v_xor_b32 v11, 16, v12\n" \
    "ds_read_b64_tr_b8 a[88:89], v11\n" \
    "ds_read_b64_tr_b8 a[90:91], v11 offset:1024\n" \
    "v_mfma_f32_16x16x128_f8f6f4 v[168:171], a[56:63], a[160:167], v[168:171] cbsz:1 blgp:1\n" \
    "s_add_u32 s58, s58, 31744\n" \
    "buffer_load_dwordx4 v4, s[8:11], s58 offen offset:1024 lds\n" \
    "ds_read_b64_tr_b8 a[92:93], v11 offset:8192\n" \
    "ds_read_b64_tr_b8 a[94:95], v11 offset:9216\n" \
    "v_mfma_f32_16x16x128_f8f6f4 v[172:175], a[56:63], a[168:175], v[172:175] cbsz:1 blgp:1\n" \
    "v_xor_b32 v11, 32, v12\n" \
    "ds_read_b64_tr_b8 a[96:97], v11\n" \
    "ds_read_b64_tr_b8 a[98:99], v11 offset:1024\n" \
    "v_mfma_f32_16x16x128_f8f6f4 v[176:179], a[56:63], a[176:183], v[176:179] cbsz:1 blgp:1\n" \
    "s_add_u32 s58, s58, 31744\n" \
    "buffer_load_dwordx4 v5, s[8:11], s58 offen offset:2048 lds\n" \
    "ds_read_b64_tr_b8 a[100:101], v11 offset:8192\n" \
    "ds_read_b64_tr_b8 a[102:103], v11 offset:9216\n" \
    "v_mfma_f32_16x16x128_f8f6f4 v[180:183], a[56:63], a[184:191], v[180:183] cbsz:1 blgp:1\n" \
    "v_xor_b32 v11, 48, v12\n" \
    "ds_read_b64_tr_b8 a[104:105], v11\n" \
    "ds_read_b64_tr_b8 a[106:107], v11 offset:1024\n" \
    "v_mfma_f32_16x16x128_f8f6f4 v[184:187], a[56:63], a[192:199], v[184:187] cbsz:1 blgp:1\n" \
    "s_add_u32 s58, s58, 31744\n" \
    "buffer_load_dwordx4 v5, s[8:11], s58 offen offset:3072 lds\n" \
    "ds_read_b64_tr_b8 a[108:109], v11 offset:8192\n" \
    "ds_read_b64_tr_b8 a[110:111], v11 offset:9216\n" \
    "v_mfma_f32_16x16x128_f8f6f4 v[188:191], a[56:63], a[200:207], v[188:191] cbsz:1 blgp:1\n" \
    "v_xor_b32 v11, 64, v12\n" \
    "ds_read_b64_tr_b8 a[112:113], v11\n" \
    "ds_read_b64_tr_b8 a[114:115], v11 offset:1024\n" \
    "v_mfma_f32_16x16x128_f8f6f4 v[192:195], a[64:71], a[144:151], v[192:195] cbsz:1 blgp:1\n" \
    "s_xor_b32 m0, s76, 12288\n" \
    "s_xor_b32 s58, s77, 393216\n" \
    "buffer_load_dwordx4 v4, s[8:11], s58 offen offset:0 lds\n" \
    "ds_read_b64_tr_b8 a[116:117], v11 offset:8192\n" \
    "ds_read_b64_tr_b8 a[118:119], v11 offset:9216\n" \
    "v_mfma_f32_16x16x128_f8f6f4 v[196:199], a[64:71], a[152:159], v[196:199] cbsz:1 blgp:1\n" \
    "v_xor_b32 v11, 80, v12\n" \
    "ds_read_b64_tr_b8 a[120:121], v11\n" \
    "ds_read_b64_tr_b8 a[122:123], v11 offset:1024\n" \
    "v_mfma_f32_16x16x128_f8f6f4 v[200:203], a[64:71], a[160:167], v[200:203] cbsz:1 blgp:1\n" \
    "s_add_u32 s58, s58, 31744\n" \
    "buffer_load_dwordx4 v4, s[8:11], s58 offen offset:1024 lds\n" \
    "ds_read_b64_tr_b8 a[124:125], v11 offset:8192\n" \
    "ds_read_b64_tr_b8 a[126:127], v11 offset:9216\n" \
    "v_mfma_f32_16x16x128_f8f6f4 v[204:207], a[64:71], a[168:175], v[204:207] cbsz:1 blgp:1\n" \
    "v_xor_b32 v11, 96, v12\n" \
    "ds_read_b64_tr_b8 a[128:129], v11\n" \
    "ds_read_b64_tr_b8 a[130:131], v11 offset:1024\n" \
    "v_mfma_f32_16x16x128_f8f6f4 v[208:211], a[64:71], a[176:183], v[208:211] cbsz:1 blgp:1\n" \
    "s_add_u32 s58, s58, 31744\n" \
    "buffer_load_dwordx4 v5, s[8:11], s58 offen offset:2048 lds\n" \
    "ds_read_b64_tr_b8 a[132:133], v11 offset:8192\n" \
    "ds_read_b64_tr_b8 a[134:135], v11 offset:9216\n" \
    "v_mfma_f32_16x16x128_f8f6f4 v[212:215], a[64:71], a[184:191], v[212:215] cbsz:1 blgp:1\n" \
    "v_xor_b32 v11, 112, v12\n" \
    "ds_read_b64_tr_b8 a[136:137], v11\n" \
    "ds_read_b64_tr_b8 a[138:139], v11 offset:1024\n" \
    "v_mfma_f32_16x16x128_f8f6f4 v[216:219], a[64:71], a[192:199], v[216:219] cbsz:1 blgp:1\n" \
    "s_add_u32 s58, s58, 31744\n" \
    "buffer_load_dwordx4 v5, s[8:11], s58 offen offset:3072 lds\n" \
    "ds_read_b64_tr_b8 a[140:141], v11 offset:8192\n" \
    "ds_read_b64_tr_b8 a[142:143], v11 offset:9216\n" \
    "v_mfma_f32_16x16x128_f8f6f4 v[220:223], a[64:71], a[200:207], v[220:223] cbsz:1 blgp:1\n" \
    "s_mov_b32 s69, s70\n" \
    "s_mov_b32 s70, 7\n" \
    "s_xor_b32 s72, s72, 16384\n" \
    "s_add_u32 s73, s73, 1\n" \
    "s_waitcnt vmcnt(17) lgkmcnt(0)\n" \
    "a2a_c1:\n" \
      \
    "a2a_u0_top:\n" \
    "v_mfma_f32_16x16x128_f8f6f4 v[96:99], a[0:7], a[80:87], v[96:99] cbsz:1 blgp:1\n" \
    "v_cmp_le_u32 vcc, s59, v10\n" \
    "s_and_b32 s55, vcc_lo, 0x7f\n" \
    "s_or_b32 s66, s66, s55\n" \
    "s_lshr_b32 s55, s66, s70\n" \
    "v_mfma_f32_16x16x128_f8f6f4 v[100:103], a[0:7], a[88:95], v[100:103] cbsz:1 blgp:1\n" \
    "s_lshr_b32 s56, s66, s69\n" \
    "s_andn2_b32 s55, s55, s56\n" \
    "s_bitcmp1_b32 s55, 0\n" \
    "s_cselect_b32 s71, s70, s69\n" \
    "v_mfma_f32_16x16x128_f8f6f4 v[104:107], a[0:7], a[96:103], v[104:107] cbsz:1 blgp:1\n" \
    "s_cselect_b32 s70, s69, s70\n" \
    "s_cselect_b32 s57, 16384, 0\n" \
    "s_mov_b32 s69, s71\n" \
    "s_xor_b32 s72, s72, s57\n" \
    "s_cmp_lt_u32 s88, 7\n" \
    "s_cbranch_scc1 a2a_u0_pk\n" \
    "a2a_u0_pkr:\n" \
    "v_mfma_f32_16x16x128_f8f6f4 v[108:111], a[0:7], a[104:111], v[108:111] cbsz:1 blgp:1\n" \
    "s_lshl_b32 s63, s71, 2\n" \
    "s_lshr_b32 s63, s61, s63\n" \
    "s_and_b32 s63, s63, 7\n" \
    "s_lshl_b32 s63, s63, 18\n" \
    "s_add_u32 s63, s63, s62\n" \
    "s_add_u32 s64, s63, 4096\n" \
    "s_add_u32 s65, s63, 8192\n" \
    "s_bitcmp1_b32 s66, s71\n" \
    "s_cbranch_scc0 a2a_u0_absent\n" \
    "v_mfma_f32_16x16x128_f8f6f4 v[112:115], a[0:7], a[112:119], v[112:115] cbsz:1 blgp:1\n" \
    "buffer_load_dwordx4 a[40:43], v14, s[16:19], s63 offen offset:0 sc0 sc1\n" \
    "s_andn2_b32 s55, s67, s68\n" \
    "s_cselect_b32 s55, s55, s67\n" \
    "s_and_b32 s56, s67, s66\n" \
    "v_mfma_f32_16x16x128_f8f6f4 v[116:119], a[0:7], a[120:127], v[116:119] cbsz:1 blgp:1\n" \
    "buffer_load_dwordx4 a[44:47], v14, s[16:19], s63 offen offset:1024 sc0 sc1\n" \
    "s_cselect_b32 s55, s56, s55\n" \
    "s_or_b32 s55, s55, 0x80\n" \
    "s_ff1_i32_b32 s79, s55\n" \
    "v_mfma_f32_16x16x128_f8f6f4 v[120:123], a[0:7], a[128:135], v[120:123] cbsz:1 blgp:1\n" \
    "buffer_load_dwordx4 a[48:51], v14, s[16:19], s63 offen offset:2048 sc0 sc1\n" \
    "s_lshl_b32 s77, s79, 2\n" \
    "s_lshr_b32 s77, s61, s77\n" \
    "s_and_b32 s77, s77, 7\n" \
    "v_mfma_f32_16x16x128_f8f6f4 v[124:127], a[0:7], a[136:143], v[124:127] cbsz:1 blgp:1\n" \
    "buffer_load_dwordx4 a[52:55], v14, s[16:19], s63 offen offset:3072 sc0 sc1\n" \
    "s_lshl_b32 s77, s77, 21\n" \
    "s_add_u32 s77, s77, s53\n" \
    "s_cmp_eq_u32 s66, 0x7f\n" \
    "v_mfma_f32_16x16x128_f8f6f4 v[128:131], a[8:15], a[80:87], v[128:131] cbsz:1 blgp:1\n" \
    "buffer_load_dwordx4 a[56:59], v14, s[16:19], s64 offen offset:0 sc0 sc1\n" \
    "s_cselect_b32 s55, 0, -1\n" \
    "v_and_b32 v15, s55, v15\n" \
    "s_mov_b32 s74, 1\n" \
    "v_mfma_f32_16x16x128_f8f6f4 v[132:135], a[8:15], a[88:95], v[132:135] cbsz:1 blgp:1\n" \
    "buffer_load_dwordx4 a[60:63], v14, s[16:19], s64 offen offset:1024 sc0 sc1\n" \
    "s_xor_b32 s76, s72, s54\n" \
    "v_add_u32 v12, s72, v13\n" \
    "s_cmp_eq_u32 s70, 7\n" \
    "v_mfma_f32_16x16x128_f8f6f4 v[136:139], a[8:15], a[96:103], v[136:139] cbsz:1 blgp:1\n" \
    "buffer_load_dwordx4 a[64:67], v14, s[16:19], s64 offen offset:2048 sc0 sc1\n" \
    "s_cselect_b32 s55, s67, 0\n" \
    "s_cmp_eq_u32 s55, 0\n" \
    "v_mfma_f32_16x16x128_f8f6f4 v[140:143], a[8:15], a[104:111], v[140:143] cbsz:1 blgp:1\n" \
    "buffer_load_dwordx4 a[68:71], v14, s[16:19], s64 offen offset:3072 sc0 sc1\n" \
    "s_cbranch_scc1 a2a_u0_nolw\n" \
    "s_bitset0_b32 s67, s79\n" \
    "s_mov_b32 s70, s79\n" \
    "s_cmp_eq_u32 s80, 0\n" \
    "s_cbranch_scc1 a2a_u0_alt4\n" \
    "v_mfma_f32_16x16x128_f8f6f4 v[224:227], a[32:39], a[80:87], v[224:227] cbsz:1 blgp:1\n" \
    "buffer_load_dwordx4 a[72:75], v14, s[16:19], s65 offen offset:0 sc0 sc1\n" \
    "v_mfma_f32_16x16x128_f8f6f4 v[228:231], a[32:39], a[88:95], v[228:231] cbsz:1 blgp:1\n" \
    "buffer_load_dwordx4 a[76:79], v14, s[16:19], s65 offen offset:1024 sc0 sc1\n" \
    "v_mfma_f32_16x16x128_f8f6f4 v[232:235], a[32:39], a[96:103], v[232:235] cbsz:1 blgp:1\n" \
    "s_xor_b32 m0, s76, 0\n" \
    "s_xor_b32 s58, s77, 0\n" \
    "buffer_load_dwordx4 v4, s[8:11], s58 offen offset:0 lds\n" \
    "v_mfma_f32_16x16x128_f8f6f4 v[236:239], a[32:39], a[104:111], v[236:239] cbsz:1 blgp:1\n" \
    "s_add_u32 s58, s58, 31744\n" \
    "buffer_load_dwordx4 v4, s[8:11], s58 offen offset:1024 lds\n" \
    "v_mfma_f32_16x16x128_f8f6f4 v[240:243], a[32:39], a[112:119], v[240:243] cbsz:1 blgp:1\n" \
    "s_add_u32 s58, s58, 31744\n" \
    "buffer_load_dwordx4 v5, s[8:11], s58 offen offset:2048 lds\n" \
    "v_mfma_f32_16x16x128_f8f6f4 v[244:247], a[32:39], a[120:127], v[244:247] cbsz:1 blgp:1\n" \
    "s_add_u32 s58, s58, 31744\n" \
    "buffer_load_dwordx4 v5, s[8:11], s58 offen offset:3072 lds\n" \
    "v_mfma_f32_16x16x128_f8f6f4 v[248:251], a[32:39], a[128:135], v[248:251] cbsz:1 blgp:1\n" \
    "s_xor_b32 m0, s76, 4096\n" \
    "s_xor_b32 s58, s77, 131072\n" \
    "buffer_load_dwordx4 v4, s[8:11], s58 offen offset:0 lds\n" \
    "v_mfma_f32_16x16x128_f8f6f4 v[252:255], a[32:39], a[136:143], v[252:255] cbsz:1 blgp:1\n" \
    "s_add_u32 s58, s58, 31744\n" \
    "buffer_load_dwordx4 v4, s[8:11], s58 offen offset:1024 lds\n" \
    "a2a_u0_a5:\n" \
    "v_mfma_f32_16x16x128_f8f6f4 v[144:147], a[8:15], a[112:119], v[144:147] cbsz:1 blgp:1\n" \
    "s_add_u32 s58, s58, 31744\n" \
    "buffer_load_dwordx4 v5, s[8:11], s58 offen offset:2048 lds\n" \
    "v_mfma_f32_16x16x128_f8f6f4 v[148:151], a[8:15], a[120:127], v[148:151] cbsz:1 blgp:1\n" \
    "s_add_u32 s58, s58, 31744\n" \
    "buffer_load_dwordx4 v5, s[8:11], s58 offen offset:3072 lds\n" \
    "v_mfma_f32_16x16x128_f8f6f4 v[152:155], a[8:15], a[128:135], v[152:155] cbsz:1 blgp:1\n" \
    "global_load_dword v10, v15, s[38:39] sc0 sc1\n" \
    "s_cmp_lg_u32 s89, 0\n" \
    "s_cbranch_scc1 a2a_u0_h2p\n" \
    "s_waitcnt vmcnt(17)\n" \
    "v_mfma_f32_16x16x128_f8f6f4 v[156:159], a[8:15], a[136:143], v[156:159] cbsz:1 blgp:1\n" \
    "v_xor_b32 v11, 0, v12\n" \
    "ds_read_b64_tr_b8 a[144:145], v11\n" \
    "ds_read_b64_tr_b8 a[146:147], v11 offset:1024\n" \
    "v_mfma_f32_16x16x128_f8f6f4 v[160:163], a[16:23], a[80:87], v[160:163] cbsz:1 blgp:1\n" \
    "s_xor_b32 m0, s76, 8192\n" \
    "s_xor_b32 s58, s77, 262144\n" \
    "buffer_load_dwordx4 v4, s[8:11], s58 offen offset:0 lds\n" \
    "ds_read_b64_tr_b8 a[148:149], v11 offset:8192\n" \
    "ds_read_b64_tr_b8 a[150:151], v11 offset:9216\n" \
    "v_mfma_f32_16x16x128_f8f6f4 v[164:167], a[16:23], a[88:95], v[164:167] cbsz:1 blgp:1\n" \
    "v_xor_b32 v11, 16, v12\n" \
    "ds_read_b64_tr_b8 a[152:153], v11\n" \
    "ds_read_b64_tr_b8 a[154:155], v11 offset:1024\n" \
    "v_mfma_f32_16x16x128_f8f6f4 v[168:171], a[16:23], a[96:103], v[168:171] cbsz:1 blgp:1\n" \
    "s_add_u32 s58, s58, 31744\n" \
    "buffer_load_dwordx4 v4, s[8:11], s58 offen offset:1024 lds\n" \
    "ds_read_b64_tr_b8 a[156:157], v11 offset:8192\n" \
    "ds_read_b64_tr_b8 a[158:159], v11 offset:9216\n" \
    "v_mfma_f32_16x16x128_f8f6f4 v[172:175], a[16:23], a[104:111], v[172:175] cbsz:1 blgp:1\n" \
    "v_xor_b32 v11, 32, v12\n" \
    "ds_read_b64_tr_b8 a[160:161], v11\n" \
    "ds_read_b64_tr_b8 a[162:163], v11 offset:1024\n" \
    "v_mfma_f32_16x16x128_f8f6f4 v[176:179], a[16:23], a[112:119], v[176:179] cbsz:1 blgp:1\n" \
    "s_add_u32 s58, s58, 31744\n" \
    "buffer_load_dwordx4 v5, s[8:11], s58 offen offset:2048 lds\n" \
    "ds_read_b64_tr_b8 a[164:165], v11 offset:8192\n" \
    "ds_read_b64_tr_b8 a[166:167], v11 offset:9216\n" \
    "v_mfma_f32_16x16x128_f8f6f4 v[180:183], a[16:23], a[120:127], v[180:183] cbsz:1 blgp:1\n" \
    "v_xor_b32 v11, 48, v12\n" \
    "ds_read_b64_tr_b8 a[168:169], v11\n" \
    "ds_read_b64_tr_b8 a[170:171], v11 offset:1024\n" \
    "v_mfma_f32_16x16x128_f8f6f4 v[184:187], a[16:23], a[128:135], v[184:187] cbsz:1 blgp:1\n" \
    "s_add_u32 s58, s58, 31744\n" \
    "buffer_load_dwordx4 v5, s[8:11], s58 offen offset:3072 lds\n" \
    "ds_read_b64_tr_b8 a[172:173], v11 offset:8192\n" \
    "ds_read_b64_tr_b8 a[174:175], v11 offset:9216\n" \
    "v_mfma_f32_16x16x128_f8f6f4 v[188:191], a[16:23], a[136:143], v[188:191] cbsz:1 blgp:1\n" \
    "v_xor_b32 v11, 64, v12\n" \
    "ds_read_b64_tr_b8 a[176:177], v11\n" \
    "ds_read_b64_tr_b8 a[178:179], v11 offset:1024\n" \
    "v_mfma_f32_16x16x128_f8f6f4 v[192:195], a[24:31], a[80:87], v[192:195] cbsz:1 blgp:1\n" \
    "s_xor_b32 m0, s76, 12288\n" \
    "s_xor_b32 s58, s77, 393216\n" \
    "buffer_load_dwordx4 v4, s[8:11], s58 offen offset:0 lds\n" \
    "ds_read_b64_tr_b8 a[180:181], v11 offset:8192\n" \
    "ds_read_b64_tr_b8 a[182:183], v11 offset:9216\n" \
    "v_mfma_f32_16x16x128_f8f6f4 v[196:199], a[24:31], a[88:95], v[196:199] cbsz:1 blgp:1\n" \
    "v_xor_b32 v11, 80, v12\n" \
    "ds_read_b64_tr_b8 a[184:185], v11\n" \
    "ds_read_b64_tr_b8 a[186:187], v11 offset:1024\n" \
    "v_mfma_f32_16x16x128_f8f6f4 v[200:203], a[24:31], a[96:103], v[200:203] cbsz:1 blgp:1\n" \
    "s_add_u32 s58, s58, 31744\n" \
    "buffer_load_dwordx4 v4, s[8:11], s58 offen offset:1024 lds\n" \
    "ds_read_b64_tr_b8 a[188:189], v11 offset:8192\n" \
    "ds_read_b64_tr_b8 a[190:191], v11 offset:9216\n" \
    "v_mfma_f32_16x16x128_f8f6f4 v[204:207], a[24:31], a[104:111], v[204:207] cbsz:1 blgp:1\n" \
    "v_xor_b32 v11, 96, v12\n" \
    "ds_read_b64_tr_b8 a[192:193], v11\n" \
    "ds_read_b64_tr_b8 a[194:195], v11 offset:1024\n" \
    "v_mfma_f32_16x16x128_f8f6f4 v[208:211], a[24:31], a[112:119], v[208:211] cbsz:1 blgp:1\n" \
    "s_add_u32 s58, s58, 31744\n" \
    "buffer_load_dwordx4 v5, s[8:11], s58 offen offset:2048 lds\n" \
    "ds_read_b64_tr_b8 a[196:197], v11 offset:8192\n" \
    "ds_read_b64_tr_b8 a[198:199], v11 offset:9216\n" \
    "v_mfma_f32_16x16x128_f8f6f4 v[212:215], a[24:31], a[120:127], v[212:215] cbsz:1 blgp:1\n" \
    "v_xor_b32 v11, 112, v12\n" \
    "ds_read_b64_tr_b8 a[200:201], v11\n" \
    "ds_read_b64_tr_b8 a[202:203], v11 offset:1024\n" \
    "v_mfma_f32_16x16x128_f8f6f4 v[216:219], a[24:31], a[128:135], v[216:219] cbsz:1 blgp:1\n" \
    "s_add_u32 s58, s58, 31744\n" \
    "buffer_load_dwordx4 v5, s[8:11], s58 offen offset:3072 lds\n" \
    "ds_read_b64_tr_b8 a[204:205], v11 offset:8192\n" \
    "ds_read_b64_tr_b8 a[206:207], v11 offset:9216\n" \
    "v_mfma_f32_16x16x128_f8f6f4 v[220:223], a[24:31], a[136:143], v[220:223] cbsz:1 blgp:1\n" \
    "s_mov_b32 s69, s70\n" \
    "s_mov_b32 s70, 7\n" \
    "s_xor_b32 s72, s72, 16384\n" \
    "s_add_u32 s73, s73, 1\n" \
    "s_waitcnt vmcnt(17) lgkmcnt(0)\n" \
      \
    "a2a_c0:\n" \
    "s_cmp_lt_u32 s73, 6\n" \
    "s_cbranch_scc1 a2a_u1_top\n" \
    "v_and_b32 v16, 4, v1\n" \
    "v_lshrrev_b32 v17, 2, v2\n" \
    "v_add_u32 v21, v16, v17\n" \
    "v_lshlrev_b32 v21, 8, v21\n" \
    "s_waitcnt vmcnt(6)\n" \
    "v_mfma_f32_16x16x128_f8f6f4 v[96:99], a[40:47], a[144:151], v[96:99] cbsz:1 blgp:1\n" \
    "v_and_b32 v16, 3, v2\n" \
    "v_lshl_or_b32 v21, v16, 6, v21\n" \
    "v_lshl_add_u32 v24, v16, 2, v17\n" \
    "v_lshlrev_b32 v24, 14, v24\n" \
    "v_and_b32 v16, 3, v1\n" \
    "v_mfma_f32_16x16x128_f8f6f4 v[100:103], a[40:47], a[152:159], v[100:103] cbsz:1 blgp:1\n" \
    "v_lshl_or_b32 v21, v16, 4, v21\n" \
    "v_and_b32 v16, 7, v1\n" \
    "v_lshl_or_b32 v24, v16, 4, v24\n" \
    "s_lshl_b32 s55, s44, 7\n" \
    "v_add_u32 v24, s55, v24\n" \
    "v_mfma_f32_16x16x128_f8f6f4 v[104:107], a[40:47], a[160:167], v[104:107] cbsz:1 blgp:1\n" \
    "s_lshl_b32 s55, s44, 11\n" \
    "s_add_u32 s55, s55, %[lds]\n" \
    "s_add_u32 s55, s55, 131072\n" \
    "v_add_u32 v23, s55, v21\n" \
    "s_lshl_b32 s55, s44, 13\n" \
    "v_mfma_f32_16x16x128_f8f6f4 v[108:111], a[40:47], a[168:175], v[108:111] cbsz:1 blgp:1\n" \
    "s_add_u32 s55, s55, %[lds]\n" \
    "v_add_u32 v21, s55, v21\n" \
    "v_add_u32 v22, 0x10000, v21\n" \
    "s_mov_b32 m0, s51\n" \
    "s_nop 0\n" \
    "ds_write_addtid_b32 v96 offset:0\n" \
    "ds_write_addtid_b32 v97 offset:256\n" \
    "ds_write_addtid_b32 v98 offset:512\n" \
    "ds_write_addtid_b32 v99 offset:768\n" \
    "v_mfma_f32_16x16x128_f8f6f4 v[112:115], a[40:47], a[176:183], v[112:115] cbsz:1 blgp:1\n" \
    "ds_write_addtid_b32 v100 offset:1024\n" \
    "ds_write_addtid_b32 v101 offset:1280\n" \
    "ds_write_addtid_b32 v102 offset:1536\n" \
    "ds_write_addtid_b32 v103 offset:1792\n" \
    "v_mfma_f32_16x16x128_f8f6f4 v[116:119], a[40:47], a[184:191], v[116:119] cbsz:1 blgp:1\n" \
    "ds_write_addtid_b32 v104 offset:8192\n" \
    "ds_write_addtid_b32 v105 offset:8448\n" \
    "ds_write_addtid_b32 v106 offset:8704\n" \
    "ds_write_addtid_b32 v107 offset:8960\n" \
    "v_mfma_f32_16x16x128_f8f6f4 v[120:123], a[40:47], a[192:199], v[120:123] cbsz:1 blgp:1\n" \
    "ds_write_addtid_b32 v108 offset:9216\n" \
    "ds_write_addtid_b32 v109 offset:9472\n" \
    "ds_write_addtid_b32 v110 offset:9728\n" \
    "ds_write_addtid_b32 v111 offset:9984\n" \
    "v_mfma_f32_16x16x128_f8f6f4 v[124:127], a[40:47], a[200:207], v[124:127] cbsz:1 blgp:1\n" \
    "ds_write_addtid_b32 v112 offset:16384\n" \
    "ds_write_addtid_b32 v113 offset:16640\n" \
    "ds_write_addtid_b32 v114 offset:16896\n" \
    "ds_write_addtid_b32 v115 offset:17152\n" \
    "s_waitcnt vmcnt(4)\n" \
    "v_mfma_f32_16x16x128_f8f6f4 v[128:131], a[48:55], a[144:151], v[128:131] cbsz:1 blgp:1\n" \
    "ds_write_addtid_b32 v116 offset:17408\n" \
    "ds_write_addtid_b32 v117 offset:17664\n" \
    "ds_write_addtid_b32 v118 offset:17920\n" \
    "ds_write_addtid_b32 v119 offset:18176\n" \
    "v_mfma_f32_16x16x128_f8f6f4 v[132:135], a[48:55], a[152:159], v[132:135] cbsz:1 blgp:1\n" \
    "ds_write_addtid_b32 v120 offset:24576\n" \
    "ds_write_addtid_b32 v121 offset:24832\n" \
    "ds_write_addtid_b32 v122 offset:25088\n" \
    "ds_write_addtid_b32 v123 offset:25344\n" \
    "v_mfma_f32_16x16x128_f8f6f4 v[136:139], a[48:55], a[160:167], v[136:139] cbsz:1 blgp:1\n" \
    "ds_write_addtid_b32 v124 offset:25600\n" \
    "ds_write_addtid_b32 v125 offset:25856\n" \
    "ds_write_addtid_b32 v126 offset:26112\n" \
    "ds_write_addtid_b32 v127 offset:26368\n" \
    "v_mfma_f32_16x16x128_f8f6f4 v[140:143], a[48:55], a[168:175], v[140:143] cbsz:1 blgp:1\n" \
    "ds_write_addtid_b32 v128 offset:2048\n" \
    "ds_write_addtid_b32 v129 offset:2304\n" \
    "ds_write_addtid_b32 v130 offset:2560\n" \
    "ds_write_addtid_b32 v131 offset:2816\n" \
    "v_mfma_f32_16x16x128_f8f6f4 v[144:147], a[48:55], a[176:183], v[144:147] cbsz:1 blgp:1\n" \
    "ds_write_addtid_b32 v132 offset:3072\n" \
    "ds_write_addtid_b32 v133 offset:3328\n" \
    "ds_write_addtid_b32 v134 offset:3584\n" \
    "ds_write_addtid_b32 v135 offset:3840\n" \
    "v_mfma_f32_16x16x128_f8f6f4 v[148:151], a[48:55], a[184:191], v[148:151] cbsz:1 blgp:1\n" \
    "ds_write_addtid_b32 v136 offset:10240\n" \
    "ds_write_addtid_b32 v137 offset:10496\n" \
    "ds_write_addtid_b32 v138 offset:10752\n" \
    "ds_write_addtid_b32 v139 offset:11008\n" \
    "v_mfma_f32_16x16x128_f8f6f4 v[152:155], a[48:55], a[192:199], v[152:155] cbsz:1 blgp:1\n" \
    "ds_write_addtid_b32 v140 offset:11264\n" \
    "ds_write_addtid_b32 v141 offset:11520\n" \
    "ds_write_addtid_b32 v142 offset:11776\n" \
    "ds_write_addtid_b32 v143 offset:12032\n" \
    "v_mfma_f32_16x16x128_f8f6f4 v[156:159], a[48:55], a[200:207], v[156:159] cbsz:1 blgp:1\n" \
    "ds_write_addtid_b32 v144 offset:18432\n" \
    "ds_write_addtid_b32 v145 offset:18688\n" \
    "ds_write_addtid_b32 v146 offset:18944\n" \
    "ds_write_addtid_b32 v147 offset:19200\n" \
    "s_waitcnt vmcnt(2)\n" \
    "v_mfma_f32_16x16x128_f8f6f4 v[160:163], a[56:63], a[144:151], v[160:163] cbsz:1 blgp:1\n" \
    "ds_write_addtid_b32 v148 offset:19456\n" \
    "ds_write_addtid_b32 v149 offset:19712\n" \
    "ds_write_addtid_b32 v150 offset:19968\n" \
    "ds_write_addtid_b32 v151 offset:20224\n" \
    "v_mfma_f32_16x16x128_f8f6f4 v[164:167], a[56:63], a[152:159], v[164:167] cbsz:1 blgp:1\n" \
    "ds_write_addtid_b32 v152 offset:26624\n" \
    "ds_write_addtid_b32 v153 offset:26880\n" \
    "ds_write_addtid_b32 v154 offset:27136\n" \
    "ds_write_addtid_b32 v155 offset:27392\n" \
    "v_mfma_f32_16x16x128_f8f6f4 v[168:171], a[56:63], a[160:167], v[168:171] cbsz:1 blgp:1\n" \
    "ds_write_addtid_b32 v156 offset:27648\n" \
    "ds_write_addtid_b32 v157 offset:27904\n" \
    "ds_write_addtid_b32 v158 offset:28160\n" \
    "ds_write_addtid_b32 v159 offset:28416\n" \
    "v_mfma_f32_16x16x128_f8f6f4 v[172:175], a[56:63], a[168:175], v[172:175] cbsz:1 blgp:1\n" \
    "ds_write_addtid_b32 v160 offset:4096\n" \
    "ds_write_addtid_b32 v161 offset:4352\n" \
    "ds_write_addtid_b32 v162 offset:4608\n" \
    "ds_write_addtid_b32 v163 offset:4864\n" \
    "v_mfma_f32_16x16x128_f8f6f4 v[176:179], a[56:63], a[176:183], v[176:179] cbsz:1 blgp:1\n" \
    "ds_write_addtid_b32 v164 offset:5120\n" \
    "ds_write_addtid_b32 v165 offset:5376\n" \
    "ds_write_addtid_b32 v166 offset:5632\n" \
    "ds_write_addtid_b32 v167 offset:5888\n" \
    "v_mfma_f32_16x16x128_f8f6f4 v[180:183], a[56:63], a[184:191], v[180:183] cbsz:1 blgp:1\n" \
    "ds_write_addtid_b32 v168 offset:12288\n" \
    "ds_write_addtid_b32 v169 offset:12544\n" \
    "ds_write_addtid_b32 v170 offset:12800\n" \
    "ds_write_addtid_b32 v171 offset:13056\n" \
    "v_mfma_f32_16x16x128_f8f6f4 v[184:187], a[56:63], a[192:199], v[184:187] cbsz:1 blgp:1\n" \
    "ds_write_addtid_b32 v172 offset:13312\n" \
    "ds_write_addtid_b32 v173 offset:13568\n" \
    "ds_write_addtid_b32 v174 offset:13824\n" \
    "ds_write_addtid_b32 v175 offset:14080\n" \
    "v_mfma_f32_16x16x128_f8f6f4 v[188:191], a[56:63], a[200:207], v[188:191] cbsz:1 blgp:1\n" \
    "ds_write_addtid_b32 v176 offset:20480\n" \
    "ds_write_addtid_b32 v177 offset:20736\n" \
    "ds_write_addtid_b32 v178 offset:20992\n" \
    "ds_write_addtid_b32 v179 offset:21248\n" \
    "s_waitcnt vmcnt(0)\n" \
    "v_mfma_f32_16x16x128_f8f6f4 v[192:195], a[64:71], a[144:151], v[192:195] cbsz:1 blgp:1\n" \
    "ds_write_addtid_b32 v180 offset:21504\n" \
    "ds_write_addtid_b32 v181 offset:21760\n" \
    "ds_write_addtid_b32 v182 offset:22016\n" \
    "ds_write_addtid_b32 v183 offset:22272\n" \
    "v_mfma_f32_16x16x128_f8f6f4 v[196:199], a[64:71], a[152:159], v[196:199] cbsz:1 blgp:1\n" \
    "ds_write_addtid_b32 v184 offset:28672\n" \
    "ds_write_addtid_b32 v185 offset:28928\n" \
    "ds_write_addtid_b32 v186 offset:29184\n" \
    "ds_write_addtid_b32 v187 offset:29440\n" \
    "v_mfma_f32_16x16x128_f8f6f4 v[200:203], a[64:71], a[160:167], v[200:203] cbsz:1 blgp:1\n" \
    "ds_write_addtid_b32 v188 offset:29696\n" \
    "ds_write_addtid_b32 v189 offset:29952\n" \
    "ds_write_addtid_b32 v190 offset:30208\n" \
    "ds_write_addtid_b32 v191 offset:30464\n" \
    "v_mfma_f32_16x16x128_f8f6f4 v[204:207], a[64:71], a[168:175], v[204:207] cbsz:1 blgp:1\n" \
    "ds_write_addtid_b32 v192 offset:6144\n" \
    "ds_write_addtid_b32 v193 offset:6400\n" \
    "ds_write_addtid_b32 v194 offset:6656\n" \
    "ds_write_addtid_b32 v195 offset:6912\n" \
    "v_mfma_f32_16x16x128_f8f6f4 v[208:211], a[64:71], a[176:183], v[208:211] cbsz:1 blgp:1\n" \
    "ds_write_addtid_b32 v196 offset:7168\n" \
    "ds_write_addtid_b32 v197 offset:7424\n" \
    "ds_write_addtid_b32 v198 offset:7680\n" \
    "ds_write_addtid_b32 v199 offset:7936\n" \
    "v_mfma_f32_16x16x128_f8f6f4 v[212:215], a[64:71], a[184:191], v[212:215] cbsz:1 blgp:1\n" \
    "ds_write_addtid_b32 v200 offset:14336\n" \
    "ds_write_addtid_b32 v201 offset:14592\n" \
    "ds_write_addtid_b32 v202 offset:14848\n" \
    "ds_write_addtid_b32 v203 offset:15104\n" \
    "v_mfma_f32_16x16x128_f8f6f4 v[216:219], a[64:71], a[192:199], v[216:219] cbsz:1 blgp:1\n" \
    "ds_write_addtid_b32 v204 offset:15360\n" \
    "ds_write_addtid_b32 v205 offset:15616\n" \
    "ds_write_addtid_b32 v206 offset:15872\n" \
    "ds_write_addtid_b32 v207 offset:16128\n" \
    "v_mfma_f32_16x16x128_f8f6f4 v[220:223], a[64:71], a[200:207], v[220:223] cbsz:1 blgp:1\n" \
    "ds_write_addtid_b32 v208 offset:22528\n" \
    "ds_write_addtid_b32 v209 offset:22784\n" \
    "ds_write_addtid_b32 v210 offset:23040\n" \
    "ds_write_addtid_b32 v211 offset:23296\n" \
    "s_cmp_eq_u32 s80, 0\n" \
    "s_cbranch_scc1 a2a_t4\n" \
    "v_mfma_f32_16x16x128_f8f6f4 v[224:227], a[72:79], a[144:151], v[224:227] cbsz:1 blgp:1\n" \
    "ds_write_addtid_b32 v212 offset:23552\n" \
    "ds_write_addtid_b32 v213 offset:23808\n" \
    "ds_write_addtid_b32 v214 offset:24064\n" \
    "ds_write_addtid_b32 v215 offset:24320\n" \
    "v_mfma_f32_16x16x128_f8f6f4 v[228:231], a[72:79], a[152:159], v[228:231] cbsz:1 blgp:1\n" \
    "ds_write_addtid_b32 v216 offset:30720\n" \
    "ds_write_addtid_b32 v217 offset:30976\n" \
    "ds_write_addtid_b32 v218 offset:31232\n" \
    "ds_write_addtid_b32 v219 offset:31488\n" \
    "v_mfma_f32_16x16x128_f8f6f4 v[232:235], a[72:79], a[160:167], v[232:235] cbsz:1 blgp:1\n" \
    "ds_write_addtid_b32 v220 offset:31744\n" \
    "ds_write_addtid_b32 v221 offset:32000\n" \
    "ds_write_addtid_b32 v222 offset:32256\n" \
    "ds_write_addtid_b32 v223 offset:32512\n" \
    "v_mfma_f32_16x16x128_f8f6f4 v[236:239], a[72:79], a[168:175], v[236:239] cbsz:1 blgp:1\n" \
    "s_mov_b32 m0, s52\n" \
    "s_nop 0\n" \
    "ds_write_addtid_b32 v224 offset:0\n" \
    "ds_write_addtid_b32 v225 offset:256\n" \
    "ds_write_addtid_b32 v226 offset:512\n" \
    "ds_write_addtid_b32 v227 offset:768\n" \
    "v_mfma_f32_16x16x128_f8f6f4 v[240:243], a[72:79], a[176:183], v[240:243] cbsz:1 blgp:1\n" \
    "ds_write_addtid_b32 v228 offset:1024\n" \
    "ds_write_addtid_b32 v229 offset:1280\n" \
    "ds_write_addtid_b32 v230 offset:1536\n" \
    "ds_write_addtid_b32 v231 offset:1792\n" \
    "v_mfma_f32_16x16x128_f8f6f4 v[244:247], a[72:79], a[184:191], v[244:247] cbsz:1 blgp:1\n" \
    "ds_write_addtid_b32 v232 offset:2048\n" \
    "ds_write_addtid_b32 v233 offset:2304\n" \
    "ds_write_addtid_b32 v234 offset:2560\n" \
    "ds_write_addtid_b32 v235 offset:2816\n" \
    "v_mfma_f32_16x16x128_f8f6f4 v[248:251], a[72:79], a[192:199], v[248:251] cbsz:1 blgp:1\n" \
    "ds_write_addtid_b32 v236 offset:3072\n" \
    "ds_write_addtid_b32 v237 offset:3328\n" \
    "ds_write_addtid_b32 v238 offset:3584\n" \
    "ds_write_addtid_b32 v239 offset:3840\n" \
    "v_mfma_f32_16x16x128_f8f6f4 v[252:255], a[72:79], a[200:207], v[252:255] cbsz:1 blgp:1\n" \
    "ds_write_addtid_b32 v240 offset:4096\n" \
    "ds_write_addtid_b32 v241 offset:4352\n" \
    "ds_write_addtid_b32 v242 offset:4608\n" \
    "ds_write_addtid_b32 v243 offset:4864\n" \
    "ds_write_addtid_b32 v244 offset:5120\n" \
    "ds_write_addtid_b32 v245 offset:5376\n" \
    "ds_write_addtid_b32 v246 offset:5632\n" \
    "ds_write_addtid_b32 v247 offset:5888\n" \
    "ds_write_addtid_b32 v248 offset:6144\n" \
    "ds_write_addtid_b32 v249 offset:6400\n" \
    "ds_write_addtid_b32 v250 offset:6656\n" \
    "ds_write_addtid_b32 v251 offset:6912\n" \
    "ds_write_addtid_b32 v252 offset:7168\n" \
    "ds_write_addtid_b32 v253 offset:7424\n" \
    "ds_write_addtid_b32 v254 offset:7680\n" \
    "ds_write_addtid_b32 v255 offset:7936\n" \
    "s_branch a2a_tb\n" \
    "a2a_t4:\n" \
    "ds_write_addtid_b32 v212 offset:23552\n" \
    "ds_write_addtid_b32 v213 offset:23808\n" \
    "ds_write_addtid_b32 v214 offset:24064\n" \
    "ds_write_addtid_b32 v215 offset:24320\n" \
    "ds_write_addtid_b32 v216 offset:30720\n" \
    "ds_write_addtid_b32 v217 offset:30976\n" \
    "ds_write_addtid_b32 v218 offset:31232\n" \
    "ds_write_addtid_b32 v219 offset:31488\n" \
    "ds_write_addtid_b32 v220 offset:31744\n" \
    "ds_write_addtid_b32 v221 offset:32000\n" \
    "ds_write_addtid_b32 v222 offset:32256\n" \
    "ds_write_addtid_b32 v223 offset:32512\n" \
    "a2a_tb:\n" \
    "s_waitcnt lgkmcnt(0)\n" \
    "s_barrier\n" \
    "ds_read_b128 v[96:99], v21 offset:0\n" \
    "ds_read_b128 v[112:115], v22 offset:0\n" \
    "ds_read_b128 v[100:103], v21 offset:512\n" \
    "ds_read_b128 v[116:119], v22 offset:512\n" \
    "ds_read_b128 v[104:107], v21 offset:32768\n" \
    "ds_read_b128 v[120:123], v22 offset:32768\n" \
    "ds_read_b128 v[108:111], v21 offset:33280\n" \
    "ds_read_b128 v[124:127], v22 offset:33280\n" \
    "s_waitcnt lgkmcnt(0)\n" \
    "ds_read_b128 v[128:131], v21 offset:2048\n" \
    "ds_read_b128 v[144:147], v22 offset:2048\n" \
    "ds_read_b128 v[132:135], v21 offset:2560\n" \
    "ds_read_b128 v[148:151], v22 offset:2560\n" \
    "ds_read_b128 v[136:139], v21 offset:34816\n" \
    "ds_read_b128 v[152:155], v22 offset:34816\n" \
    "ds_read_b128 v[140:143], v21 offset:35328\n" \
    "ds_read_b128 v[156:159], v22 offset:35328\n" \
    "v_pk_add_f32 v[96:97], v[96:97], v[112:113]\n" \
    "v_pk_add_f32 v[98:99], v[98:99], v[114:115]\n" \
    "v_pk_add_f32 v[100:101], v[100:101], v[116:117]\n" \
    "v_pk_add_f32 v[102:103], v[102:103], v[118:119]\n" \
    "v_pk_add_f32 v[104:105], v[104:105], v[120:121]\n" \
    "v_pk_add_f32 v[106:107], v[106:107], v[122:123]\n" \
    "v_pk_add_f32 v[108:109], v[108:109], v[124:125]\n" \
    "v_pk_add_f32 v[110:111], v[110:111], v[126:127]\n" \
    "v_pk_add_f32 v[96:97], v[96:97], v[104:105]\n" \
    "v_pk_add_f32 v[98:99], v[98:99], v[106:107]\n" \
    "v_pk_add_f32 v[100:101], v[100:101], v[108:109]\n" \
    "v_pk_add_f32 v[102:103], v[102:103], v[110:111]\n" \
    "v_pk_mul_f32 v[104:105], v[96:97], v[30:31]\n" \
    "v_pk_mul_f32 v[106:107], v[98:99], v[30:31]\n" \
    "v_pk_mul_f32 v[108:109], v[100:101], v[30:31]\n" \
    "v_pk_mul_f32 v[110:111], v[102:103], v[30:31]\n" \
    "v_med3_f32 v104, v104, -v25, v25\n" \
    "v_med3_f32 v105, v105, -v25, v25\n" \
    "v_med3_f32 v106, v106, -v25, v25\n" \
    "v_med3_f32 v107, v107, -v25, v25\n" \
    "v_med3_f32 v108, v108, -v25, v25\n" \
    "v_med3_f32 v109, v109, -v25, v25\n" \
    "v_med3_f32 v110, v110, -v25, v25\n" \
    "v_med3_f32 v111, v111, -v25, v25\n" \
    "v_exp_f32 v104, v104\n" \
    "v_exp_f32 v105, v105\n" \
    "v_exp_f32 v106, v106\n" \
    "v_exp_f32 v107, v107\n" \
    "v_exp_f32 v108, v108\n" \
    "v_exp_f32 v109, v109\n" \
    "v_exp_f32 v110, v110\n" \
    "v_exp_f32 v111, v111\n" \
    "v_pk_mul_f32 v[96:97], v[96:97], v[28:29]\n" \
    "v_pk_mul_f32 v[98:99], v[98:99], v[28:29]\n" \
    "v_pk_mul_f32 v[100:101], v[100:101], v[28:29]\n" \
    "v_pk_mul_f32 v[102:103], v[102:103], v[28:29]\n" \
    "v_pk_add_f32 v[104:105], v[104:105], 1.0 op_sel_hi:[1,0]\n" \
    "v_pk_add_f32 v[106:107], v[106:107], 1.0 op_sel_hi:[1,0]\n" \
    "v_pk_add_f32 v[108:109], v[108:109], 1.0 op_sel_hi:[1,0]\n" \
    "v_pk_add_f32 v[110:111], v[110:111], 1.0 op_sel_hi:[1,0]\n" \
    "v_rcp_f32 v104, v104\n" \
    "v_rcp_f32 v105, v105\n" \
    "v_rcp_f32 v106, v106\n" \
    "v_rcp_f32 v107, v107\n" \
    "v_rcp_f32 v108, v108\n" \
    "v_pk_mul_f32 v[96:97], v[96:97], v[104:105]\n" \
    "v_pk_mul_f32 v[98:99], v[98:99], v[106:107]\n" \
    "global_store_dwordx4 v24, v[96:99], s[22:23] sc1\n" \
    "v_rcp_f32 v109, v109\n" \
    "v_rcp_f32 v110, v110\n" \
    "v_rcp_f32 v111, v111\n" \
    "s_nop 0\n" \
    "v_pk_mul_f32 v[100:101], v[100:101], v[108:109]\n" \
    "v_pk_mul_f32 v[102:103], v[102:103], v[110:111]\n" \
    "global_store_dwordx4 v24, v[100:103], s[40:41] sc1\n" \
    "v_add_u32 v24, 0x40000, v24\n" \
    "s_waitcnt lgkmcnt(0)\n" \
    "ds_read_b128 v[160:163], v21 offset:4096\n" \
    "ds_read_b128 v[176:179], v22 offset:4096\n" \
    "ds_read_b128 v[164:167], v21 offset:4608\n" \
    "ds_read_b128 v[180:183], v22 offset:4608\n" \
    "ds_read_b128 v[168:171], v21 offset:36864\n" \
    "ds_read_b128 v[184:187], v22 offset:36864\n" \
    "ds_read_b128 v[172:175], v21 offset:37376\n" \
    "ds_read_b128 v[188:191], v22 offset:37376\n" \
    "v_pk_add_f32 v[128:129], v[128:129], v[144:145]\n" \
    "v_pk_add_f32 v[130:131], v[130:131], v[146:147]\n" \
    "v_pk_add_f32 v[132:133], v[132:133], v[148:149]\n" \
    "v_pk_add_f32 v[134:135], v[134:135], v[150:151]\n" \
    "v_pk_add_f32 v[136:137], v[136:137], v[152:153]\n" \
    "v_pk_add_f32 v[138:139], v[138:139], v[154:155]\n" \
    "v_pk_add_f32 v[140:141], v[140:141], v[156:157]\n" \
    "v_pk_add_f32 v[142:143], v[142:143], v[158:159]\n" \
    "v_pk_add_f32 v[128:129], v[128:129], v[136:137]\n" \
    "v_pk_add_f32 v[130:131], v[130:131], v[138:139]\n" \
    "v_pk_add_f32 v[132:133], v[132:133], v[140:141]\n" \
    "v_pk_add_f32 v[134:135], v[134:135], v[142:143]\n" \
    "v_pk_mul_f32 v[136:137], v[128:129], v[30:31]\n" \
    "v_pk_mul_f32 v[138:139], v[130:131], v[30:31]\n" \
    "v_pk_mul_f32 v[140:141], v[132:133], v[30:31]\n" \
    "v_pk_mul_f32 v[142:143], v[134:135], v[30:31]\n" \
    "v_med3_f32 v136, v136, -v25, v25\n" \
    "v_med3_f32 v137, v137, -v25, v25\n" \
    "v_med3_f32 v138, v138, -v25, v25\n" \
    "v_med3_f32 v139, v139, -v25, v25\n" \
    "v_med3_f32 v140, v140, -v25, v25\n" \
    "v_med3_f32 v141, v141, -v25, v25\n" \
    "v_med3_f32 v142, v142, -v25, v25\n" \
    "v_med3_f32 v143, v143, -v25, v25\n" \
    "v_exp_f32 v136, v136\n" \
    "v_exp_f32 v137, v137\n" \
    "v_exp_f32 v138, v138\n" \
    "v_exp_f32 v139, v139\n" \
    "v_exp_f32 v140, v140\n" \
    "v_exp_f32 v141, v141\n" \
    "v_exp_f32 v142, v142\n" \
    "v_exp_f32 v143, v143\n" \
    "v_pk_mul_f32 v[128:129], v[128:129], v[28:29]\n" \
    "v_pk_mul_f32 v[130:131], v[130:131], v[28:29]\n" \
    "v_pk_mul_f32 v[132:133], v[132:133], v[28:29]\n" \
    "v_pk_mul_f32 v[134:135], v[134:135], v[28:29]\n" \
    "v_pk_add_f32 v[136:137], v[136:137], 1.0 op_sel_hi:[1,0]\n" \
    "v_pk_add_f32 v[138:139], v[138:139], 1.0 op_sel_hi:[1,0]\n" \
    "v_pk_add_f32 v[140:141], v[140:141], 1.0 op_sel_hi:[1,0]\n" \
    "v_pk_add_f32 v[142:143], v[142:143], 1.0 op_sel_hi:[1,0]\n" \
    "v_rcp_f32 v136, v136\n" \
    "v_rcp_f32 v137, v137\n" \
    "v_rcp_f32 v138, v138\n" \
    "v_rcp_f32 v139, v139\n" \
    "v_rcp_f32 v140, v140\n" \
    "v_pk_mul_f32 v[128:129], v[128:129], v[136:137]\n" \
    "v_pk_mul_f32 v[130:131], v[130:131], v[138:139]\n" \
    "global_store_dwordx4 v24, v[128:131], s[22:23] sc1\n" \
    "v_rcp_f32 v141, v141\n" \
    "v_rcp_f32 v142, v142\n" \
    "v_rcp_f32 v143, v143\n" \
    "s_nop 0\n" \
    "v_pk_mul_f32 v[132:133], v[132:133], v[140:141]\n" \
    "v_pk_mul_f32 v[134:135], v[134:135], v[142:143]\n" \
    "global_store_dwordx4 v24, v[132:135], s[40:41] sc1\n" \
    "v_add_u32 v24, 0x40000, v24\n" \
    "s_waitcnt lgkmcnt(0)\n" \
    "ds_read_b128 v[192:195], v21 offset:6144\n" \
    "ds_read_b128 v[208:211], v22 offset:6144\n" \
    "ds_read_b128 v[196:199], v21 offset:6656\n" \
    "ds_read_b128 v[212:215], v22 offset:6656\n" \
    "ds_read_b128 v[200:203], v21 offset:38912\n" \
    "ds_read_b128 v[216:219], v22 offset:38912\n" \
    "ds_read_b128 v[204:207], v21 offset:39424\n" \
    "ds_read_b128 v[220:223], v22 offset:39424\n" \
    "v_pk_add_f32 v[160:161], v[160:161], v[176:177]\n" \
    "v_pk_add_f32 v[162:163], v[162:163], v[178:179]\n" \
    "v_pk_add_f32 v[164:165], v[164:165], v[180:181]\n" \
    "v_pk_add_f32 v[166:167], v[166:167], v[182:183]\n" \
    "v_pk_add_f32 v[168:169], v[168:169], v[184:185]\n" \
    "v_pk_add_f32 v[170:171], v[170:171], v[186:187]\n" \
    "v_pk_add_f32 v[172:173], v[172:173], v[188:189]\n" \
    "v_pk_add_f32 v[174:175], v[174:175], v[190:191]\n" \
    "v_pk_add_f32 v[160:161], v[160:161], v[168:169]\n" \
    "v_pk_add_f32 v[162:163], v[162:163], v[170:171]\n" \
    "v_pk_add_f32 v[164:165], v[164:165], v[172:173]\n" \
    "v_pk_add_f32 v[166:167], v[166:167], v[174:175]\n" \
    "v_pk_mul_f32 v[168:169], v[160:161], v[30:31]\n" \
    "v_pk_mul_f32 v[170:171], v[162:163], v[30:31]\n" \
    "v_pk_mul_f32 v[172:173], v[164:165], v[30:31]\n" \
    "v_pk_mul_f32 v[174:175], v[166:167], v[30:31]\n" \
    "v_med3_f32 v168, v168, -v25, v25\n" \
    "v_med3_f32 v169, v169, -v25, v25\n" \
    "v_med3_f32 v170, v170, -v25, v25\n" \
    "v_med3_f32 v171, v171, -v25, v25\n" \
    "v_med3_f32 v172, v172, -v25, v25\n" \
    "v_med3_f32 v173, v173, -v25, v25\n" \
    "v_med3_f32 v174, v174, -v25, v25\n" \
    "v_med3_f32 v175, v175, -v25, v25\n" \
    "v_exp_f32 v168, v168\n" \
    "v_exp_f32 v169, v169\n" \
    "v_exp_f32 v170, v170\n" \
    "v_exp_f32 v171, v171\n" \
    "v_exp_f32 v172, v172\n" \
    "v_exp_f32 v173, v173\n" \
    "v_exp_f32 v174, v174\n" \
    "v_exp_f32 v175, v175\n" \
    "v_pk_mul_f32 v[160:161], v[160:161], v[28:29]\n" \
    "v_pk_mul_f32 v[162:163], v[162:163], v[28:29]\n" \
    "v_pk_mul_f32 v[164:165], v[164:165], v[28:29]\n" \
    "v_pk_mul_f32 v[166:167], v[166:167], v[28:29]\n" \
    "v_pk_add_f32 v[168:169], v[168:169], 1.0 op_sel_hi:[1,0]\n" \
    "v_pk_add_f32 v[170:171], v[170:171], 1.0 op_sel_hi:[1,0]\n" \
    "v_pk_add_f32 v[172:173], v[172:173], 1.0 op_sel_hi:[1,0]\n" \
    "v_pk_add_f32 v[174:175], v[174:175], 1.0 op_sel_hi:[1,0]\n" \
    "v_rcp_f32 v168, v168\n" \
    "v_rcp_f32 v169, v169\n" \
    "v_rcp_f32 v170, v170\n" \
    "v_rcp_f32 v171, v171\n" \
    "v_rcp_f32 v172, v172\n" \
    "v_pk_mul_f32 v[160:161], v[160:161], v[168:169]\n" \
    "v_pk_mul_f32 v[162:163], v[162:163], v[170:171]\n" \
    "global_store_dwordx4 v24, v[160:163], s[22:23] sc1\n" \
    "v_rcp_f32 v173, v173\n" \
    "v_rcp_f32 v174, v174\n" \
    "v_rcp_f32 v175, v175\n" \
    "s_nop 0\n" \
    "v_pk_mul_f32 v[164:165], v[164:165], v[172:173]\n" \
    "v_pk_mul_f32 v[166:167], v[166:167], v[174:175]\n" \
    "global_store_dwordx4 v24, v[164:167], s[40:41] sc1\n" \
    "v_add_u32 v24, 0x40000, v24\n" \
    "s_waitcnt lgkmcnt(0)\n" \
    "ds_read_b128 v[224:227], v23 offset:0\n" \
    "ds_read_b128 v[240:243], v23 offset:16384\n" \
    "ds_read_b128 v[228:231], v23 offset:512\n" \
    "ds_read_b128 v[244:247], v23 offset:16896\n" \
    "ds_read_b128 v[232:235], v23 offset:8192\n" \
    "ds_read_b128 v[248:251], v23 offset:24576\n" \
    "ds_read_b128 v[236:239], v23 offset:8704\n" \
    "ds_read_b128 v[252:255], v23 offset:25088\n" \
    "v_pk_add_f32 v[192:193], v[192:193], v[208:209]\n" \
    "v_pk_add_f32 v[194:195], v[194:195], v[210:211]\n" \
    "v_pk_add_f32 v[196:197], v[196:197], v[212:213]\n" \
    "v_pk_add_f32 v[198:199], v[198:199], v[214:215]\n" \
    "v_pk_add_f32 v[200:201], v[200:201], v[216:217]\n" \
    "v_pk_add_f32 v[202:203], v[202:203], v[218:219]\n" \
    "v_pk_add_f32 v[204:205], v[204:205], v[220:221]\n" \
    "v_pk_add_f32 v[206:207], v[206:207], v[222:223]\n" \
    "v_pk_add_f32 v[192:193], v[192:193], v[200:201]\n" \
    "v_pk_add_f32 v[194:195], v[194:195], v[202:203]\n" \
    "v_pk_add_f32 v[196:197], v[196:197], v[204:205]\n" \
    "v_pk_add_f32 v[198:199], v[198:199], v[206:207]\n" \
    "v_pk_mul_f32 v[200:201], v[192:193], v[30:31]\n" \
    "v_pk_mul_f32 v[202:203], v[194:195], v[30:31]\n" \
    "v_pk_mul_f32 v[204:205], v[196:197], v[30:31]\n" \
    "v_pk_mul_f32 v[206:207], v[198:199], v[30:31]\n" \
    "v_med3_f32 v200, v200, -v25, v25\n" \
    "v_med3_f32 v201, v201, -v25, v25\n" \
    "v_med3_f32 v202, v202, -v25, v25\n" \
    "v_med3_f32 v203, v203, -v25, v25\n" \
    "v_med3_f32 v204, v204, -v25, v25\n" \
    "v_med3_f32 v205, v205, -v25, v25\n" \
    "v_med3_f32 v206, v206, -v25, v25\n" \
    "v_med3_f32 v207, v207, -v25, v25\n" \
    "v_exp_f32 v200, v200\n" \
    "v_exp_f32 v201, v201\n" \
    "v_exp_f32 v202, v202\n" \
    "v_exp_f32 v203, v203\n" \
    "v_exp_f32 v204, v204\n" \
    "v_exp_f32 v205, v205\n" \
    "v_exp_f32 v206, v206\n" \
    "v_exp_f32 v207, v207\n" \
    "v_pk_mul_f32 v[192:193], v[192:193], v[28:29]\n" \
    "v_pk_mul_f32 v[194:195], v[194:195], v[28:29]\n" \
    "v_pk_mul_f32 v[196:197], v[196:197], v[28:29]\n" \
    "v_pk_mul_f32 v[198:199], v[198:199], v[28:29]\n" \
    "v_pk_add_f32 v[200:201], v[200:201], 1.0 op_sel_hi:[1,0]\n" \
    "v_pk_add_f32 v[202:203], v[202:203], 1.0 op_sel_hi:[1,0]\n" \
    "v_pk_add_f32 v[204:205], v[204:205], 1.0 op_sel_hi:[1,0]\n" \
    "v_pk_add_f32 v[206:207], v[206:207], 1.0 op_sel_hi:[1,0]\n" \
    "v_rcp_f32 v200, v200\n" \
    "v_rcp_f32 v201, v201\n" \
    "v_rcp_f32 v202, v202\n" \
    "v_rcp_f32 v203, v203\n" \
    "v_rcp_f32 v204, v204\n" \
    "v_pk_mul_f32 v[192:193], v[192:193], v[200:201]\n" \
    "v_pk_mul_f32 v[194:195], v[194:195], v[202:203]\n" \
    "global_store_dwordx4 v24, v[192:195], s[22:23] sc1\n" \
    "v_rcp_f32 v205, v205\n" \
    "v_rcp_f32 v206, v206\n" \
    "v_rcp_f32 v207, v207\n" \
    "s_nop 0\n" \
    "v_pk_mul_f32 v[196:197], v[196:197], v[204:205]\n" \
    "v_pk_mul_f32 v[198:199], v[198:199], v[206:207]\n" \
    "global_store_dwordx4 v24, v[196:199], s[40:41] sc1\n" \
    "v_add_u32 v24, 0x40000, v24\n" \
    "s_cmp_eq_u32 s80, 0\n" \
    "s_cbranch_scc1 a2a_done\n" \
    "s_waitcnt lgkmcnt(0)\n" \
    "v_pk_add_f32 v[224:225], v[224:225], v[240:241]\n" \
    "v_pk_add_f32 v[226:227], v[226:227], v[242:243]\n" \
    "v_pk_add_f32 v[228:229], v[228:229], v[244:245]\n" \
    "v_pk_add_f32 v[230:231], v[230:231], v[246:247]\n" \
    "v_pk_add_f32 v[232:233], v[232:233], v[248:249]\n" \
    "v_pk_add_f32 v[234:235], v[234:235], v[250:251]\n" \
    "v_pk_add_f32 v[236:237], v[236:237], v[252:253]\n" \
    "v_pk_add_f32 v[238:239], v[238:239], v[254:255]\n" \
    "v_pk_add_f32 v[224:225], v[224:225], v[232:233]\n" \
    "v_pk_add_f32 v[226:227], v[226:227], v[234:235]\n" \
    "v_pk_add_f32 v[228:229], v[228:229], v[236:237]\n" \
    "v_pk_add_f32 v[230:231], v[230:231], v[238:239]\n" \
    "v_pk_mul_f32 v[232:233], v[224:225], v[30:31]\n" \
    "v_pk_mul_f32 v[234:235], v[226:227], v[30:31]\n" \
    "v_pk_mul_f32 v[236:237], v[228:229], v[30:31]\n" \
    "v_pk_mul_f32 v[238:239], v[230:231], v[30:31]\n" \
    "v_med3_f32 v232, v232, -v25, v25\n" \
    "v_med3_f32 v233, v233, -v25, v25\n" \
    "v_med3_f32 v234, v234, -v25, v25\n" \
    "v_med3_f32 v235, v235, -v25, v25\n" \
    "v_med3_f32 v236, v236, -v25, v25\n" \
    "v_med3_f32 v237, v237, -v25, v25\n" \
    "v_med3_f32 v238, v238, -v25, v25\n" \
    "v_med3_f32 v239, v239, -v25, v25\n" \
    "v_exp_f32 v232, v232\n" \
    "v_exp_f32 v233, v233\n" \
    "v_exp_f32 v234, v234\n" \
    "v_exp_f32 v235, v235\n" \
    "v_exp_f32 v236, v236\n" \
    "v_exp_f32 v237, v237\n" \
    "v_exp_f32 v238, v238\n" \
    "v_exp_f32 v239, v239\n" \
    "v_pk_mul_f32 v[224:225], v[224:225], v[28:29]\n" \
    "v_pk_mul_f32 v[226:227], v[226:227], v[28:29]\n" \
    "v_pk_mul_f32 v[228:229], v[228:229], v[28:29]\n" \
    "v_pk_mul_f32 v[230:231], v[230:231], v[28:29]\n" \
    "v_pk_add_f32 v[232:233], v[232:233], 1.0 op_sel_hi:[1,0]\n" \
    "v_pk_add_f32 v[234:235], v[234:235], 1.0 op_sel_hi:[1,0]\n" \
    "v_pk_add_f32 v[236:237], v[236:237], 1.0 op_sel_hi:[1,0]\n" \
    "v_pk_add_f32 v[238:239], v[238:239], 1.0 op_sel_hi:[1,0]\n" \
    "v_rcp_f32 v232, v232\n" \
    "v_rcp_f32 v233, v233\n" \
    "v_rcp_f32 v234, v234\n" \
    "v_rcp_f32 v235, v235\n" \
    "v_rcp_f32 v236, v236\n" \
    "v_pk_mul_f32 v[224:225], v[224:225], v[232:233]\n" \
    "v_pk_mul_f32 v[226:227], v[226:227], v[234:235]\n" \
    "global_store_dwordx4 v24, v[224:227], s[22:23] sc1\n" \
    "v_rcp_f32 v237, v237\n" \
    "v_rcp_f32 v238, v238\n" \
    "v_rcp_f32 v239, v239\n" \
    "s_nop 0\n" \
    "v_pk_mul_f32 v[228:229], v[228:229], v[236:237]\n" \
    "v_pk_mul_f32 v[230:231], v[230:231], v[238:239]\n" \
    "global_store_dwordx4 v24, v[228:231], s[40:41] sc1\n" \
    "v_add_u32 v24, 0x40000, v24\n" \
      \
    "a2a_done:\n" \
    "s_cmp_lg_u32 s44, 0\n" \
    "s_cbranch_scc1 a2a_end\n" \
    "v_mov_b32 v10, s59\n" \
    "s_mov_b64 exec, 1\n" \
    "global_store_dword v9, v10, s[16:17] sc1\n" \
    "s_mov_b64 exec, -1\n" \
    "s_branch a2a_end\n" \
      \
    "a2a_u1_alt4:\n" \
    "s_xor_b32 m0, s76, 0\n" \
    "s_xor_b32 s58, s77, 0\n" \
    "buffer_load_dwordx4 v4, s[8:11], s58 offen offset:0 lds\n" \
    "s_add_u32 s58, s58, 31744\n" \
    "buffer_load_dwordx4 v4, s[8:11], s58 offen offset:1024 lds\n" \
    "s_add_u32 s58, s58, 31744\n" \
    "buffer_load_dwordx4 v5, s[8:11], s58 offen offset:2048 lds\n" \
    "s_add_u32 s58, s58, 31744\n" \
    "buffer_load_dwordx4 v5, s[8:11], s58 offen offset:3072 lds\n" \
    "s_xor_b32 m0, s76, 4096\n" \
    "s_xor_b32 s58, s77, 131072\n" \
    "buffer_load_dwordx4 v4, s[8:11], s58 offen offset:0 lds\n" \
    "s_add_u32 s58, s58, 31744\n" \
    "buffer_load_dwordx4 v4, s[8:11], s58 offen offset:1024 lds\n" \
    "s_branch a2a_u1_a5\n" \
    "a2a_u1_nolw:\n" \
    "s_cmp_eq_u32 s80, 0\n" \
    "s_cbranch_scc1 a2a_u1_n5\n" \
    "v_mfma_f32_16x16x128_f8f6f4 v[224:227], a[72:79], a[144:151], v[224:227] cbsz:1 blgp:1\n" \
    "buffer_load_dwordx4 a[32:35], v14, s[16:19], s65 offen offset:0 sc0 sc1\n" \
    "v_mfma_f32_16x16x128_f8f6f4 v[228:231], a[72:79], a[152:159], v[228:231] cbsz:1 blgp:1\n" \
    "buffer_load_dwordx4 a[36:39], v14, s[16:19], s65 offen offset:1024 sc0 sc1\n" \
    "v_mfma_f32_16x16x128_f8f6f4 v[232:235], a[72:79], a[160:167], v[232:235] cbsz:1 blgp:1\n" \
    "v_mfma_f32_16x16x128_f8f6f4 v[236:239], a[72:79], a[168:175], v[236:239] cbsz:1 blgp:1\n" \
    "v_mfma_f32_16x16x128_f8f6f4 v[240:243], a[72:79], a[176:183], v[240:243] cbsz:1 blgp:1\n" \
    "v_mfma_f32_16x16x128_f8f6f4 v[244:247], a[72:79], a[184:191], v[244:247] cbsz:1 blgp:1\n" \
    "v_mfma_f32_16x16x128_f8f6f4 v[248:251], a[72:79], a[192:199], v[248:251] cbsz:1 blgp:1\n" \
    "v_mfma_f32_16x16x128_f8f6f4 v[252:255], a[72:79], a[200:207], v[252:255] cbsz:1 blgp:1\n" \
    "a2a_u1_n5:\n" \
    "v_mfma_f32_16x16x128_f8f6f4 v[144:147], a[48:55], a[176:183], v[144:147] cbsz:1 blgp:1\n" \
    "v_mfma_f32_16x16x128_f8f6f4 v[148:151], a[48:55], a[184:191], v[148:151] cbsz:1 blgp:1\n" \
    "s_waitcnt vmcnt(8)\n" \
    "a2a_u1_h2n:\n" \
    "v_mfma_f32_16x16x128_f8f6f4 v[152:155], a[48:55], a[192:199], v[152:155] cbsz:1 blgp:1\n" \
    "global_load_dword v10, v15, s[38:39] sc0 sc1\n" \
    "s_cmp_lg_u32 s89, 0\n" \
    "s_cbranch_scc1 a2a_u1_h2q\n" \
    "v_mfma_f32_16x16x128_f8f6f4 v[156:159], a[48:55], a[200:207], v[156:159] cbsz:1 blgp:1\n" \
    "v_xor_b32 v11, 0, v12\n" \
    "ds_read_b64_tr_b8 a[80:81], v11\n" \
    "ds_read_b64_tr_b8 a[82:83], v11 offset:1024\n" \
    "v_mfma_f32_16x16x128_f8f6f4 v[160:163], a[56:63], a[144:151], v[160:163] cbsz:1 blgp:1\n" \
    "ds_read_b64_tr_b8 a[84:85], v11 offset:8192\n" \
    "ds_read_b64_tr_b8 a[86:87], v11 offset:9216\n" \
    "v_mfma_f32_16x16x128_f8f6f4 v[164:167], a[56:63], a[152:159], v[164:167] cbsz:1 blgp:1\n" \
    "v_xor_b32 v11, 16, v12\n" \
    "ds_read_b64_tr_b8 a[88:89], v11\n" \
    "ds_read_b64_tr_b8 a[90:91], v11 offset:1024\n" \
    "v_mfma_f32_16x16x128_f8f6f4 v[168:171], a[56:63], a[160:167], v[168:171] cbsz:1 blgp:1\n" \
    "ds_read_b64_tr_b8 a[92:93], v11 offset:8192\n" \
    "ds_read_b64_tr_b8 a[94:95], v11 offset:9216\n" \
    "v_mfma_f32_16x16x128_f8f6f4 v[172:175], a[56:63], a[168:175], v[172:175] cbsz:1 blgp:1\n" \
    "v_xor_b32 v11, 32, v12\n" \
    "ds_read_b64_tr_b8 a[96:97], v11\n" \
    "ds_read_b64_tr_b8 a[98:99], v11 offset:1024\n" \
    "v_mfma_f32_16x16x128_f8f6f4 v[176:179], a[56:63], a[176:183], v[176:179] cbsz:1 blgp:1\n" \
    "ds_read_b64_tr_b8 a[100:101], v11 offset:8192\n" \
    "ds_read_b64_tr_b8 a[102:103], v11 offset:9216\n" \
    "v_mfma_f32_16x16x128_f8f6f4 v[180:183], a[56:63], a[184:191], v[180:183] cbsz:1 blgp:1\n" \
    "v_xor_b32 v11, 48, v12\n" \
    "ds_read_b64_tr_b8 a[104:105], v11\n" \
    "ds_read_b64_tr_b8 a[106:107], v11 offset:1024\n" \
    "v_mfma_f32_16x16x128_f8f6f4 v[184:187], a[56:63], a[192:199], v[184:187] cbsz:1 blgp:1\n" \
    "ds_read_b64_tr_b8 a[108:109], v11 offset:8192\n" \
    "ds_read_b64_tr_b8 a[110:111], v11 offset:9216\n" \
    "v_mfma_f32_16x16x128_f8f6f4 v[188:191], a[56:63], a[200:207], v[188:191] cbsz:1 blgp:1\n" \
    "v_xor_b32 v11, 64, v12\n" \
    "ds_read_b64_tr_b8 a[112:113], v11\n" \
    "ds_read_b64_tr_b8 a[114:115], v11 offset:1024\n" \
    "v_mfma_f32_16x16x128_f8f6f4 v[192:195], a[64:71], a[144:151], v[192:195] cbsz:1 blgp:1\n" \
    "ds_read_b64_tr_b8 a[116:117], v11 offset:8192\n" \
    "ds_read_b64_tr_b8 a[118:119], v11 offset:9216\n" \
    "v_mfma_f32_16x16x128_f8f6f4 v[196:199], a[64:71], a[152:159], v[196:199] cbsz:1 blgp:1\n" \
    "v_xor_b32 v11, 80, v12\n" \
    "ds_read_b64_tr_b8 a[120:121], v11\n" \
    "ds_read_b64_tr_b8 a[122:123], v11 offset:1024\n" \
    "v_mfma_f32_16x16x128_f8f6f4 v[200:203], a[64:71], a[160:167], v[200:203] cbsz:1 blgp:1\n" \
    "ds_read_b64_tr_b8 a[124:125], v11 offset:8192\n" \
    "ds_read_b64_tr_b8 a[126:127], v11 offset:9216\n" \
    "v_mfma_f32_16x16x128_f8f6f4 v[204:207], a[64:71], a[168:175], v[204:207] cbsz:1 blgp:1\n" \
    "v_xor_b32 v11, 96, v12\n" \
    "ds_read_b64_tr_b8 a[128:129], v11\n" \
    "ds_read_b64_tr_b8 a[130:131], v11 offset:1024\n" \
    "v_mfma_f32_16x16x128_f8f6f4 v[208:211], a[64:71], a[176:183], v[208:211] cbsz:1 blgp:1\n" \
    "ds_read_b64_tr_b8 a[132:133], v11 offset:8192\n" \
    "ds_read_b64_tr_b8 a[134:135], v11 offset:9216\n" \
    "v_mfma_f32_16x16x128_f8f6f4 v[212:215], a[64:71], a[184:191], v[212:215] cbsz:1 blgp:1\n" \
    "v_xor_b32 v11, 112, v12\n" \
    "ds_read_b64_tr_b8 a[136:137], v11\n" \
    "ds_read_b64_tr_b8 a[138:139], v11 offset:1024\n" \
    "v_mfma_f32_16x16x128_f8f6f4 v[216:219], a[64:71], a[192:199], v[216:219] cbsz:1 blgp:1\n" \
    "ds_read_b64_tr_b8 a[140:141], v11 offset:8192\n" \
    "ds_read_b64_tr_b8 a[142:143], v11 offset:9216\n" \
    "v_mfma_f32_16x16x128_f8f6f4 v[220:223], a[64:71], a[200:207], v[220:223] cbsz:1 blgp:1\n" \
    "s_mov_b32 s69, s70\n" \
    "s_mov_b32 s70, 7\n" \
    "s_xor_b32 s72, s72, 16384\n" \
    "s_add_u32 s73, s73, 1\n" \
    "s_mov_b32 s75, 1\n" \
    "s_cmp_eq_u32 s74, 0\n" \
    "s_waitcnt vmcnt(0) lgkmcnt(0)\n" \
    "s_cbranch_scc1 a2a_stall\n" \
    "s_branch a2a_c1\n" \
    "a2a_u1_pk:\n" \
    "s_lshr_b32 s55, s66, s71\n" \
    "s_cmp_ge_u32 s73, s44\n" \
    "s_cselect_b32 s55, 0, s55\n" \
    "s_lshr_b32 s56, s66, s88\n" \
    "s_andn2_b32 s89, s56, s55\n" \
    "s_cmp_eq_u32 s69, 7\n" \
    "s_cselect_b32 s89, 1, s89\n" \
    "s_and_b32 s89, s89, 1\n" \
    "s_cselect_b32 s71, s88, s71\n" \
    "s_branch a2a_u1_pkr\n" \
    "a2a_u1_h2p:\n" \
    "v_mfma_f32_16x16x128_f8f6f4 v[156:159], a[48:55], a[200:207], v[156:159] cbsz:1 blgp:1\n" \
    "v_accvgpr_write_b32 a80, v32\n" \
    "v_accvgpr_write_b32 a81, v33\n" \
    "v_accvgpr_write_b32 a82, v34\n" \
    "v_accvgpr_write_b32 a83, v35\n" \
    "v_mfma_f32_16x16x128_f8f6f4 v[160:163], a[56:63], a[144:151], v[160:163] cbsz:1 blgp:1\n" \
    "s_xor_b32 m0, s76, 8192\n" \
    "s_xor_b32 s58, s77, 262144\n" \
    "buffer_load_dwordx4 v4, s[8:11], s58 offen offset:0 lds\n" \
    "v_accvgpr_write_b32 a84, v36\n" \
    "v_accvgpr_write_b32 a85, v37\n" \
    "v_accvgpr_write_b32 a86, v38\n" \
    "v_accvgpr_write_b32 a87, v39\n" \
    "v_mfma_f32_16x16x128_f8f6f4 v[164:167], a[56:63], a[152:159], v[164:167] cbsz:1 blgp:1\n" \
    "v_accvgpr_write_b32 a88, v40\n" \
    "v_accvgpr_write_b32 a89, v41\n" \
    "v_accvgpr_write_b32 a90, v42\n" \
    "v_accvgpr_write_b32 a91, v43\n" \
    "v_mfma_f32_16x16x128_f8f6f4 v[168:171], a[56:63], a[160:167], v[168:171] cbsz:1 blgp:1\n" \
    "s_add_u32 s58, s58, 31744\n" \
    "buffer_load_dwordx4 v4, s[8:11], s58 offen offset:1024 lds\n" \
    "v_accvgpr_write_b32 a92, v44\n" \
    "v_accvgpr_write_b32 a93, v45\n" \
    "v_accvgpr_write_b32 a94, v46\n" \
    "v_accvgpr_write_b32 a95, v47\n" \
    "v_mfma_f32_16x16x128_f8f6f4 v[172:175], a[56:63], a[168:175], v[172:175] cbsz:1 blgp:1\n" \
    "v_accvgpr_write_b32 a96, v48\n" \
    "v_accvgpr_write_b32 a97, v49\n" \
    "v_accvgpr_write_b32 a98, v50\n" \
    "v_accvgpr_write_b32 a99, v51\n" \
    "v_mfma_f32_16x16x128_f8f6f4 v[176:179], a[56:63], a[176:183], v[176:179] cbsz:1 blgp:1\n" \
    "s_add_u32 s58, s58, 31744\n" \
    "buffer_load_dwordx4 v5, s[8:11], s58 offen offset:2048 lds\n" \
    "v_accvgpr_write_b32 a100, v52\n" \
    "v_accvgpr_write_b32 a101, v53\n" \
    "v_accvgpr_write_b32 a102, v54\n" \
    "v_accvgpr_write_b32 a103, v55\n" \
    "v_mfma_f32_16x16x128_f8f6f4 v[180:183], a[56:63], a[184:191], v[180:183] cbsz:1 blgp:1\n" \
    "v_accvgpr_write_b32 a104, v56\n" \
    "v_accvgpr_write_b32 a105, v57\n" \
    "v_accvgpr_write_b32 a106, v58\n" \
    "v_accvgpr_write_b32 a107, v59\n" \
    "v_mfma_f32_16x16x128_f8f6f4 v[184:187], a[56:63], a[192:199], v[184:187] cbsz:1 blgp:1\n" \
    "s_add_u32 s58, s58, 31744\n" \
    "buffer_load_dwordx4 v5, s[8:11], s58 offen offset:3072 lds\n" \
    "v_accvgpr_write_b32 a108, v60\n" \
    "v_accvgpr_write_b32 a109, v61\n" \
    "v_accvgpr_write_b32 a110, v62\n" \
    "v_accvgpr_write_b32 a111, v63\n" \
    "v_mfma_f32_16x16x128_f8f6f4 v[188:191], a[56:63], a[200:207], v[188:191] cbsz:1 blgp:1\n" \
    "v_accvgpr_write_b32 a112, v64\n" \
    "v_accvgpr_write_b32 a113, v65\n" \
    "v_accvgpr_write_b32 a114, v66\n" \
    "v_accvgpr_write_b32 a115, v67\n" \
    "v_mfma_f32_16x16x128_f8f6f4 v[192:195], a[64:71], a[144:151], v[192:195] cbsz:1 blgp:1\n" \
    "s_xor_b32 m0, s76, 12288\n" \
    "s_xor_b32 s58, s77, 393216\n" \
    "buffer_load_dwordx4 v4, s[8:11], s58 offen offset:0 lds\n" \
    "v_accvgpr_write_b32 a116, v68\n" \
    "v_accvgpr_write_b32 a117, v69\n" \
    "v_accvgpr_write_b32 a118, v70\n" \
    "v_accvgpr_write_b32 a119, v71\n" \
    "v_mfma_f32_16x16x128_f8f6f4 v[196:199], a[64:71], a[152:159], v[196:199] cbsz:1 blgp:1\n" \
    "v_accvgpr_write_b32 a120, v72\n" \
    "v_accvgpr_write_b32 a121, v73\n" \
    "v_accvgpr_write_b32 a122, v74\n" \
    "v_accvgpr_write_b32 a123, v75\n" \
    "v_mfma_f32_16x16x128_f8f6f4 v[200:203], a[64:71], a[160:167], v[200:203] cbsz:1 blgp:1\n" \
    "s_add_u32 s58, s58, 31744\n" \
    "buffer_load_dwordx4 v4, s[8:11], s58 offen offset:1024 lds\n" \
    "v_accvgpr_write_b32 a124, v76\n" \
    "v_accvgpr_write_b32 a125, v77\n" \
    "v_accvgpr_write_b32 a126, v78\n" \
    "v_accvgpr_write_b32 a127, v79\n" \
    "v_mfma_f32_16x16x128_f8f6f4 v[204:207], a[64:71], a[168:175], v[204:207] cbsz:1 blgp:1\n" \
    "v_accvgpr_write_b32 a128, v80\n" \
    "v_accvgpr_write_b32 a129, v81\n" \
    "v_accvgpr_write_b32 a130, v82\n" \
    "v_accvgpr_write_b32 a131, v83\n" \
    "v_mfma_f32_16x16x128_f8f6f4 v[208:211], a[64:71], a[176:183], v[208:211] cbsz:1 blgp:1\n" \
    "s_add_u32 s58, s58, 31744\n" \
    "buffer_load_dwordx4 v5, s[8:11], s58 offen offset:2048 lds\n" \
    "v_accvgpr_write_b32 a132, v84\n" \
    "v_accvgpr_write_b32 a133, v85\n" \
    "v_accvgpr_write_b32 a134, v86\n" \
    "v_accvgpr_write_b32 a135, v87\n" \
    "v_mfma_f32_16x16x128_f8f6f4 v[212:215], a[64:71], a[184:191], v[212:215] cbsz:1 blgp:1\n" \
    "v_accvgpr_write_b32 a136, v88\n" \
    "v_accvgpr_write_b32 a137, v89\n" \
    "v_accvgpr_write_b32 a138, v90\n" \
    "v_accvgpr_write_b32 a139, v91\n" \
    "v_mfma_f32_16x16x128_f8f6f4 v[216:219], a[64:71], a[192:199], v[216:219] cbsz:1 blgp:1\n" \
    "s_add_u32 s58, s58, 31744\n" \
    "buffer_load_dwordx4 v5, s[8:11], s58 offen offset:3072 lds\n" \
    "v_accvgpr_write_b32 a140, v92\n" \
    "v_accvgpr_write_b32 a141, v93\n" \
    "v_accvgpr_write_b32 a142, v94\n" \
    "v_accvgpr_write_b32 a143, v95\n" \
    "v_mfma_f32_16x16x128_f8f6f4 v[220:223], a[64:71], a[200:207], v[220:223] cbsz:1 blgp:1\n" \
    "s_mov_b32 s88, 7\n" \
    "s_mov_b32 s89, 0\n" \
    "s_add_u32 s73, s73, 1\n" \
    "s_waitcnt vmcnt(17)\n" \
    "s_branch a2a_c1\n" \
    "a2a_u1_h2q:\n" \
    "v_mfma_f32_16x16x128_f8f6f4 v[156:159], a[48:55], a[200:207], v[156:159] cbsz:1 blgp:1\n" \
    "v_accvgpr_write_b32 a80, v32\n" \
    "v_accvgpr_write_b32 a81, v33\n" \
    "v_accvgpr_write_b32 a82, v34\n" \
    "v_accvgpr_write_b32 a83, v35\n" \
    "v_mfma_f32_16x16x128_f8f6f4 v[160:163], a[56:63], a[144:151], v[160:163] cbsz:1 blgp:1\n" \
    "v_accvgpr_write_b32 a84, v36\n" \
    "v_accvgpr_write_b32 a85, v37\n" \
    "v_accvgpr_write_b32 a86, v38\n" \
    "v_accvgpr_write_b32 a87, v39\n" \
    "v_mfma_f32_16x16x128_f8f6f4 v[164:167], a[56:63], a[152:159], v[164:167] cbsz:1 blgp:1\n" \
    "v_accvgpr_write_b32 a88, v40\n" \
    "v_accvgpr_write_b32 a89, v41\n" \
    "v_accvgpr_write_b32 a90, v42\n" \
    "v_accvgpr_write_b32 a91, v43\n" \
    "v_mfma_f32_16x16x128_f8f6f4 v[168:171], a[56:63], a[160:167], v[168:171] cbsz:1 blgp:1\n" \
    "v_accvgpr_write_b32 a92, v44\n" \
    "v_accvgpr_write_b32 a93, v45\n" \
    "v_accvgpr_write_b32 a94, v46\n" \
    "v_accvgpr_write_b32 a95, v47\n" \
    "v_mfma_f32_16x16x128_f8f6f4 v[172:175], a[56:63], a[168:175], v[172:175] cbsz:1 blgp:1\n" \
    "v_accvgpr_write_b32 a96, v48\n" \
    "v_accvgpr_write_b32 a97, v49\n" \
    "v_accvgpr_write_b32 a98, v50\n" \
    "v_accvgpr_write_b32 a99, v51\n" \
    "v_mfma_f32_16x16x128_f8f6f4 v[176:179], a[56:63], a[176:183], v[176:179] cbsz:1 blgp:1\n" \
    "v_accvgpr_write_b32 a100, v52\n" \
    "v_accvgpr_write_b32 a101, v53\n" \
    "v_accvgpr_write_b32 a102, v54\n" \
    "v_accvgpr_write_b32 a103, v55\n" \
    "v_mfma_f32_16x16x128_f8f6f4 v[180:183], a[56:63], a[184:191], v[180:183] cbsz:1 blgp:1\n" \
    "v_accvgpr_write_b32 a104, v56\n" \
    "v_accvgpr_write_b32 a105, v57\n" \
    "v_accvgpr_write_b32 a106, v58\n" \
    "v_accvgpr_write_b32 a107, v59\n" \
    "v_mfma_f32_16x16x128_f8f6f4 v[184:187], a[56:63], a[192:199], v[184:187] cbsz:1 blgp:1\n" \
    "v_accvgpr_write_b32 a108, v60\n" \
    "v_accvgpr_write_b32 a109, v61\n" \
    "v_accvgpr_write_b32 a110, v62\n" \
    "v_accvgpr_write_b32 a111, v63\n" \
    "v_mfma_f32_16x16x128_f8f6f4 v[188:191], a[56:63], a[200:207], v[188:191] cbsz:1 blgp:1\n" \
    "v_accvgpr_write_b32 a112, v64\n" \
    "v_accvgpr_write_b32 a113, v65\n" \
    "v_accvgpr_write_b32 a114, v66\n" \
    "v_accvgpr_write_b32 a115, v67\n" \
    "v_mfma_f32_16x16x128_f8f6f4 v[192:195], a[64:71], a[144:151], v[192:195] cbsz:1 blgp:1\n" \
    "v_accvgpr_write_b32 a116, v68\n" \
    "v_accvgpr_write_b32 a117, v69\n" \
    "v_accvgpr_write_b32 a118, v70\n" \
    "v_accvgpr_write_b32 a119, v71\n" \
    "v_mfma_f32_16x16x128_f8f6f4 v[196:199], a[64:71], a[152:159], v[196:199] cbsz:1 blgp:1\n" \
    "v_accvgpr_write_b32 a120, v72\n" \
    "v_accvgpr_write_b32 a121, v73\n" \
    "v_accvgpr_write_b32 a122, v74\n" \
    "v_accvgpr_write_b32 a123, v75\n" \
    "v_mfma_f32_16x16x128_f8f6f4 v[200:203], a[64:71], a[160:167], v[200:203] cbsz:1 blgp:1\n" \
    "v_accvgpr_write_b32 a124, v76\n" \
    "v_accvgpr_write_b32 a125, v77\n" \
    "v_accvgpr_write_b32 a126, v78\n" \
    "v_accvgpr_write_b32 a127, v79\n" \
    "v_mfma_f32_16x16x128_f8f6f4 v[204:207], a[64:71], a[168:175], v[204:207] cbsz:1 blgp:1\n" \
    "v_accvgpr_write_b32 a128, v80\n" \
    "v_accvgpr_write_b32 a129, v81\n" \
    "v_accvgpr_write_b32 a130, v82\n" \
    "v_accvgpr_write_b32 a131, v83\n" \
    "v_mfma_f32_16x16x128_f8f6f4 v[208:211], a[64:71], a[176:183], v[208:211] cbsz:1 blgp:1\n" \
    "v_accvgpr_write_b32 a132, v84\n" \
    "v_accvgpr_write_b32 a133, v85\n" \
    "v_accvgpr_write_b32 a134, v86\n" \
    "v_accvgpr_write_b32 a135, v87\n" \
    "v_mfma_f32_16x16x128_f8f6f4 v[212:215], a[64:71], a[184:191], v[212:215] cbsz:1 blgp:1\n" \
    "v_accvgpr_write_b32 a136, v88\n" \
    "v_accvgpr_write_b32 a137, v89\n" \
    "v_accvgpr_write_b32 a138, v90\n" \
    "v_accvgpr_write_b32 a139, v91\n" \
    "v_mfma_f32_16x16x128_f8f6f4 v[216:219], a[64:71], a[192:199], v[216:219] cbsz:1 blgp:1\n" \
    "v_accvgpr_write_b32 a140, v92\n" \
    "v_accvgpr_write_b32 a141, v93\n" \
    "v_accvgpr_write_b32 a142, v94\n" \
    "v_accvgpr_write_b32 a143, v95\n" \
    "v_mfma_f32_16x16x128_f8f6f4 v[220:223], a[64:71], a[200:207], v[220:223] cbsz:1 blgp:1\n" \
    "s_mov_b32 s88, 7\n" \
    "s_mov_b32 s89, 0\n" \
    "s_add_u32 s73, s73, 1\n" \
    "s_mov_b32 s75, 1\n" \
    "s_cmp_eq_u32 s74, 0\n" \
    "s_waitcnt vmcnt(0)\n" \
    "s_cbranch_scc1 a2a_stall\n" \
    "s_branch a2a_c1\n" \
    "a2a_u1_absent:\n" \
    "global_load_dword v10, v15, s[38:39] sc0 sc1\n" \
    "s_cmp_eq_u32 s70, 7\n" \
    "s_cselect_b32 s55, s67, 0\n" \
    "s_cmp_eq_u32 s55, 0\n" \
    "s_cbranch_scc1 a2a_u1_abn\n" \
    "s_andn2_b32 s55, s67, s68\n" \
    "s_cselect_b32 s55, s55, s67\n" \
    "s_and_b32 s56, s67, s66\n" \
    "s_cselect_b32 s55, s56, s55\n" \
    "s_or_b32 s55, s55, 0x80\n" \
    "s_ff1_i32_b32 s79, s55\n" \
    "s_lshl_b32 s77, s79, 2\n" \
    "s_lshr_b32 s77, s61, s77\n" \
    "s_and_b32 s77, s77, 7\n" \
    "s_lshl_b32 s77, s77, 21\n" \
    "s_add_u32 s77, s77, s53\n" \
    "s_bitset0_b32 s67, s79\n" \
    "s_mov_b32 s70, s79\n" \
    "s_xor_b32 s76, s72, s54\n" \
    "v_mfma_f32_16x16x128_f8f6f4 v[112:115], a[40:47], a[176:183], v[112:115] cbsz:1 blgp:1\n" \
    "s_xor_b32 m0, s76, 0\n" \
    "s_xor_b32 s58, s77, 0\n" \
    "buffer_load_dwordx4 v4, s[8:11], s58 offen offset:0 lds\n" \
    "s_add_u32 s58, s58, 31744\n" \
    "buffer_load_dwordx4 v4, s[8:11], s58 offen offset:1024 lds\n" \
    "v_mfma_f32_16x16x128_f8f6f4 v[116:119], a[40:47], a[184:191], v[116:119] cbsz:1 blgp:1\n" \
    "s_add_u32 s58, s58, 31744\n" \
    "buffer_load_dwordx4 v5, s[8:11], s58 offen offset:2048 lds\n" \
    "s_add_u32 s58, s58, 31744\n" \
    "buffer_load_dwordx4 v5, s[8:11], s58 offen offset:3072 lds\n" \
    "v_mfma_f32_16x16x128_f8f6f4 v[120:123], a[40:47], a[192:199], v[120:123] cbsz:1 blgp:1\n" \
    "s_xor_b32 m0, s76, 4096\n" \
    "s_xor_b32 s58, s77, 131072\n" \
    "buffer_load_dwordx4 v4, s[8:11], s58 offen offset:0 lds\n" \
    "s_add_u32 s58, s58, 31744\n" \
    "buffer_load_dwordx4 v4, s[8:11], s58 offen offset:1024 lds\n" \
    "v_mfma_f32_16x16x128_f8f6f4 v[124:127], a[40:47], a[200:207], v[124:127] cbsz:1 blgp:1\n" \
    "s_add_u32 s58, s58, 31744\n" \
    "buffer_load_dwordx4 v5, s[8:11], s58 offen offset:2048 lds\n" \
    "s_add_u32 s58, s58, 31744\n" \
    "buffer_load_dwordx4 v5, s[8:11], s58 offen offset:3072 lds\n" \
    "v_mfma_f32_16x16x128_f8f6f4 v[128:131], a[48:55], a[144:151], v[128:131] cbsz:1 blgp:1\n" \
    "s_xor_b32 m0, s76, 8192\n" \
    "s_xor_b32 s58, s77, 262144\n" \
    "buffer_load_dwordx4 v4, s[8:11], s58 offen offset:0 lds\n" \
    "s_add_u32 s58, s58, 31744\n" \
    "buffer_load_dwordx4 v4, s[8:11], s58 offen offset:1024 lds\n" \
    "v_mfma_f32_16x16x128_f8f6f4 v[132:135], a[48:55], a[152:159], v[132:135] cbsz:1 blgp:1\n" \
    "s_add_u32 s58, s58, 31744\n" \
    "buffer_load_dwordx4 v5, s[8:11], s58 offen offset:2048 lds\n" \
    "s_add_u32 s58, s58, 31744\n" \
    "buffer_load_dwordx4 v5, s[8:11], s58 offen offset:3072 lds\n" \
    "v_mfma_f32_16x16x128_f8f6f4 v[136:139], a[48:55], a[160:167], v[136:139] cbsz:1 blgp:1\n" \
    "s_xor_b32 m0, s76, 12288\n" \
    "s_xor_b32 s58, s77, 393216\n" \
    "buffer_load_dwordx4 v4, s[8:11], s58 offen offset:0 lds\n" \
    "s_add_u32 s58, s58, 31744\n" \
    "buffer_load_dwordx4 v4, s[8:11], s58 offen offset:1024 lds\n" \
    "v_mfma_f32_16x16x128_f8f6f4 v[140:143], a[48:55], a[168:175], v[140:143] cbsz:1 blgp:1\n" \
    "s_add_u32 s58, s58, 31744\n" \
    "buffer_load_dwordx4 v5, s[8:11], s58 offen offset:2048 lds\n" \
    "s_add_u32 s58, s58, 31744\n" \
    "buffer_load_dwordx4 v5, s[8:11], s58 offen offset:3072 lds\n" \
    "s_waitcnt vmcnt(16)\n" \
    "s_branch a2a_u1_abj\n" \
    "a2a_u1_abn:\n" \
    "v_mfma_f32_16x16x128_f8f6f4 v[112:115], a[40:47], a[176:183], v[112:115] cbsz:1 blgp:1\n" \
    "v_mfma_f32_16x16x128_f8f6f4 v[116:119], a[40:47], a[184:191], v[116:119] cbsz:1 blgp:1\n" \
    "v_mfma_f32_16x16x128_f8f6f4 v[120:123], a[40:47], a[192:199], v[120:123] cbsz:1 blgp:1\n" \
    "v_mfma_f32_16x16x128_f8f6f4 v[124:127], a[40:47], a[200:207], v[124:127] cbsz:1 blgp:1\n" \
    "v_mfma_f32_16x16x128_f8f6f4 v[128:131], a[48:55], a[144:151], v[128:131] cbsz:1 blgp:1\n" \
    "v_mfma_f32_16x16x128_f8f6f4 v[132:135], a[48:55], a[152:159], v[132:135] cbsz:1 blgp:1\n" \
    "v_mfma_f32_16x16x128_f8f6f4 v[136:139], a[48:55], a[160:167], v[136:139] cbsz:1 blgp:1\n" \
    "v_mfma_f32_16x16x128_f8f6f4 v[140:143], a[48:55], a[168:175], v[140:143] cbsz:1 blgp:1\n" \
    "s_waitcnt vmcnt(0)\n" \
    "a2a_u1_abj:\n" \
    "v_mfma_f32_16x16x128_f8f6f4 v[144:147], a[48:55], a[176:183], v[144:147] cbsz:1 blgp:1\n" \
    "v_cmp_le_u32 vcc, s59, v10\n" \
    "s_and_b32 s55, vcc_lo, 0x7f\n" \
    "s_or_b32 s66, s66, s55\n" \
    "s_lshr_b32 s55, s66, s70\n" \
    "s_lshr_b32 s56, s66, s69\n" \
    "s_andn2_b32 s55, s55, s56\n" \
    "s_bitcmp1_b32 s55, 0\n" \
    "s_cselect_b32 s71, s70, s69\n" \
    "v_mfma_f32_16x16x128_f8f6f4 v[148:151], a[48:55], a[184:191], v[148:151] cbsz:1 blgp:1\n" \
    "s_cselect_b32 s70, s69, s70\n" \
    "s_cselect_b32 s57, 16384, 0\n" \
    "s_mov_b32 s69, s71\n" \
    "s_xor_b32 s72, s72, s57\n" \
    "s_lshr_b32 s55, s66, s71\n" \
    "s_cmp_ge_u32 s73, s44\n" \
    "s_cselect_b32 s55, 0, s55\n" \
    "s_lshr_b32 s56, s66, s88\n" \
    "s_andn2_b32 s89, s56, s55\n" \
    "s_cmp_eq_u32 s69, 7\n" \
    "s_cselect_b32 s89, 1, s89\n" \
    "s_and_b32 s89, s89, 1\n" \
    "s_cselect_b32 s71, s88, s71\n" \
    "s_lshl_b32 s63, s71, 2\n" \
    "s_lshr_b32 s63, s61, s63\n" \
    "s_and_b32 s63, s63, 7\n" \
    "s_lshl_b32 s63, s63, 18\n" \
    "s_add_u32 s63, s63, s62\n" \
    "s_add_u32 s64, s63, 4096\n" \
    "s_add_u32 s65, s63, 8192\n" \
    "v_add_u32 v12, s72, v13\n" \
    "s_lshr_b32 s74, s66, s71\n" \
    "s_and_b32 s74, s74, 1\n" \
    "s_cbranch_scc0 a2a_u1_abx\n" \
    "buffer_load_dwordx4 a[0:3], v14, s[16:19], s63 offen offset:0 sc0 sc1\n" \
    "buffer_load_dwordx4 a[4:7], v14, s[16:19], s63 offen offset:1024 sc0 sc1\n" \
    "buffer_load_dwordx4 a[8:11], v14, s[16:19], s63 offen offset:2048 sc0 sc1\n" \
    "buffer_load_dwordx4 a[12:15], v14, s[16:19], s63 offen offset:3072 sc0 sc1\n" \
    "buffer_load_dwordx4 a[16:19], v14, s[16:19], s64 offen offset:0 sc0 sc1\n" \
    "buffer_load_dwordx4 a[20:23], v14, s[16:19], s64 offen offset:1024 sc0 sc1\n" \
    "buffer_load_dwordx4 a[24:27], v14, s[16:19], s64 offen offset:2048 sc0 sc1\n" \
    "buffer_load_dwordx4 a[28:31], v14, s[16:19], s64 offen offset:3072 sc0 sc1\n" \
    "s_cmp_eq_u32 s80, 0\n" \
    "s_cbranch_scc1 a2a_u1_b5\n" \
    "v_mfma_f32_16x16x128_f8f6f4 v[224:227], a[72:79], a[144:151], v[224:227] cbsz:1 blgp:1\n" \
    "buffer_load_dwordx4 a[32:35], v14, s[16:19], s65 offen offset:0 sc0 sc1\n" \
    "v_mfma_f32_16x16x128_f8f6f4 v[228:231], a[72:79], a[152:159], v[228:231] cbsz:1 blgp:1\n" \
    "buffer_load_dwordx4 a[36:39], v14, s[16:19], s65 offen offset:1024 sc0 sc1\n" \
    "v_mfma_f32_16x16x128_f8f6f4 v[232:235], a[72:79], a[160:167], v[232:235] cbsz:1 blgp:1\n" \
    "v_mfma_f32_16x16x128_f8f6f4 v[236:239], a[72:79], a[168:175], v[236:239] cbsz:1 blgp:1\n" \
    "v_mfma_f32_16x16x128_f8f6f4 v[240:243], a[72:79], a[176:183], v[240:243] cbsz:1 blgp:1\n" \
    "v_mfma_f32_16x16x128_f8f6f4 v[244:247], a[72:79], a[184:191], v[244:247] cbsz:1 blgp:1\n" \
    "v_mfma_f32_16x16x128_f8f6f4 v[248:251], a[72:79], a[192:199], v[248:251] cbsz:1 blgp:1\n" \
    "v_mfma_f32_16x16x128_f8f6f4 v[252:255], a[72:79], a[200:207], v[252:255] cbsz:1 blgp:1\n" \
    "a2a_u1_b5:\n" \
    "s_waitcnt vmcnt(8)\n" \
    "s_branch a2a_u1_h2n\n" \
    "a2a_u1_abx:\n" \
    "global_load_dword v10, v15, s[38:39] sc0 sc1\n" \
    "s_cmp_eq_u32 s80, 0\n" \
    "s_cbranch_scc1 a2a_u1_c5\n" \
    "v_mfma_f32_16x16x128_f8f6f4 v[224:227], a[72:79], a[144:151], v[224:227] cbsz:1 blgp:1\n" \
    "v_mfma_f32_16x16x128_f8f6f4 v[228:231], a[72:79], a[152:159], v[228:231] cbsz:1 blgp:1\n" \
    "v_mfma_f32_16x16x128_f8f6f4 v[232:235], a[72:79], a[160:167], v[232:235] cbsz:1 blgp:1\n" \
    "v_mfma_f32_16x16x128_f8f6f4 v[236:239], a[72:79], a[168:175], v[236:239] cbsz:1 blgp:1\n" \
    "v_mfma_f32_16x16x128_f8f6f4 v[240:243], a[72:79], a[176:183], v[240:243] cbsz:1 blgp:1\n" \
    "v_mfma_f32_16x16x128_f8f6f4 v[244:247], a[72:79], a[184:191], v[244:247] cbsz:1 blgp:1\n" \
    "v_mfma_f32_16x16x128_f8f6f4 v[248:251], a[72:79], a[192:199], v[248:251] cbsz:1 blgp:1\n" \
    "v_mfma_f32_16x16x128_f8f6f4 v[252:255], a[72:79], a[200:207], v[252:255] cbsz:1 blgp:1\n" \
    "a2a_u1_c5:\n" \
    "s_waitcnt vmcnt(0)\n" \
    "v_cmp_le_u32 vcc, s59, v10\n" \
    "s_and_b32 s55, vcc_lo, 0x7f\n" \
    "s_or_b32 s66, s66, s55\n" \
    "s_lshr_b32 s55, s66, s70\n" \
    "s_lshr_b32 s56, s66, s69\n" \
    "s_andn2_b32 s55, s55, s56\n" \
    "s_bitcmp1_b32 s55, 0\n" \
    "s_cselect_b32 s71, s70, s69\n" \
    "s_cselect_b32 s70, s69, s70\n" \
    "s_cselect_b32 s57, 16384, 0\n" \
    "s_mov_b32 s69, s71\n" \
    "s_xor_b32 s72, s72, s57\n" \
    "s_lshr_b32 s55, s66, s71\n" \
    "s_cmp_ge_u32 s73, s44\n" \
    "s_cselect_b32 s55, 0, s55\n" \
    "s_lshr_b32 s56, s66, s88\n" \
    "s_andn2_b32 s89, s56, s55\n" \
    "s_cmp_eq_u32 s69, 7\n" \
    "s_cselect_b32 s89, 1, s89\n" \
    "s_and_b32 s89, s89, 1\n" \
    "s_cselect_b32 s71, s88, s71\n" \
    "s_lshl_b32 s63, s71, 2\n" \
    "s_lshr_b32 s63, s61, s63\n" \
    "s_and_b32 s63, s63, 7\n" \
    "s_lshl_b32 s63, s63, 18\n" \
    "s_add_u32 s63, s63, s62\n" \
    "s_add_u32 s64, s63, 4096\n" \
    "s_add_u32 s65, s63, 8192\n" \
    "v_add_u32 v12, s72, v13\n" \
    "s_lshr_b32 s74, s66, s71\n" \
    "s_and_b32 s74, s74, 1\n" \
    "s_cbranch_scc0 a2a_u1_h2n\n" \
    "buffer_load_dwordx4 a[0:3], v14, s[16:19], s63 offen offset:0 sc0 sc1\n" \
    "buffer_load_dwordx4 a[4:7], v14, s[16:19], s63 offen offset:1024 sc0 sc1\n" \
    "buffer_load_dwordx4 a[8:11], v14, s[16:19], s63 offen offset:2048 sc0 sc1\n" \
    "buffer_load_dwordx4 a[12:15], v14, s[16:19], s63 offen offset:3072 sc0 sc1\n" \
    "buffer_load_dwordx4 a[16:19], v14, s[16:19], s64 offen offset:0 sc0 sc1\n" \
    "buffer_load_dwordx4 a[20:23], v14, s[16:19], s64 offen offset:1024 sc0 sc1\n" \
    "buffer_load_dwordx4 a[24:27], v14, s[16:19], s64 offen offset:2048 sc0 sc1\n" \
    "buffer_load_dwordx4 a[28:31], v14, s[16:19], s64 offen offset:3072 sc0 sc1\n" \
    "s_cmp_eq_u32 s80, 0\n" \
    "s_cbranch_scc1 a2a_u1_aby\n" \
    "buffer_load_dwordx4 a[32:35], v14, s[16:19], s65 offen offset:0 sc0 sc1\n" \
    "buffer_load_dwordx4 a[36:39], v14, s[16:19], s65 offen offset:1024 sc0 sc1\n" \
    "a2a_u1_aby:\n" \
    "s_branch a2a_u1_h2n\n" \
    "a2a_u0_alt4:\n" \
    "s_xor_b32 m0, s76, 0\n" \
    "s_xor_b32 s58, s77, 0\n" \
    "buffer_load_dwordx4 v4, s[8:11], s58 offen offset:0 lds\n" \
    "s_add_u32 s58, s58, 31744\n" \
    "buffer_load_dwordx4 v4, s[8:11], s58 offen offset:1024 lds\n" \
    "s_add_u32 s58, s58, 31744\n" \
    "buffer_load_dwordx4 v5, s[8:11], s58 offen offset:2048 lds\n" \
    "s_add_u32 s58, s58, 31744\n" \
    "buffer_load_dwordx4 v5, s[8:11], s58 offen offset:3072 lds\n" \
    "s_xor_b32 m0, s76, 4096\n" \
    "s_xor_b32 s58, s77, 131072\n" \
    "buffer_load_dwordx4 v4, s[8:11], s58 offen offset:0 lds\n" \
    "s_add_u32 s58, s58, 31744\n" \
    "buffer_load_dwordx4 v4, s[8:11], s58 offen offset:1024 lds\n" \
    "s_branch a2a_u0_a5\n" \
    "a2a_u0_nolw:\n" \
    "s_cmp_eq_u32 s80, 0\n" \
    "s_cbranch_scc1 a2a_u0_n5\n" \
    "v_mfma_f32_16x16x128_f8f6f4 v[224:227], a[32:39], a[80:87], v[224:227] cbsz:1 blgp:1\n" \
    "buffer_load_dwordx4 a[72:75], v14, s[16:19], s65 offen offset:0 sc0 sc1\n" \
    "v_mfma_f32_16x16x128_f8f6f4 v[228:231], a[32:39], a[88:95], v[228:231] cbsz:1 blgp:1\n" \
    "buffer_load_dwordx4 a[76:79], v14, s[16:19], s65 offen offset:1024 sc0 sc1\n" \
    "v_mfma_f32_16x16x128_f8f6f4 v[232:235], a[32:39], a[96:103], v[232:235] cbsz:1 blgp:1\n" \
    "v_mfma_f32_16x16x128_f8f6f4 v[236:239], a[32:39], a[104:111], v[236:239] cbsz:1 blgp:1\n" \
    "v_mfma_f32_16x16x128_f8f6f4 v[240:243], a[32:39], a[112:119], v[240:243] cbsz:1 blgp:1\n" \
    "v_mfma_f32_16x16x128_f8f6f4 v[244:247], a[32:39], a[120:127], v[244:247] cbsz:1 blgp:1\n" \
    "v_mfma_f32_16x16x128_f8f6f4 v[248:251], a[32:39], a[128:135], v[248:251] cbsz:1 blgp:1\n" \
    "v_mfma_f32_16x16x128_f8f6f4 v[252:255], a[32:39], a[136:143], v[252:255] cbsz:1 blgp:1\n" \
    "a2a_u0_n5:\n" \
    "v_mfma_f32_16x16x128_f8f6f4 v[144:147], a[8:15], a[112:119], v[144:147] cbsz:1 blgp:1\n" \
    "v_mfma_f32_16x16x128_f8f6f4 v[148:151], a[8:15], a[120:127], v[148:151] cbsz:1 blgp:1\n" \
    "s_waitcnt vmcnt(8)\n" \
    "a2a_u0_h2n:\n" \
    "v_mfma_f32_16x16x128_f8f6f4 v[152:155], a[8:15], a[128:135], v[152:155] cbsz:1 blgp:1\n" \
    "global_load_dword v10, v15, s[38:39] sc0 sc1\n" \
    "s_cmp_lg_u32 s89, 0\n" \
    "s_cbranch_scc1 a2a_u0_h2q\n" \
    "v_mfma_f32_16x16x128_f8f6f4 v[156:159], a[8:15], a[136:143], v[156:159] cbsz:1 blgp:1\n" \
    "v_xor_b32 v11, 0, v12\n" \
    "ds_read_b64_tr_b8 a[144:145], v11\n" \
    "ds_read_b64_tr_b8 a[146:147], v11 offset:1024\n" \
    "v_mfma_f32_16x16x128_f8f6f4 v[160:163], a[16:23], a[80:87], v[160:163] cbsz:1 blgp:1\n" \
    "ds_read_b64_tr_b8 a[148:149], v11 offset:8192\n" \
    "ds_read_b64_tr_b8 a[150:151], v11 offset:9216\n" \
    "v_mfma_f32_16x16x128_f8f6f4 v[164:167], a[16:23], a[88:95], v[164:167] cbsz:1 blgp:1\n" \
    "v_xor_b32 v11, 16, v12\n" \
    "ds_read_b64_tr_b8 a[152:153], v11\n" \
    "ds_read_b64_tr_b8 a[154:155], v11 offset:1024\n" \
    "v_mfma_f32_16x16x128_f8f6f4 v[168:171], a[16:23], a[96:103], v[168:171] cbsz:1 blgp:1\n" \
    "ds_read_b64_tr_b8 a[156:157], v11 offset:8192\n" \
    "ds_read_b64_tr_b8 a[158:159], v11 offset:9216\n" \
    "v_mfma_f32_16x16x128_f8f6f4 v[172:175], a[16:23], a[104:111], v[172:175] cbsz:1 blgp:1\n" \
    "v_xor_b32 v11, 32, v12\n" \
    "ds_read_b64_tr_b8 a[160:161], v11\n" \
    "ds_read_b64_tr_b8 a[162:163], v11 offset:1024\n" \
    "v_mfma_f32_16x16x128_f8f6f4 v[176:179], a[16:23], a[112:119], v[176:179] cbsz:1 blgp:1\n" \
    "ds_read_b64_tr_b8 a[164:165], v11 offset:8192\n" \
    "ds_read_b64_tr_b8 a[166:167], v11 offset:9216\n" \
    "v_mfma_f32_16x16x128_f8f6f4 v[180:183], a[16:23], a[120:127], v[180:183] cbsz:1 blgp:1\n" \
    "v_xor_b32 v11, 48, v12\n" \
    "ds_read_b64_tr_b8 a[168:169], v11\n" \
    "ds_read_b64_tr_b8 a[170:171], v11 offset:1024\n" \
    "v_mfma_f32_16x16x128_f8f6f4 v[184:187], a[16:23], a[128:135], v[184:187] cbsz:1 blgp:1\n" \
    "ds_read_b64_tr_b8 a[172:173], v11 offset:8192\n" \
    "ds_read_b64_tr_b8 a[174:175], v11 offset:9216\n" \
    "v_mfma_f32_16x16x128_f8f6f4 v[188:191], a[16:23], a[136:143], v[188:191] cbsz:1 blgp:1\n" \
    "v_xor_b32 v11, 64, v12\n" \
    "ds_read_b64_tr_b8 a[176:177], v11\n" \
    "ds_read_b64_tr_b8 a[178:179], v11 offset:1024\n" \
    "v_mfma_f32_16x16x128_f8f6f4 v[192:195], a[24:31], a[80:87], v[192:195] cbsz:1 blgp:1\n" \
    "ds_read_b64_tr_b8 a[180:181], v11 offset:8192\n" \
    "ds_read_b64_tr_b8 a[182:183], v11 offset:9216\n" \
    "v_mfma_f32_16x16x128_f8f6f4 v[196:199], a[24:31], a[88:95], v[196:199] cbsz:1 blgp:1\n" \
    "v_xor_b32 v11, 80, v12\n" \
    "ds_read_b64_tr_b8 a[184:185], v11\n" \
    "ds_read_b64_tr_b8 a[186:187], v11 offset:1024\n" \
    "v_mfma_f32_16x16x128_f8f6f4 v[200:203], a[24:31], a[96:103], v[200:203] cbsz:1 blgp:1\n" \
    "ds_read_b64_tr_b8 a[188:189], v11 offset:8192\n" \
    "ds_read_b64_tr_b8 a[190:191], v11 offset:9216\n" \
    "v_mfma_f32_16x16x128_f8f6f4 v[204:207], a[24:31], a[104:111], v[204:207] cbsz:1 blgp:1\n" \
    "v_xor_b32 v11, 96, v12\n" \
    "ds_read_b64_tr_b8 a[192:193], v11\n" \
    "ds_read_b64_tr_b8 a[194:195], v11 offset:1024\n" \
    "v_mfma_f32_16x16x128_f8f6f4 v[208:211], a[24:31], a[112:119], v[208:211] cbsz:1 blgp:1\n" \
    "ds_read_b64_tr_b8 a[196:197], v11 offset:8192\n" \
    "ds_read_b64_tr_b8 a[198:199], v11 offset:9216\n" \
    "v_mfma_f32_16x16x128_f8f6f4 v[212:215], a[24:31], a[120:127], v[212:215] cbsz:1 blgp:1\n" \
    "v_xor_b32 v11, 112, v12\n" \
    "ds_read_b64_tr_b8 a[200:201], v11\n" \
    "ds_read_b64_tr_b8 a[202:203], v11 offset:1024\n" \
    "v_mfma_f32_16x16x128_f8f6f4 v[216:219], a[24:31], a[128:135], v[216:219] cbsz:1 blgp:1\n" \
    "ds_read_b64_tr_b8 a[204:205], v11 offset:8192\n" \
    "ds_read_b64_tr_b8 a[206:207], v11 offset:9216\n" \
    "v_mfma_f32_16x16x128_f8f6f4 v[220:223], a[24:31], a[136:143], v[220:223] cbsz:1 blgp:1\n" \
    "s_mov_b32 s69, s70\n" \
    "s_mov_b32 s70, 7\n" \
    "s_xor_b32 s72, s72, 16384\n" \
    "s_add_u32 s73, s73, 1\n" \
    "s_mov_b32 s75, 0\n" \
    "s_cmp_eq_u32 s74, 0\n" \
    "s_waitcnt vmcnt(0) lgkmcnt(0)\n" \
    "s_cbranch_scc1 a2a_stall\n" \
    "s_branch a2a_c0\n" \
    "a2a_u0_pk:\n" \
    "s_lshr_b32 s55, s66, s71\n" \
    "s_cmp_ge_u32 s73, s44\n" \
    "s_cselect_b32 s55, 0, s55\n" \
    "s_lshr_b32 s56, s66, s88\n" \
    "s_andn2_b32 s89, s56, s55\n" \
    "s_cmp_eq_u32 s69, 7\n" \
    "s_cselect_b32 s89, 1, s89\n" \
    "s_and_b32 s89, s89, 1\n" \
    "s_cselect_b32 s71, s88, s71\n" \
    "s_branch a2a_u0_pkr\n" \
    "a2a_u0_h2p:\n" \
    "v_mfma_f32_16x16x128_f8f6f4 v[156:159], a[8:15], a[136:143], v[156:159] cbsz:1 blgp:1\n" \
    "v_accvgpr_write_b32 a144, v32\n" \
    "v_accvgpr_write_b32 a145, v33\n" \
    "v_accvgpr_write_b32 a146, v34\n" \
    "v_accvgpr_write_b32 a147, v35\n" \
    "v_mfma_f32_16x16x128_f8f6f4 v[160:163], a[16:23], a[80:87], v[160:163] cbsz:1 blgp:1\n" \
    "s_xor_b32 m0, s76, 8192\n" \
    "s_xor_b32 s58, s77, 262144\n" \
    "buffer_load_dwordx4 v4, s[8:11], s58 offen offset:0 lds\n" \
    "v_accvgpr_write_b32 a148, v36\n" \
    "v_accvgpr_write_b32 a149, v37\n" \
    "v_accvgpr_write_b32 a150, v38\n" \
    "v_accvgpr_write_b32 a151, v39\n" \
    "v_mfma_f32_16x16x128_f8f6f4 v[164:167], a[16:23], a[88:95], v[164:167] cbsz:1 blgp:1\n" \
    "v_accvgpr_write_b32 a152, v40\n" \
    "v_accvgpr_write_b32 a153, v41\n" \
    "v_accvgpr_write_b32 a154, v42\n" \
    "v_accvgpr_write_b32 a155, v43\n" \
    "v_mfma_f32_16x16x128_f8f6f4 v[168:171], a[16:23], a[96:103], v[168:171] cbsz:1 blgp:1\n" \
    "s_add_u32 s58, s58, 31744\n" \
    "buffer_load_dwordx4 v4, s[8:11], s58 offen offset:1024 lds\n" \
    "v_accvgpr_write_b32 a156, v44\n" \
    "v_accvgpr_write_b32 a157, v45\n" \
    "v_accvgpr_write_b32 a158, v46\n" \
    "v_accvgpr_write_b32 a159, v47\n" \
    "v_mfma_f32_16x16x128_f8f6f4 v[172:175], a[16:23], a[104:111], v[172:175] cbsz:1 blgp:1\n" \
    "v_accvgpr_write_b32 a160, v48\n" \
    "v_accvgpr_write_b32 a161, v49\n" \
    "v_accvgpr_write_b32 a162, v50\n" \
    "v_accvgpr_write_b32 a163, v51\n" \
    "v_mfma_f32_16x16x128_f8f6f4 v[176:179], a[16:23], a[112:119], v[176:179] cbsz:1 blgp:1\n" \
    "s_add_u32 s58, s58, 31744\n" \
    "buffer_load_dwordx4 v5, s[8:11], s58 offen offset:2048 lds\n" \
    "v_accvgpr_write_b32 a164, v52\n" \
    "v_accvgpr_write_b32 a165, v53\n" \
    "v_accvgpr_write_b32 a166, v54\n" \
    "v_accvgpr_write_b32 a167, v55\n" \
    "v_mfma_f32_16x16x128_f8f6f4 v[180:183], a[16:23], a[120:127], v[180:183] cbsz:1 blgp:1\n" \
    "v_accvgpr_write_b32 a168, v56\n" \
    "v_accvgpr_write_b32 a169, v57\n" \
    "v_accvgpr_write_b32 a170, v58\n" \
    "v_accvgpr_write_b32 a171, v59\n" \
    "v_mfma_f32_16x16x128_f8f6f4 v[184:187], a[16:23], a[128:135], v[184:187] cbsz:1 blgp:1\n" \
    "s_add_u32 s58, s58, 31744\n" \
    "buffer_load_dwordx4 v5, s[8:11], s58 offen offset:3072 lds\n" \
    "v_accvgpr_write_b32 a172, v60\n" \
    "v_accvgpr_write_b32 a173, v61\n" \
    "v_accvgpr_write_b32 a174, v62\n" \
    "v_accvgpr_write_b32 a175, v63\n" \
    "v_mfma_f32_16x16x128_f8f6f4 v[188:191], a[16:23], a[136:143], v[188:191] cbsz:1 blgp:1\n" \
    "v_accvgpr_write_b32 a176, v64\n" \
    "v_accvgpr_write_b32 a177, v65\n" \
    "v_accvgpr_write_b32 a178, v66\n" \
    "v_accvgpr_write_b32 a179, v67\n" \
    "v_mfma_f32_16x16x128_f8f6f4 v[192:195], a[24:31], a[80:87], v[192:195] cbsz:1 blgp:1\n" \
    "s_xor_b32 m0, s76, 12288\n" \
    "s_xor_b32 s58, s77, 393216\n" \
    "buffer_load_dwordx4 v4, s[8:11], s58 offen offset:0 lds\n" \
    "v_accvgpr_write_b32 a180, v68\n" \
    "v_accvgpr_write_b32 a181, v69\n" \
    "v_accvgpr_write_b32 a182, v70\n" \
    "v_accvgpr_write_b32 a183, v71\n" \
    "v_mfma_f32_16x16x128_f8f6f4 v[196:199], a[24:31], a[88:95], v[196:199] cbsz:1 blgp:1\n" \
    "v_accvgpr_write_b32 a184, v72\n" \
    "v_accvgpr_write_b32 a185, v73\n" \
    "v_accvgpr_write_b32 a186, v74\n" \
    "v_accvgpr_write_b32 a187, v75\n" \
    "v_mfma_f32_16x16x128_f8f6f4 v[200:203], a[24:31], a[96:103], v[200:203] cbsz:1 blgp:1\n" \
    "s_add_u32 s58, s58, 31744\n" \
    "buffer_load_dwordx4 v4, s[8:11], s58 offen offset:1024 lds\n" \
    "v_accvgpr_write_b32 a188, v76\n" \
    "v_accvgpr_write_b32 a189, v77\n" \
    "v_accvgpr_write_b32 a190, v78\n" \
    "v_accvgpr_write_b32 a191, v79\n" \
    "v_mfma_f32_16x16x128_f8f6f4 v[204:207], a[24:31], a[104:111], v[204:207] cbsz:1 blgp:1\n" \
    "v_accvgpr_write_b32 a192, v80\n" \
    "v_accvgpr_write_b32 a193, v81\n" \
    "v_accvgpr_write_b32 a194, v82\n" \
    "v_accvgpr_write_b32 a195, v83\n" \
    "v_mfma_f32_16x16x128_f8f6f4 v[208:211], a[24:31], a[112:119], v[208:211] cbsz:1 blgp:1\n" \
    "s_add_u32 s58, s58, 31744\n" \
    "buffer_load_dwordx4 v5, s[8:11], s58 offen offset:2048 lds\n" \
    "v_accvgpr_write_b32 a196, v84\n" \
    "v_accvgpr_write_b32 a197, v85\n" \
    "v_accvgpr_write_b32 a198, v86\n" \
    "v_accvgpr_write_b32 a199, v87\n" \
    "v_mfma_f32_16x16x128_f8f6f4 v[212:215], a[24:31], a[120:127], v[212:215] cbsz:1 blgp:1\n" \
    "v_accvgpr_write_b32 a200, v88\n" \
    "v_accvgpr_write_b32 a201, v89\n" \
    "v_accvgpr_write_b32 a202, v90\n" \
    "v_accvgpr_write_b32 a203, v91\n" \
    "v_mfma_f32_16x16x128_f8f6f4 v[216:219], a[24:31], a[128:135], v[216:219] cbsz:1 blgp:1\n" \
    "s_add_u32 s58, s58, 31744\n" \
    "buffer_load_dwordx4 v5, s[8:11], s58 offen offset:3072 lds\n" \
    "v_accvgpr_write_b32 a204, v92\n" \
    "v_accvgpr_write_b32 a205, v93\n" \
    "v_accvgpr_write_b32 a206, v94\n" \
    "v_accvgpr_write_b32 a207, v95\n" \
    "v_mfma_f32_16x16x128_f8f6f4 v[220:223], a[24:31], a[136:143], v[220:223] cbsz:1 blgp:1\n" \
    "s_mov_b32 s88, 7\n" \
    "s_mov_b32 s89, 0\n" \
    "s_add_u32 s73, s73, 1\n" \
    "s_waitcnt vmcnt(17)\n" \
    "s_branch a2a_c0\n" \
    "a2a_u0_h2q:\n" \
    "v_mfma_f32_16x16x128_f8f6f4 v[156:159], a[8:15], a[136:143], v[156:159] cbsz:1 blgp:1\n" \
    "v_accvgpr_write_b32 a144, v32\n" \
    "v_accvgpr_write_b32 a145, v33\n" \
    "v_accvgpr_write_b32 a146, v34\n" \
    "v_accvgpr_write_b32 a147, v35\n" \
    "v_mfma_f32_16x16x128_f8f6f4 v[160:163], a[16:23], a[80:87], v[160:163] cbsz:1 blgp:1\n" \
    "v_accvgpr_write_b32 a148, v36\n" \
    "v_accvgpr_write_b32 a149, v37\n" \
    "v_accvgpr_write_b32 a150, v38\n" \
    "v_accvgpr_write_b32 a151, v39\n" \
    "v_mfma_f32_16x16x128_f8f6f4 v[164:167], a[16:23], a[88:95], v[164:167] cbsz:1 blgp:1\n" \
    "v_accvgpr_write_b32 a152, v40\n" \
    "v_accvgpr_write_b32 a153, v41\n" \
    "v_accvgpr_write_b32 a154, v42\n" \
    "v_accvgpr_write_b32 a155, v43\n" \
    "v_mfma_f32_16x16x128_f8f6f4 v[168:171], a[16:23], a[96:103], v[168:171] cbsz:1 blgp:1\n" \
    "v_accvgpr_write_b32 a156, v44\n" \
    "v_accvgpr_write_b32 a157, v45\n" \
    "v_accvgpr_write_b32 a158, v46\n" \
    "v_accvgpr_write_b32 a159, v47\n" \
    "v_mfma_f32_16x16x128_f8f6f4 v[172:175], a[16:23], a[104:111], v[172:175] cbsz:1 blgp:1\n" \
    "v_accvgpr_write_b32 a160, v48\n" \
    "v_accvgpr_write_b32 a161, v49\n" \
    "v_accvgpr_write_b32 a162, v50\n" \
    "v_accvgpr_write_b32 a163, v51\n" \
    "v_mfma_f32_16x16x128_f8f6f4 v[176:179], a[16:23], a[112:119], v[176:179] cbsz:1 blgp:1\n" \
    "v_accvgpr_write_b32 a164, v52\n" \
    "v_accvgpr_write_b32 a165, v53\n" \
    "v_accvgpr_write_b32 a166, v54\n" \
    "v_accvgpr_write_b32 a167, v55\n" \
    "v_mfma_f32_16x16x128_f8f6f4 v[180:183], a[16:23], a[120:127], v[180:183] cbsz:1 blgp:1\n" \
    "v_accvgpr_write_b32 a168, v56\n" \
    "v_accvgpr_write_b32 a169, v57\n" \
    "v_accvgpr_write_b32 a170, v58\n" \
    "v_accvgpr_write_b32 a171, v59\n" \
    "v_mfma_f32_16x16x128_f8f6f4 v[184:187], a[16:23], a[128:135], v[184:187] cbsz:1 blgp:1\n" \
    "v_accvgpr_write_b32 a172, v60\n" \
    "v_accvgpr_write_b32 a173, v61\n" \
    "v_accvgpr_write_b32 a174, v62\n" \
    "v_accvgpr_write_b32 a175, v63\n" \
    "v_mfma_f32_16x16x128_f8f6f4 v[188:191], a[16:23], a[136:143], v[188:191] cbsz:1 blgp:1\n" \
    "v_accvgpr_write_b32 a176, v64\n" \
    "v_accvgpr_write_b32 a177, v65\n" \
    "v_accvgpr_write_b32 a178, v66\n" \
    "v_accvgpr_write_b32 a179, v67\n" \
    "v_mfma_f32_16x16x128_f8f6f4 v[192:195], a[24:31], a[80:87], v[192:195] cbsz:1 blgp:1\n" \
    "v_accvgpr_write_b32 a180, v68\n" \
    "v_accvgpr_write_b32 a181, v69\n" \
    "v_accvgpr_write_b32 a182, v70\n" \
    "v_accvgpr_write_b32 a183, v71\n" \
    "v_mfma_f32_16x16x128_f8f6f4 v[196:199], a[24:31], a[88:95], v[196:199] cbsz:1 blgp:1\n" \
    "v_accvgpr_write_b32 a184, v72\n" \
    "v_accvgpr_write_b32 a185, v73\n" \
    "v_accvgpr_write_b32 a186, v74\n" \
    "v_accvgpr_write_b32 a187, v75\n" \
    "v_mfma_f32_16x16x128_f8f6f4 v[200:203], a[24:31], a[96:103], v[200:203] cbsz:1 blgp:1\n" \
    "v_accvgpr_write_b32 a188, v76\n" \
    "v_accvgpr_write_b32 a189, v77\n" \
    "v_accvgpr_write_b32 a190, v78\n" \
    "v_accvgpr_write_b32 a191, v79\n" \
    "v_mfma_f32_16x16x128_f8f6f4 v[204:207], a[24:31], a[104:111], v[204:207] cbsz:1 blgp:1\n" \
    "v_accvgpr_write_b32 a192, v80\n" \
    "v_accvgpr_write_b32 a193, v81\n" \
    "v_accvgpr_write_b32 a194, v82\n" \
    "v_accvgpr_write_b32 a195, v83\n" \
    "v_mfma_f32_16x16x128_f8f6f4 v[208:211], a[24:31], a[112:119], v[208:211] cbsz:1 blgp:1\n" \
    "v_accvgpr_write_b32 a196, v84\n" \
    "v_accvgpr_write_b32 a197, v85\n" \
    "v_accvgpr_write_b32 a198, v86\n" \
    "v_accvgpr_write_b32 a199, v87\n" \
    "v_mfma_f32_16x16x128_f8f6f4 v[212:215], a[24:31], a[120:127], v[212:215] cbsz:1 blgp:1\n" \
    "v_accvgpr_write_b32 a200, v88\n" \
    "v_accvgpr_write_b32 a201, v89\n" \
    "v_accvgpr_write_b32 a202, v90\n" \
    "v_accvgpr_write_b32 a203, v91\n" \
    "v_mfma_f32_16x16x128_f8f6f4 v[216:219], a[24:31], a[128:135], v[216:219] cbsz:1 blgp:1\n" \
    "v_accvgpr_write_b32 a204, v92\n" \
    "v_accvgpr_write_b32 a205, v93\n" \
    "v_accvgpr_write_b32 a206, v94\n" \
    "v_accvgpr_write_b32 a207, v95\n" \
    "v_mfma_f32_16x16x128_f8f6f4 v[220:223], a[24:31], a[136:143], v[220:223] cbsz:1 blgp:1\n" \
    "s_mov_b32 s88, 7\n" \
    "s_mov_b32 s89, 0\n" \
    "s_add_u32 s73, s73, 1\n" \
    "s_mov_b32 s75, 0\n" \
    "s_cmp_eq_u32 s74, 0\n" \
    "s_waitcnt vmcnt(0)\n" \
    "s_cbranch_scc1 a2a_stall\n" \
    "s_branch a2a_c0\n" \
    "a2a_u0_absent:\n" \
    "global_load_dword v10, v15, s[38:39] sc0 sc1\n" \
    "s_cmp_eq_u32 s70, 7\n" \
    "s_cselect_b32 s55, s67, 0\n" \
    "s_cmp_eq_u32 s55, 0\n" \
    "s_cbranch_scc1 a2a_u0_abn\n" \
    "s_andn2_b32 s55, s67, s68\n" \
    "s_cselect_b32 s55, s55, s67\n" \
    "s_and_b32 s56, s67, s66\n" \
    "s_cselect_b32 s55, s56, s55\n" \
    "s_or_b32 s55, s55, 0x80\n" \
    "s_ff1_i32_b32 s79, s55\n" \
    "s_lshl_b32 s77, s79, 2\n" \
    "s_lshr_b32 s77, s61, s77\n" \
    "s_and_b32 s77, s77, 7\n" \
    "s_lshl_b32 s77, s77, 21\n" \
    "s_add_u32 s77, s77, s53\n" \
    "s_bitset0_b32 s67, s79\n" \
    "s_mov_b32 s70, s79\n" \
    "s_xor_b32 s76, s72, s54\n" \
    "v_mfma_f32_16x16x128_f8f6f4 v[112:115], a[0:7], a[112:119], v[112:115] cbsz:1 blgp:1\n" \
    "s_xor_b32 m0, s76, 0\n" \
    "s_xor_b32 s58, s77, 0\n" \
    "buffer_load_dwordx4 v4, s[8:11], s58 offen offset:0 lds\n" \
    "s_add_u32 s58, s58, 31744\n" \
    "buffer_load_dwordx4 v4, s[8:11], s58 offen offset:1024 lds\n" \
    "v_mfma_f32_16x16x128_f8f6f4 v[116:119], a[0:7], a[120:127], v[116:119] cbsz:1 blgp:1\n" \
    "s_add_u32 s58, s58, 31744\n" \
    "buffer_load_dwordx4 v5, s[8:11], s58 offen offset:2048 lds\n" \
    "s_add_u32 s58, s58, 31744\n" \
    "buffer_load_dwordx4 v5, s[8:11], s58 offen offset:3072 lds\n" \
    "v_mfma_f32_16x16x128_f8f6f4 v[120:123], a[0:7], a[128:135], v[120:123] cbsz:1 blgp:1\n" \
    "s_xor_b32 m0, s76, 4096\n" \
    "s_xor_b32 s58, s77, 131072\n" \
    "buffer_load_dwordx4 v4, s[8:11], s58 offen offset:0 lds\n" \
    "s_add_u32 s58, s58, 31744\n" \
    "buffer_load_dwordx4 v4, s[8:11], s58 offen offset:1024 lds\n" \
    "v_mfma_f32_16x16x128_f8f6f4 v[124:127], a[0:7], a[136:143], v[124:127] cbsz:1 blgp:1\n" \
    "s_add_u32 s58, s58, 31744\n" \
    "buffer_load_dwordx4 v5, s[8:11], s58 offen offset:2048 lds\n" \
    "s_add_u32 s58, s58, 31744\n" \
    "buffer_load_dwordx4 v5, s[8:11], s58 offen offset:3072 lds\n" \
    "v_mfma_f32_16x16x128_f8f6f4 v[128:131], a[8:15], a[80:87], v[128:131] cbsz:1 blgp:1\n" \
    "s_xor_b32 m0, s76, 8192\n" \
    "s_xor_b32 s58, s77, 262144\n" \
    "buffer_load_dwordx4 v4, s[8:11], s58 offen offset:0 lds\n" \
    "s_add_u32 s58, s58, 31744\n" \
    "buffer_load_dwordx4 v4, s[8:11], s58 offen offset:1024 lds\n" \
    "v_mfma_f32_16x16x128_f8f6f4 v[132:135], a[8:15], a[88:95], v[132:135] cbsz:1 blgp:1\n" \
    "s_add_u32 s58, s58, 31744\n" \
    "buffer_load_dwordx4 v5, s[8:11], s58 offen offset:2048 lds\n" \
    "s_add_u32 s58, s58, 31744\n" \
    "buffer_load_dwordx4 v5, s[8:11], s58 offen offset:3072 lds\n" \
    "v_mfma_f32_16x16x128_f8f6f4 v[136:139], a[8:15], a[96:103], v[136:139] cbsz:1 blgp:1\n" \
    "s_xor_b32 m0, s76, 12288\n" \
    "s_xor_b32 s58, s77, 393216\n" \
    "buffer_load_dwordx4 v4, s[8:11], s58 offen offset:0 lds\n" \
    "s_add_u32 s58, s58, 31744\n" \
    "buffer_load_dwordx4 v4, s[8:11], s58 offen offset:1024 lds\n" \
    "v_mfma_f32_16x16x128_f8f6f4 v[140:143], a[8:15], a[104:111], v[140:143] cbsz:1 blgp:1\n" \
    "s_add_u32 s58, s58, 31744\n" \
    "buffer_load_dwordx4 v5, s[8:11], s58 offen offset:2048 lds\n" \
    "s_add_u32 s58, s58, 31744\n" \
    "buffer_load_dwordx4 v5, s[8:11], s58 offen offset:3072 lds\n" \
    "s_waitcnt vmcnt(16)\n" \
    "s_branch a2a_u0_abj\n" \
    "a2a_u0_abn:\n" \
    "v_mfma_f32_16x16x128_f8f6f4 v[112:115], a[0:7], a[112:119], v[112:115] cbsz:1 blgp:1\n" \
    "v_mfma_f32_16x16x128_f8f6f4 v[116:119], a[0:7], a[120:127], v[116:119] cbsz:1 blgp:1\n" \
    "v_mfma_f32_16x16x128_f8f6f4 v[120:123], a[0:7], a[128:135], v[120:123] cbsz:1 blgp:1\n" \
    "v_mfma_f32_16x16x128_f8f6f4 v[124:127], a[0:7], a[136:143], v[124:127] cbsz:1 blgp:1\n" \
    "v_mfma_f32_16x16x128_f8f6f4 v[128:131], a[8:15], a[80:87], v[128:131] cbsz:1 blgp:1\n" \
    "v_mfma_f32_16x16x128_f8f6f4 v[132:135], a[8:15], a[88:95], v[132:135] cbsz:1 blgp:1\n" \
    "v_mfma_f32_16x16x128_f8f6f4 v[136:139], a[8:15], a[96:103], v[136:139] cbsz:1 blgp:1\n" \
    "v_mfma_f32_16x16x128_f8f6f4 v[140:143], a[8:15], a[104:111], v[140:143] cbsz:1 blgp:1\n" \
    "s_waitcnt vmcnt(0)\n" \
    "a2a_u0_abj:\n" \
    "v_mfma_f32_16x16x128_f8f6f4 v[144:147], a[8:15], a[112:119], v[144:147] cbsz:1 blgp:1\n" \
    "v_cmp_le_u32 vcc, s59, v10\n" \
    "s_and_b32 s55, vcc_lo, 0x7f\n" \
    "s_or_b32 s66, s66, s55\n" \
    "s_lshr_b32 s55, s66, s70\n" \
    "s_lshr_b32 s56, s66, s69\n" \
    "s_andn2_b32 s55, s55, s56\n" \
    "s_bitcmp1_b32 s55, 0\n" \
    "s_cselect_b32 s71, s70, s69\n" \
    "v_mfma_f32_16x16x128_f8f6f4 v[148:151], a[8:15], a[120:127], v[148:151] cbsz:1 blgp:1\n" \
    "s_cselect_b32 s70, s69, s70\n" \
    "s_cselect_b32 s57, 16384, 0\n" \
    "s_mov_b32 s69, s71\n" \
    "s_xor_b32 s72, s72, s57\n" \
    "s_lshr_b32 s55, s66, s71\n" \
    "s_cmp_ge_u32 s73, s44\n" \
    "s_cselect_b32 s55, 0, s55\n" \
    "s_lshr_b32 s56, s66, s88\n" \
    "s_andn2_b32 s89, s56, s55\n" \
    "s_cmp_eq_u32 s69, 7\n" \
    "s_cselect_b32 s89, 1, s89\n" \
    "s_and_b32 s89, s89, 1\n" \
    "s_cselect_b32 s71, s88, s71\n" \
    "s_lshl_b32 s63, s71, 2\n" \
    "s_lshr_b32 s63, s61, s63\n" \
    "s_and_b32 s63, s63, 7\n" \
    "s_lshl_b32 s63, s63, 18\n" \
    "s_add_u32 s63, s63, s62\n" \
    "s_add_u32 s64, s63, 4096\n" \
    "s_add_u32 s65, s63, 8192\n" \
    "v_add_u32 v12, s72, v13\n" \
    "s_lshr_b32 s74, s66, s71\n" \
    "s_and_b32 s74, s74, 1\n" \
    "s_cbranch_scc0 a2a_u0_abx\n" \
    "buffer_load_dwordx4 a[40:43], v14, s[16:19], s63 offen offset:0 sc0 sc1\n" \
    "buffer_load_dwordx4 a[44:47], v14, s[16:19], s63 offen offset:1024 sc0 sc1\n" \
    "buffer_load_dwordx4 a[48:51], v14, s[16:19], s63 offen offset:2048 sc0 sc1\n" \
    "buffer_load_dwordx4 a[52:55], v14, s[16:19], s63 offen offset:3072 sc0 sc1\n" \
    "buffer_load_dwordx4 a[56:59], v14, s[16:19], s64 offen offset:0 sc0 sc1\n" \
    "buffer_load_dwordx4 a[60:63], v14, s[16:19], s64 offen offset:1024 sc0 sc1\n" \
    "buffer_load_dwordx4 a[64:67], v14, s[16:19], s64 offen offset:2048 sc0 sc1\n" \
    "buffer_load_dwordx4 a[68:71], v14, s[16:19], s64 offen offset:3072 sc0 sc1\n" \
    "s_cmp_eq_u32 s80, 0\n" \
    "s_cbranch_scc1 a2a_u0_b5\n" \
    "v_mfma_f32_16x16x128_f8f6f4 v[224:227], a[32:39], a[80:87], v[224:227] cbsz:1 blgp:1\n" \
    "buffer_load_dwordx4 a[72:75], v14, s[16:19], s65 offen offset:0 sc0 sc1\n" \
    "v_mfma_f32_16x16x128_f8f6f4 v[228:231], a[32:39], a[88:95], v[228:231] cbsz:1 blgp:1\n" \
    "buffer_load_dwordx4 a[76:79], v14, s[16:19], s65 offen offset:1024 sc0 sc1\n" \
    "v_mfma_f32_16x16x128_f8f6f4 v[232:235], a[32:39], a[96:103], v[232:235] cbsz:1 blgp:1\n" \
    "v_mfma_f32_16x16x128_f8f6f4 v[236:239], a[32:39], a[104:111], v[236:239] cbsz:1 blgp:1\n" \
    "v_mfma_f32_16x16x128_f8f6f4 v[240:243], a[32:39], a[112:119], v[240:243] cbsz:1 blgp:1\n" \
    "v_mfma_f32_16x16x128_f8f6f4 v[244:247], a[32:39], a[120:127], v[244:247] cbsz:1 blgp:1\n" \
    "v_mfma_f32_16x16x128_f8f6f4 v[248:251], a[32:39], a[128:135], v[248:251] cbsz:1 blgp:1\n" \
    "v_mfma_f32_16x16x128_f8f6f4 v[252:255], a[32:39], a[136:143], v[252:255] cbsz:1 blgp:1\n" \
    "a2a_u0_b5:\n" \
    "s_waitcnt vmcnt(8)\n" \
    "s_branch a2a_u0_h2n\n" \
    "a2a_u0_abx:\n" \
    "global_load_dword v10, v15, s[38:39] sc0 sc1\n" \
    "s_cmp_eq_u32 s80, 0\n" \
    "s_cbranch_scc1 a2a_u0_c5\n" \
    "v_mfma_f32_16x16x128_f8f6f4 v[224:227], a[32:39], a[80:87], v[224:227] cbsz:1 blgp:1\n" \
    "v_mfma_f32_16x16x128_f8f6f4 v[228:231], a[32:39], a[88:95], v[228:231] cbsz:1 blgp:1\n" \
    "v_mfma_f32_16x16x128_f8f6f4 v[232:235], a[32:39], a[96:103], v[232:235] cbsz:1 blgp:1\n" \
    "v_mfma_f32_16x16x128_f8f6f4 v[236:239], a[32:39], a[104:111], v[236:239] cbsz:1 blgp:1\n" \
    "v_mfma_f32_16x16x128_f8f6f4 v[240:243], a[32:39], a[112:119], v[240:243] cbsz:1 blgp:1\n" \
    "v_mfma_f32_16x16x128_f8f6f4 v[244:247], a[32:39], a[120:127], v[244:247] cbsz:1 blgp:1\n" \
    "v_mfma_f32_16x16x128_f8f6f4 v[248:251], a[32:39], a[128:135], v[248:251] cbsz:1 blgp:1\n" \
    "v_mfma_f32_16x16x128_f8f6f4 v[252:255], a[32:39], a[136:143], v[252:255] cbsz:1 blgp:1\n" \
    "a2a_u0_c5:\n" \
    "s_waitcnt vmcnt(0)\n" \
    "v_cmp_le_u32 vcc, s59, v10\n" \
    "s_and_b32 s55, vcc_lo, 0x7f\n" \
    "s_or_b32 s66, s66, s55\n" \
    "s_lshr_b32 s55, s66, s70\n" \
    "s_lshr_b32 s56, s66, s69\n" \
    "s_andn2_b32 s55, s55, s56\n" \
    "s_bitcmp1_b32 s55, 0\n" \
    "s_cselect_b32 s71, s70, s69\n" \
    "s_cselect_b32 s70, s69, s70\n" \
    "s_cselect_b32 s57, 16384, 0\n" \
    "s_mov_b32 s69, s71\n" \
    "s_xor_b32 s72, s72, s57\n" \
    "s_lshr_b32 s55, s66, s71\n" \
    "s_cmp_ge_u32 s73, s44\n" \
    "s_cselect_b32 s55, 0, s55\n" \
    "s_lshr_b32 s56, s66, s88\n" \
    "s_andn2_b32 s89, s56, s55\n" \
    "s_cmp_eq_u32 s69, 7\n" \
    "s_cselect_b32 s89, 1, s89\n" \
    "s_and_b32 s89, s89, 1\n" \
    "s_cselect_b32 s71, s88, s71\n" \
    "s_lshl_b32 s63, s71, 2\n" \
    "s_lshr_b32 s63, s61, s63\n" \
    "s_and_b32 s63, s63, 7\n" \
    "s_lshl_b32 s63, s63, 18\n" \
    "s_add_u32 s63, s63, s62\n" \
    "s_add_u32 s64, s63, 4096\n" \
    "s_add_u32 s65, s63, 8192\n" \
    "v_add_u32 v12, s72, v13\n" \
    "s_lshr_b32 s74, s66, s71\n" \
    "s_and_b32 s74, s74, 1\n" \
    "s_cbranch_scc0 a2a_u0_h2n\n" \
    "buffer_load_dwordx4 a[40:43], v14, s[16:19], s63 offen offset:0 sc0 sc1\n" \
    "buffer_load_dwordx4 a[44:47], v14, s[16:19], s63 offen offset:1024 sc0 sc1\n" \
    "buffer_load_dwordx4 a[48:51], v14, s[16:19], s63 offen offset:2048 sc0 sc1\n" \
    "buffer_load_dwordx4 a[52:55], v14, s[16:19], s63 offen offset:3072 sc0 sc1\n" \
    "buffer_load_dwordx4 a[56:59], v14, s[16:19], s64 offen offset:0 sc0 sc1\n" \
    "buffer_load_dwordx4 a[60:63], v14, s[16:19], s64 offen offset:1024 sc0 sc1\n" \
    "buffer_load_dwordx4 a[64:67], v14, s[16:19], s64 offen offset:2048 sc0 sc1\n" \
    "buffer_load_dwordx4 a[68:71], v14, s[16:19], s64 offen offset:3072 sc0 sc1\n" \
    "s_cmp_eq_u32 s80, 0\n" \
    "s_cbranch_scc1 a2a_u0_aby\n" \
    "buffer_load_dwordx4 a[72:75], v14, s[16:19], s65 offen offset:0 sc0 sc1\n" \
    "buffer_load_dwordx4 a[76:79], v14, s[16:19], s65 offen offset:1024 sc0 sc1\n" \
    "a2a_u0_aby:\n" \
    "s_branch a2a_u0_h2n\n" \
      \
    "a2a_stall:\n" \
    "s_mov_b32 s78, 0\n" \
    "s_mov_b32 s90, 0\n" \
    "a2a_st_note:\n" \
    "v_cmp_le_u32 vcc, s59, v10\n" \
    "s_and_b32 s55, vcc_lo, 0x7f\n" \
    "s_or_b32 s66, s66, s55\n" \
    "s_bitcmp1_b32 s66, s71\n" \
    "s_cbranch_scc1 a2a_st_go\n" \
    "s_bitcmp1_b32 s66, s69\n" \
    "s_cbranch_scc1 a2a_st_slot\n" \
    "s_bitcmp1_b32 s66, s70\n" \
    "s_cbranch_scc1 a2a_st_slot\n" \
    "s_bitcmp1_b32 s66, s88\n" \
    "s_cbranch_scc1 a2a_st_park\n" \
    "s_cmp_eq_u32 s67, 0\n" \
    "s_cbranch_scc1 a2a_st_poll\n" \
    "s_cmp_eq_u32 s70, 7\n" \
    "s_cbranch_scc1 a2a_st_fill\n" \
    "s_cmp_lt_u32 s88, 7\n" \
    "s_cbranch_scc1 a2a_st_full\n" \
    "s_xor_b32 s55, s72, 16384\n" \
    "v_add_u32 v12, s55, v13\n" \
    "s_waitcnt vmcnt(0)\n" \
    "v_xor_b32 v11, 0, v12\n" \
    "ds_read_b64_tr_b8 v[32:33], v11\n" \
    "ds_read_b64_tr_b8 v[34:35], v11 offset:1024\n" \
    "ds_read_b64_tr_b8 v[36:37], v11 offset:8192\n" \
    "ds_read_b64_tr_b8 v[38:39], v11 offset:9216\n" \
    "v_xor_b32 v11, 16, v12\n" \
    "ds_read_b64_tr_b8 v[40:41], v11\n" \
    "ds_read_b64_tr_b8 v[42:43], v11 offset:1024\n" \
    "ds_read_b64_tr_b8 v[44:45], v11 offset:8192\n" \
    "ds_read_b64_tr_b8 v[46:47], v11 offset:9216\n" \
    "v_xor_b32 v11, 32, v12\n" \
    "ds_read_b64_tr_b8 v[48:49], v11\n" \
    "ds_read_b64_tr_b8 v[50:51], v11 offset:1024\n" \
    "ds_read_b64_tr_b8 v[52:53], v11 offset:8192\n" \
    "ds_read_b64_tr_b8 v[54:55], v11 offset:9216\n" \
    "v_xor_b32 v11, 48, v12\n" \
    "ds_read_b64_tr_b8 v[56:57], v11\n" \
    "ds_read_b64_tr_b8 v[58:59], v11 offset:1024\n" \
    "ds_read_b64_tr_b8 v[60:61], v11 offset:8192\n" \
    "ds_read_b64_tr_b8 v[62:63], v11 offset:9216\n" \
    "v_xor_b32 v11, 64, v12\n" \
    "ds_read_b64_tr_b8 v[64:65], v11\n" \
    "ds_read_b64_tr_b8 v[66:67], v11 offset:1024\n" \
    "ds_read_b64_tr_b8 v[68:69], v11 offset:8192\n" \
    "ds_read_b64_tr_b8 v[70:71], v11 offset:9216\n" \
    "v_xor_b32 v11, 80, v12\n" \
    "ds_read_b64_tr_b8 v[72:73], v11\n" \
    "ds_read_b64_tr_b8 v[74:75], v11 offset:1024\n" \
    "ds_read_b64_tr_b8 v[76:77], v11 offset:8192\n" \
    "ds_read_b64_tr_b8 v[78:79], v11 offset:9216\n" \
    "v_xor_b32 v11, 96, v12\n" \
    "ds_read_b64_tr_b8 v[80:81], v11\n" \
    "ds_read_b64_tr_b8 v[82:83], v11 offset:1024\n" \
    "ds_read_b64_tr_b8 v[84:85], v11 offset:8192\n" \
    "ds_read_b64_tr_b8 v[86:87], v11 offset:9216\n" \
    "v_xor_b32 v11, 112, v12\n" \
    "ds_read_b64_tr_b8 v[88:89], v11\n" \
    "ds_read_b64_tr_b8 v[90:91], v11 offset:1024\n" \
    "ds_read_b64_tr_b8 v[92:93], v11 offset:8192\n" \
    "ds_read_b64_tr_b8 v[94:95], v11 offset:9216\n" \
    "s_mov_b32 s88, s70\n" \
    "s_waitcnt lgkmcnt(0)\n" \
    "s_branch a2a_st_fill\n" \
    "a2a_st_full:\n" \
    "s_and_b32 s55, s67, s66\n" \
    "s_cbranch_scc0 a2a_st_poll\n" \
    "s_add_u32 s78, s78, 1\n" \
    "s_cmp_lt_u32 s78, 2\n" \
    "s_cbranch_scc1 a2a_st_poll\n" \
    "s_bitset1_b32 s67, s70\n" \
    "s_bitset1_b32 s68, s70\n" \
    "a2a_st_fill:\n" \
    "s_andn2_b32 s55, s67, s68\n" \
    "s_cselect_b32 s55, s55, s67\n" \
    "s_and_b32 s56, s67, s66\n" \
    "s_cselect_b32 s55, s56, s55\n" \
    "s_or_b32 s55, s55, 0x80\n" \
    "s_ff1_i32_b32 s79, s55\n" \
    "s_lshl_b32 s77, s79, 2\n" \
    "s_lshr_b32 s77, s61, s77\n" \
    "s_and_b32 s77, s77, 7\n" \
    "s_lshl_b32 s77, s77, 21\n" \
    "s_add_u32 s77, s77, s53\n" \
    "s_bitset0_b32 s67, s79\n" \
    "s_mov_b32 s70, s79\n" \
    "s_xor_b32 s76, s72, s54\n" \
    "global_load_dword v10, v15, s[38:39] sc0 sc1\n" \
    "s_xor_b32 m0, s76, 0\n" \
    "s_xor_b32 s58, s77, 0\n" \
    "buffer_load_dwordx4 v4, s[8:11], s58 offen offset:0 lds\n" \
    "s_add_u32 s58, s58, 31744\n" \
    "buffer_load_dwordx4 v4, s[8:11], s58 offen offset:1024 lds\n" \
    "s_add_u32 s58, s58, 31744\n" \
    "buffer_load_dwordx4 v5, s[8:11], s58 offen offset:2048 lds\n" \
    "s_add_u32 s58, s58, 31744\n" \
    "buffer_load_dwordx4 v5, s[8:11], s58 offen offset:3072 lds\n" \
    "s_xor_b32 m0, s76, 4096\n" \
    "s_xor_b32 s58, s77, 131072\n" \
    "buffer_load_dwordx4 v4, s[8:11], s58 offen offset:0 lds\n" \
    "s_add_u32 s58, s58, 31744\n" \
    "buffer_load_dwordx4 v4, s[8:11], s58 offen offset:1024 lds\n" \
    "s_add_u32 s58, s58, 31744\n" \
    "buffer_load_dwordx4 v5, s[8:11], s58 offen offset:2048 lds\n" \
    "s_add_u32 s58, s58, 31744\n" \
    "buffer_load_dwordx4 v5, s[8:11], s58 offen offset:3072 lds\n" \
    "s_xor_b32 m0, s76, 8192\n" \
    "s_xor_b32 s58, s77, 262144\n" \
    "buffer_load_dwordx4 v4, s[8:11], s58 offen offset:0 lds\n" \
    "s_add_u32 s58, s58, 31744\n" \
    "buffer_load_dwordx4 v4, s[8:11], s58 offen offset:1024 lds\n" \
    "s_add_u32 s58, s58, 31744\n" \
    "buffer_load_dwordx4 v5, s[8:11], s58 offen offset:2048 lds\n" \
    "s_add_u32 s58, s58, 31744\n" \
    "buffer_load_dwordx4 v5, s[8:11], s58 offen offset:3072 lds\n" \
    "s_xor_b32 m0, s76, 12288\n" \
    "s_xor_b32 s58, s77, 393216\n" \
    "buffer_load_dwordx4 v4, s[8:11], s58 offen offset:0 lds\n" \
    "s_add_u32 s58, s58, 31744\n" \
    "buffer_load_dwordx4 v4, s[8:11], s58 offen offset:1024 lds\n" \
    "s_add_u32 s58, s58, 31744\n" \
    "buffer_load_dwordx4 v5, s[8:11], s58 offen offset:2048 lds\n" \
    "s_add_u32 s58, s58, 31744\n" \
    "buffer_load_dwordx4 v5, s[8:11], s58 offen offset:3072 lds\n" \
    "s_waitcnt vmcnt(16)\n" \
    "s_branch a2a_st_note\n" \
    "a2a_st_poll:\n" \
    "global_load_dword v10, v15, s[38:39] sc0 sc1\n" \
    "s_memrealtime s[36:37]\n" \
    "s_waitcnt vmcnt(0) lgkmcnt(0)\n" \
    "s_sub_u32 s36, s36, s60\n" \
    "s_cmp_le_u32 s36, 100000000\n" \
    "s_cbranch_scc1 a2a_st_note\n" \
    "v_mov_b32 v16, 0x7f7f7f7f\n" \
    "v_accvgpr_write_b32 a0, v16\n" \
    "v_accvgpr_write_b32 a8, v16\n" \
    "v_accvgpr_write_b32 a16, v16\n" \
    "v_accvgpr_write_b32 a24, v16\n" \
    "v_accvgpr_write_b32 a32, v16\n" \
    "v_accvgpr_write_b32 a40, v16\n" \
    "v_accvgpr_write_b32 a48, v16\n" \
    "v_accvgpr_write_b32 a56, v16\n" \
    "v_accvgpr_write_b32 a64, v16\n" \
    "v_accvgpr_write_b32 a72, v16\n" \
    "s_nop 1\n" \
    "s_branch a2a_st_ret\n" \
    "a2a_st_park:\n" \
    "s_bitset1_b32 s67, s71\n" \
    "s_bitset1_b32 s68, s71\n" \
    "s_mov_b32 s71, s88\n" \
    "s_mov_b32 s88, 7\n" \
    "s_mov_b32 s90, 3\n" \
    "s_branch a2a_st_go\n" \
    "a2a_st_slot:\n" \
    "s_mov_b32 s90, 2\n" \
    "s_cmp_eq_u32 s88, 7\n" \
    "s_cselect_b32 s88, s71, s88\n" \
    "s_cbranch_scc1 a2a_st_sl2\n" \
    "s_mov_b32 s90, 1\n" \
    "s_bitset1_b32 s67, s71\n" \
    "s_bitset1_b32 s68, s71\n" \
    "a2a_st_sl2:\n" \
    "s_lshr_b32 s55, s66, s70\n" \
    "s_lshr_b32 s56, s66, s69\n" \
    "s_andn2_b32 s55, s55, s56\n" \
    "s_bitcmp1_b32 s55, 0\n" \
    "s_cselect_b32 s71, s70, s69\n" \
    "s_cselect_b32 s70, s69, s70\n" \
    "s_cselect_b32 s57, 16384, 0\n" \
    "s_mov_b32 s69, s71\n" \
    "s_xor_b32 s72, s72, s57\n" \
    "v_add_u32 v12, s72, v13\n" \
    "s_mov_b32 s69, s70\n" \
    "s_mov_b32 s70, 7\n" \
    "s_xor_b32 s72, s72, 16384\n" \
    "a2a_st_go:\n" \
    "s_lshl_b32 s63, s71, 2\n" \
    "s_lshr_b32 s63, s61, s63\n" \
    "s_and_b32 s63, s63, 7\n" \
    "s_lshl_b32 s63, s63, 18\n" \
    "s_add_u32 s63, s63, s62\n" \
    "s_add_u32 s64, s63, 4096\n" \
    "s_add_u32 s65, s63, 8192\n" \
    "s_cmp_eq_u32 s75, 0\n" \
    "s_cbranch_scc1 a2a_st_l1\n" \
    "a2a_st_l0:\n" \
    "buffer_load_dwordx4 a[0:3], v14, s[16:19], s63 offen offset:0 sc0 sc1\n" \
    "buffer_load_dwordx4 a[4:7], v14, s[16:19], s63 offen offset:1024 sc0 sc1\n" \
    "buffer_load_dwordx4 a[8:11], v14, s[16:19], s63 offen offset:2048 sc0 sc1\n" \
    "buffer_load_dwordx4 a[12:15], v14, s[16:19], s63 offen offset:3072 sc0 sc1\n" \
    "buffer_load_dwordx4 a[16:19], v14, s[16:19], s64 offen offset:0 sc0 sc1\n" \
    "buffer_load_dwordx4 a[20:23], v14, s[16:19], s64 offen offset:1024 sc0 sc1\n" \
    "buffer_load_dwordx4 a[24:27], v14, s[16:19], s64 offen offset:2048 sc0 sc1\n" \
    "buffer_load_dwordx4 a[28:31], v14, s[16:19], s64 offen offset:3072 sc0 sc1\n" \
    "s_cmp_eq_u32 s80, 0\n" \
    "s_cbranch_scc1 a2a_st_l0x\n" \
    "buffer_load_dwordx4 a[32:35], v14, s[16:19], s65 offen offset:0 sc0 sc1\n" \
    "buffer_load_dwordx4 a[36:39], v14, s[16:19], s65 offen offset:1024 sc0 sc1\n" \
    "a2a_st_l0x:\n" \
    "s_cmp_eq_u32 s90, 0\n" \
    "s_cbranch_scc1 a2a_st_x\n" \
    "s_cmp_eq_u32 s90, 1\n" \
    "s_cbranch_scc1 a2a_st_r0\n" \
    "s_cmp_eq_u32 s90, 2\n" \
    "s_cbranch_scc1 a2a_st_s0\n" \
    "v_accvgpr_write_b32 a80, v32\n" \
    "v_accvgpr_write_b32 a81, v33\n" \
    "v_accvgpr_write_b32 a82, v34\n" \
    "v_accvgpr_write_b32 a83, v35\n" \
    "v_accvgpr_write_b32 a84, v36\n" \
    "v_accvgpr_write_b32 a85, v37\n" \
    "v_accvgpr_write_b32 a86, v38\n" \
    "v_accvgpr_write_b32 a87, v39\n" \
    "v_accvgpr_write_b32 a88, v40\n" \
    "v_accvgpr_write_b32 a89, v41\n" \
    "v_accvgpr_write_b32 a90, v42\n" \
    "v_accvgpr_write_b32 a91, v43\n" \
    "v_accvgpr_write_b32 a92, v44\n" \
    "v_accvgpr_write_b32 a93, v45\n" \
    "v_accvgpr_write_b32 a94, v46\n" \
    "v_accvgpr_write_b32 a95, v47\n" \
    "v_accvgpr_write_b32 a96, v48\n" \
    "v_accvgpr_write_b32 a97, v49\n" \
    "v_accvgpr_write_b32 a98, v50\n" \
    "v_accvgpr_write_b32 a99, v51\n" \
    "v_accvgpr_write_b32 a100, v52\n" \
    "v_accvgpr_write_b32 a101, v53\n" \
    "v_accvgpr_write_b32 a102, v54\n" \
    "v_accvgpr_write_b32 a103, v55\n" \
    "v_accvgpr_write_b32 a104, v56\n" \
    "v_accvgpr_write_b32 a105, v57\n" \
    "v_accvgpr_write_b32 a106, v58\n" \
    "v_accvgpr_write_b32 a107, v59\n" \
    "v_accvgpr_write_b32 a108, v60\n" \
    "v_accvgpr_write_b32 a109, v61\n" \
    "v_accvgpr_write_b32 a110, v62\n" \
    "v_accvgpr_write_b32 a111, v63\n" \
    "v_accvgpr_write_b32 a112, v64\n" \
    "v_accvgpr_write_b32 a113, v65\n" \
    "v_accvgpr_write_b32 a114, v66\n" \
    "v_accvgpr_write_b32 a115, v67\n" \
    "v_accvgpr_write_b32 a116, v68\n" \
    "v_accvgpr_write_b32 a117, v69\n" \
    "v_accvgpr_write_b32 a118, v70\n" \
    "v_accvgpr_write_b32 a119, v71\n" \
    "v_accvgpr_write_b32 a120, v72\n" \
    "v_accvgpr_write_b32 a121, v73\n" \
    "v_accvgpr_write_b32 a122, v74\n" \
    "v_accvgpr_write_b32 a123, v75\n" \
    "v_accvgpr_write_b32 a124, v76\n" \
    "v_accvgpr_write_b32 a125, v77\n" \
    "v_accvgpr_write_b32 a126, v78\n" \
    "v_accvgpr_write_b32 a127, v79\n" \
    "v_accvgpr_write_b32 a128, v80\n" \
    "v_accvgpr_write_b32 a129, v81\n" \
    "v_accvgpr_write_b32 a130, v82\n" \
    "v_accvgpr_write_b32 a131, v83\n" \
    "v_accvgpr_write_b32 a132, v84\n" \
    "v_accvgpr_write_b32 a133, v85\n" \
    "v_accvgpr_write_b32 a134, v86\n" \
    "v_accvgpr_write_b32 a135, v87\n" \
    "v_accvgpr_write_b32 a136, v88\n" \
    "v_accvgpr_write_b32 a137, v89\n" \
    "v_accvgpr_write_b32 a138, v90\n" \
    "v_accvgpr_write_b32 a139, v91\n" \
    "v_accvgpr_write_b32 a140, v92\n" \
    "v_accvgpr_write_b32 a141, v93\n" \
    "v_accvgpr_write_b32 a142, v94\n" \
    "v_accvgpr_write_b32 a143, v95\n" \
    "s_branch a2a_st_x\n" \
    "a2a_st_s0:\n" \
    "v_accvgpr_read_b32 v32, a80\n" \
    "v_accvgpr_read_b32 v33, a81\n" \
    "v_accvgpr_read_b32 v34, a82\n" \
    "v_accvgpr_read_b32 v35, a83\n" \
    "v_accvgpr_read_b32 v36, a84\n" \
    "v_accvgpr_read_b32 v37, a85\n" \
    "v_accvgpr_read_b32 v38, a86\n" \
    "v_accvgpr_read_b32 v39, a87\n" \
    "v_accvgpr_read_b32 v40, a88\n" \
    "v_accvgpr_read_b32 v41, a89\n" \
    "v_accvgpr_read_b32 v42, a90\n" \
    "v_accvgpr_read_b32 v43, a91\n" \
    "v_accvgpr_read_b32 v44, a92\n" \
    "v_accvgpr_read_b32 v45, a93\n" \
    "v_accvgpr_read_b32 v46, a94\n" \
    "v_accvgpr_read_b32 v47, a95\n" \
    "v_accvgpr_read_b32 v48, a96\n" \
    "v_accvgpr_read_b32 v49, a97\n" \
    "v_accvgpr_read_b32 v50, a98\n" \
    "v_accvgpr_read_b32 v51, a99\n" \
    "v_accvgpr_read_b32 v52, a100\n" \
    "v_accvgpr_read_b32 v53, a101\n" \
    "v_accvgpr_read_b32 v54, a102\n" \
    "v_accvgpr_read_b32 v55, a103\n" \
    "v_accvgpr_read_b32 v56, a104\n" \
    "v_accvgpr_read_b32 v57, a105\n" \
    "v_accvgpr_read_b32 v58, a106\n" \
    "v_accvgpr_read_b32 v59, a107\n" \
    "v_accvgpr_read_b32 v60, a108\n" \
    "v_accvgpr_read_b32 v61, a109\n" \
    "v_accvgpr_read_b32 v62, a110\n" \
    "v_accvgpr_read_b32 v63, a111\n" \
    "v_accvgpr_read_b32 v64, a112\n" \
    "v_accvgpr_read_b32 v65, a113\n" \
    "v_accvgpr_read_b32 v66, a114\n" \
    "v_accvgpr_read_b32 v67, a115\n" \
    "v_accvgpr_read_b32 v68, a116\n" \
    "v_accvgpr_read_b32 v69, a117\n" \
    "v_accvgpr_read_b32 v70, a118\n" \
    "v_accvgpr_read_b32 v71, a119\n" \
    "v_accvgpr_read_b32 v72, a120\n" \
    "v_accvgpr_read_b32 v73, a121\n" \
    "v_accvgpr_read_b32 v74, a122\n" \
    "v_accvgpr_read_b32 v75, a123\n" \
    "v_accvgpr_read_b32 v76, a124\n" \
    "v_accvgpr_read_b32 v77, a125\n" \
    "v_accvgpr_read_b32 v78, a126\n" \
    "v_accvgpr_read_b32 v79, a127\n" \
    "v_accvgpr_read_b32 v80, a128\n" \
    "v_accvgpr_read_b32 v81, a129\n" \
    "v_accvgpr_read_b32 v82, a130\n" \
    "v_accvgpr_read_b32 v83, a131\n" \
    "v_accvgpr_read_b32 v84, a132\n" \
    "v_accvgpr_read_b32 v85, a133\n" \
    "v_accvgpr_read_b32 v86, a134\n" \
    "v_accvgpr_read_b32 v87, a135\n" \
    "v_accvgpr_read_b32 v88, a136\n" \
    "v_accvgpr_read_b32 v89, a137\n" \
    "v_accvgpr_read_b32 v90, a138\n" \
    "v_accvgpr_read_b32 v91, a139\n" \
    "v_accvgpr_read_b32 v92, a140\n" \
    "v_accvgpr_read_b32 v93, a141\n" \
    "v_accvgpr_read_b32 v94, a142\n" \
    "v_accvgpr_read_b32 v95, a143\n" \
    "a2a_st_r0:\n" \
    "s_waitcnt vmcnt(8)\n" \
    "v_xor_b32 v11, 0, v12\n" \
    "ds_read_b64_tr_b8 a[80:81], v11\n" \
    "ds_read_b64_tr_b8 a[82:83], v11 offset:1024\n" \
    "ds_read_b64_tr_b8 a[84:85], v11 offset:8192\n" \
    "ds_read_b64_tr_b8 a[86:87], v11 offset:9216\n" \
    "v_xor_b32 v11, 16, v12\n" \
    "ds_read_b64_tr_b8 a[88:89], v11\n" \
    "ds_read_b64_tr_b8 a[90:91], v11 offset:1024\n" \
    "ds_read_b64_tr_b8 a[92:93], v11 offset:8192\n" \
    "ds_read_b64_tr_b8 a[94:95], v11 offset:9216\n" \
    "v_xor_b32 v11, 32, v12\n" \
    "ds_read_b64_tr_b8 a[96:97], v11\n" \
    "ds_read_b64_tr_b8 a[98:99], v11 offset:1024\n" \
    "ds_read_b64_tr_b8 a[100:101], v11 offset:8192\n" \
    "ds_read_b64_tr_b8 a[102:103], v11 offset:9216\n" \
    "v_xor_b32 v11, 48, v12\n" \
    "ds_read_b64_tr_b8 a[104:105], v11\n" \
    "ds_read_b64_tr_b8 a[106:107], v11 offset:1024\n" \
    "ds_read_b64_tr_b8 a[108:109], v11 offset:8192\n" \
    "ds_read_b64_tr_b8 a[110:111], v11 offset:9216\n" \
    "v_xor_b32 v11, 64, v12\n" \
    "ds_read_b64_tr_b8 a[112:113], v11\n" \
    "ds_read_b64_tr_b8 a[114:115], v11 offset:1024\n" \
    "ds_read_b64_tr_b8 a[116:117], v11 offset:8192\n" \
    "ds_read_b64_tr_b8 a[118:119], v11 offset:9216\n" \
    "v_xor_b32 v11, 80, v12\n" \
    "ds_read_b64_tr_b8 a[120:121], v11\n" \
    "ds_read_b64_tr_b8 a[122:123], v11 offset:1024\n" \
    "ds_read_b64_tr_b8 a[124:125], v11 offset:8192\n" \
    "ds_read_b64_tr_b8 a[126:127], v11 offset:9216\n" \
    "v_xor_b32 v11, 96, v12\n" \
    "ds_read_b64_tr_b8 a[128:129], v11\n" \
    "ds_read_b64_tr_b8 a[130:131], v11 offset:1024\n" \
    "ds_read_b64_tr_b8 a[132:133], v11 offset:8192\n" \
    "ds_read_b64_tr_b8 a[134:135], v11 offset:9216\n" \
    "v_xor_b32 v11, 112, v12\n" \
    "ds_read_b64_tr_b8 a[136:137], v11\n" \
    "ds_read_b64_tr_b8 a[138:139], v11 offset:1024\n" \
    "ds_read_b64_tr_b8 a[140:141], v11 offset:8192\n" \
    "ds_read_b64_tr_b8 a[142:143], v11 offset:9216\n" \
    "s_waitcnt lgkmcnt(0)\n" \
    "s_branch a2a_st_x\n" \
    "a2a_st_l1:\n" \
    "buffer_load_dwordx4 a[40:43], v14, s[16:19], s63 offen offset:0 sc0 sc1\n" \
    "buffer_load_dwordx4 a[44:47], v14, s[16:19], s63 offen offset:1024 sc0 sc1\n" \
    "buffer_load_dwordx4 a[48:51], v14, s[16:19], s63 offen offset:2048 sc0 sc1\n" \
    "buffer_load_dwordx4 a[52:55], v14, s[16:19], s63 offen offset:3072 sc0 sc1\n" \
    "buffer_load_dwordx4 a[56:59], v14, s[16:19], s64 offen offset:0 sc0 sc1\n" \
    "buffer_load_dwordx4 a[60:63], v14, s[16:19], s64 offen offset:1024 sc0 sc1\n" \
    "buffer_load_dwordx4 a[64:67], v14, s[16:19], s64 offen offset:2048 sc0 sc1\n" \
    "buffer_load_dwordx4 a[68:71], v14, s[16:19], s64 offen offset:3072 sc0 sc1\n" \
    "s_cmp_eq_u32 s80, 0\n" \
    "s_cbranch_scc1 a2a_st_l1x\n" \
    "buffer_load_dwordx4 a[72:75], v14, s[16:19], s65 offen offset:0 sc0 sc1\n" \
    "buffer_load_dwordx4 a[76:79], v14, s[16:19], s65 offen offset:1024 sc0 sc1\n" \
    "a2a_st_l1x:\n" \
    "s_cmp_eq_u32 s90, 0\n" \
    "s_cbranch_scc1 a2a_st_x\n" \
    "s_cmp_eq_u32 s90, 1\n" \
    "s_cbranch_scc1 a2a_st_r1\n" \
    "s_cmp_eq_u32 s90, 2\n" \
    "s_cbranch_scc1 a2a_st_s1\n" \
    "v_accvgpr_write_b32 a144, v32\n" \
    "v_accvgpr_write_b32 a145, v33\n" \
    "v_accvgpr_write_b32 a146, v34\n" \
    "v_accvgpr_write_b32 a147, v35\n" \
    "v_accvgpr_write_b32 a148, v36\n" \
    "v_accvgpr_write_b32 a149, v37\n" \
    "v_accvgpr_write_b32 a150, v38\n" \
    "v_accvgpr_write_b32 a151, v39\n" \
    "v_accvgpr_write_b32 a152, v40\n" \
    "v_accvgpr_write_b32 a153, v41\n" \
    "v_accvgpr_write_b32 a154, v42\n" \
    "v_accvgpr_write_b32 a155, v43\n" \
    "v_accvgpr_write_b32 a156, v44\n" \
    "v_accvgpr_write_b32 a157, v45\n" \
    "v_accvgpr_write_b32 a158, v46\n" \
    "v_accvgpr_write_b32 a159, v47\n" \
    "v_accvgpr_write_b32 a160, v48\n" \
    "v_accvgpr_write_b32 a161, v49\n" \
    "v_accvgpr_write_b32 a162, v50\n" \
    "v_accvgpr_write_b32 a163, v51\n" \
    "v_accvgpr_write_b32 a164, v52\n" \
    "v_accvgpr_write_b32 a165, v53\n" \
    "v_accvgpr_write_b32 a166, v54\n" \
    "v_accvgpr_write_b32 a167, v55\n" \
    "v_accvgpr_write_b32 a168, v56\n" \
    "v_accvgpr_write_b32 a169, v57\n" \
    "v_accvgpr_write_b32 a170, v58\n" \
    "v_accvgpr_write_b32 a171, v59\n" \
    "v_accvgpr_write_b32 a172, v60\n" \
    "v_accvgpr_write_b32 a173, v61\n" \
    "v_accvgpr_write_b32 a174, v62\n" \
    "v_accvgpr_write_b32 a175, v63\n" \
    "v_accvgpr_write_b32 a176, v64\n" \
    "v_accvgpr_write_b32 a177, v65\n" \
    "v_accvgpr_write_b32 a178, v66\n" \
    "v_accvgpr_write_b32 a179, v67\n" \
    "v_accvgpr_write_b32 a180, v68\n" \
    "v_accvgpr_write_b32 a181, v69\n" \
    "v_accvgpr_write_b32 a182, v70\n" \
    "v_accvgpr_write_b32 a183, v71\n" \
    "v_accvgpr_write_b32 a184, v72\n" \
    "v_accvgpr_write_b32 a185, v73\n" \
    "v_accvgpr_write_b32 a186, v74\n" \
    "v_accvgpr_write_b32 a187, v75\n" \
    "v_accvgpr_write_b32 a188, v76\n" \
    "v_accvgpr_write_b32 a189, v77\n" \
    "v_accvgpr_write_b32 a190, v78\n" \
    "v_accvgpr_write_b32 a191, v79\n" \
    "v_accvgpr_write_b32 a192, v80\n" \
    "v_accvgpr_write_b32 a193, v81\n" \
    "v_accvgpr_write_b32 a194, v82\n" \
    "v_accvgpr_write_b32 a195, v83\n" \
    "v_accvgpr_write_b32 a196, v84\n" \
    "v_accvgpr_write_b32 a197, v85\n" \
    "v_accvgpr_write_b32 a198, v86\n" \
    "v_accvgpr_write_b32 a199, v87\n" \
    "v_accvgpr_write_b32 a200, v88\n" \
    "v_accvgpr_write_b32 a201, v89\n" \
    "v_accvgpr_write_b32 a202, v90\n" \
    "v_accvgpr_write_b32 a203, v91\n" \
    "v_accvgpr_write_b32 a204, v92\n" \
    "v_accvgpr_write_b32 a205, v93\n" \
    "v_accvgpr_write_b32 a206, v94\n" \
    "v_accvgpr_write_b32 a207, v95\n" \
    "s_branch a2a_st_x\n" \
    "a2a_st_s1:\n" \
    "v_accvgpr_read_b32 v32, a144\n" \
    "v_accvgpr_read_b32 v33, a145\n" \
    "v_accvgpr_read_b32 v34, a146\n" \
    "v_accvgpr_read_b32 v35, a147\n" \
    "v_accvgpr_read_b32 v36, a148\n" \
    "v_accvgpr_read_b32 v37, a149\n" \
    "v_accvgpr_read_b32 v38, a150\n" \
    "v_accvgpr_read_b32 v39, a151\n" \
    "v_accvgpr_read_b32 v40, a152\n" \
    "v_accvgpr_read_b32 v41, a153\n" \
    "v_accvgpr_read_b32 v42, a154\n" \
    "v_accvgpr_read_b32 v43, a155\n" \
    "v_accvgpr_read_b32 v44, a156\n" \
    "v_accvgpr_read_b32 v45, a157\n" \
    "v_accvgpr_read_b32 v46, a158\n" \
    "v_accvgpr_read_b32 v47, a159\n" \
    "v_accvgpr_read_b32 v48, a160\n" \
    "v_accvgpr_read_b32 v49, a161\n" \
    "v_accvgpr_read_b32 v50, a162\n" \
    "v_accvgpr_read_b32 v51, a163\n" \
    "v_accvgpr_read_b32 v52, a164\n" \
    "v_accvgpr_read_b32 v53, a165\n" \
    "v_accvgpr_read_b32 v54, a166\n" \
    "v_accvgpr_read_b32 v55, a167\n" \
    "v_accvgpr_read_b32 v56, a168\n" \
    "v_accvgpr_read_b32 v57, a169\n" \
    "v_accvgpr_read_b32 v58, a170\n" \
    "v_accvgpr_read_b32 v59, a171\n" \
    "v_accvgpr_read_b32 v60, a172\n" \
    "v_accvgpr_read_b32 v61, a173\n" \
    "v_accvgpr_read_b32 v62, a174\n" \
    "v_accvgpr_read_b32 v63, a175\n" \
    "v_accvgpr_read_b32 v64, a176\n" \
    "v_accvgpr_read_b32 v65, a177\n" \
    "v_accvgpr_read_b32 v66, a178\n" \
    "v_accvgpr_read_b32 v67, a179\n" \
    "v_accvgpr_read_b32 v68, a180\n" \
    "v_accvgpr_read_b32 v69, a181\n" \
    "v_accvgpr_read_b32 v70, a182\n" \
    "v_accvgpr_read_b32 v71, a183\n" \
    "v_accvgpr_read_b32 v72, a184\n" \
    "v_accvgpr_read_b32 v73, a185\n" \
    "v_accvgpr_read_b32 v74, a186\n" \
    "v_accvgpr_read_b32 v75, a187\n" \
    "v_accvgpr_read_b32 v76, a188\n" \
    "v_accvgpr_read_b32 v77, a189\n" \
    "v_accvgpr_read_b32 v78, a190\n" \
    "v_accvgpr_read_b32 v79, a191\n" \
    "v_accvgpr_read_b32 v80, a192\n" \
    "v_accvgpr_read_b32 v81, a193\n" \
    "v_accvgpr_read_b32 v82, a194\n" \
    "v_accvgpr_read_b32 v83, a195\n" \
    "v_accvgpr_read_b32 v84, a196\n" \
    "v_accvgpr_read_b32 v85, a197\n" \
    "v_accvgpr_read_b32 v86, a198\n" \
    "v_accvgpr_read_b32 v87, a199\n" \
    "v_accvgpr_read_b32 v88, a200\n" \
    "v_accvgpr_read_b32 v89, a201\n" \
    "v_accvgpr_read_b32 v90, a202\n" \
    "v_accvgpr_read_b32 v91, a203\n" \
    "v_accvgpr_read_b32 v92, a204\n" \
    "v_accvgpr_read_b32 v93, a205\n" \
    "v_accvgpr_read_b32 v94, a206\n" \
    "v_accvgpr_read_b32 v95, a207\n" \
    "a2a_st_r1:\n" \
    "s_waitcnt vmcnt(8)\n" \
    "v_xor_b32 v11, 0, v12\n" \
    "ds_read_b64_tr_b8 a[144:145], v11\n" \
    "ds_read_b64_tr_b8 a[146:147], v11 offset:1024\n" \
    "ds_read_b64_tr_b8 a[148:149], v11 offset:8192\n" \
    "ds_read_b64_tr_b8 a[150:151], v11 offset:9216\n" \
    "v_xor_b32 v11, 16, v12\n" \
    "ds_read_b64_tr_b8 a[152:153], v11\n" \
    "ds_read_b64_tr_b8 a[154:155], v11 offset:1024\n" \
    "ds_read_b64_tr_b8 a[156:157], v11 offset:8192\n" \
    "ds_read_b64_tr_b8 a[158:159], v11 offset:9216\n" \
    "v_xor_b32 v11, 32, v12\n" \
    "ds_read_b64_tr_b8 a[160:161], v11\n" \
    "ds_read_b64_tr_b8 a[162:163], v11 offset:1024\n" \
    "ds_read_b64_tr_b8 a[164:165], v11 offset:8192\n" \
    "ds_read_b64_tr_b8 a[166:167], v11 offset:9216\n" \
    "v_xor_b32 v11, 48, v12\n" \
    "ds_read_b64_tr_b8 a[168:169], v11\n" \
    "ds_read_b64_tr_b8 a[170:171], v11 offset:1024\n" \
    "ds_read_b64_tr_b8 a[172:173], v11 offset:8192\n" \
    "ds_read_b64_tr_b8 a[174:175], v11 offset:9216\n" \
    "v_xor_b32 v11, 64, v12\n" \
    "ds_read_b64_tr_b8 a[176:177], v11\n" \
    "ds_read_b64_tr_b8 a[178:179], v11 offset:1024\n" \
    "ds_read_b64_tr_b8 a[180:181], v11 offset:8192\n" \
    "ds_read_b64_tr_b8 a[182:183], v11 offset:9216\n" \
    "v_xor_b32 v11, 80, v12\n" \
    "ds_read_b64_tr_b8 a[184:185], v11\n" \
    "ds_read_b64_tr_b8 a[186:187], v11 offset:1024\n" \
    "ds_read_b64_tr_b8 a[188:189], v11 offset:8192\n" \
    "ds_read_b64_tr_b8 a[190:191], v11 offset:9216\n" \
    "v_xor_b32 v11, 96, v12\n" \
    "ds_read_b64_tr_b8 a[192:193], v11\n" \
    "ds_read_b64_tr_b8 a[194:195], v11 offset:1024\n" \
    "ds_read_b64_tr_b8 a[196:197], v11 offset:8192\n" \
    "ds_read_b64_tr_b8 a[198:199], v11 offset:9216\n" \
    "v_xor_b32 v11, 112, v12\n" \
    "ds_read_b64_tr_b8 a[200:201], v11\n" \
    "ds_read_b64_tr_b8 a[202:203], v11 offset:1024\n" \
    "ds_read_b64_tr_b8 a[204:205], v11 offset:8192\n" \
    "ds_read_b64_tr_b8 a[206:207], v11 offset:9216\n" \
    "s_waitcnt lgkmcnt(0)\n" \
    "a2a_st_x:\n" \
    "s_cmp_eq_u32 s73, 6\n" \
    "s_cbranch_scc1 a2a_c0\n" \
    "global_load_dword v10, v15, s[38:39] sc0 sc1\n" \
    "s_cmp_eq_u32 s70, 7\n" \
    "s_cselect_b32 s55, s67, 0\n" \
    "s_cmp_eq_u32 s55, 0\n" \
    "s_cbranch_scc1 a2a_st_w0\n" \
    "s_andn2_b32 s55, s67, s68\n" \
    "s_cselect_b32 s55, s55, s67\n" \
    "s_and_b32 s56, s67, s66\n" \
    "s_cselect_b32 s55, s56, s55\n" \
    "s_or_b32 s55, s55, 0x80\n" \
    "s_ff1_i32_b32 s79, s55\n" \
    "s_lshl_b32 s77, s79, 2\n" \
    "s_lshr_b32 s77, s61, s77\n" \
    "s_and_b32 s77, s77, 7\n" \
    "s_lshl_b32 s77, s77, 21\n" \
    "s_add_u32 s77, s77, s53\n" \
    "s_bitset0_b32 s67, s79\n" \
    "s_mov_b32 s70, s79\n" \
    "s_xor_b32 s76, s72, s54\n" \
    "s_xor_b32 m0, s76, 0\n" \
    "s_xor_b32 s58, s77, 0\n" \
    "buffer_load_dwordx4 v4, s[8:11], s58 offen offset:0 lds\n" \
    "s_add_u32 s58, s58, 31744\n" \
    "buffer_load_dwordx4 v4, s[8:11], s58 offen offset:1024 lds\n" \
    "s_add_u32 s58, s58, 31744\n" \
    "buffer_load_dwordx4 v5, s[8:11], s58 offen offset:2048 lds\n" \
    "s_add_u32 s58, s58, 31744\n" \
    "buffer_load_dwordx4 v5, s[8:11], s58 offen offset:3072 lds\n" \
    "s_xor_b32 m0, s76, 4096\n" \
    "s_xor_b32 s58, s77, 131072\n" \
    "buffer_load_dwordx4 v4, s[8:11], s58 offen offset:0 lds\n" \
    "s_add_u32 s58, s58, 31744\n" \
    "buffer_load_dwordx4 v4, s[8:11], s58 offen offset:1024 lds\n" \
    "s_add_u32 s58, s58, 31744\n" \
    "buffer_load_dwordx4 v5, s[8:11], s58 offen offset:2048 lds\n" \
    "s_add_u32 s58, s58, 31744\n" \
    "buffer_load_dwordx4 v5, s[8:11], s58 offen offset:3072 lds\n" \
    "s_xor_b32 m0, s76, 8192\n" \
    "s_xor_b32 s58, s77, 262144\n" \
    "buffer_load_dwordx4 v4, s[8:11], s58 offen offset:0 lds\n" \
    "s_add_u32 s58, s58, 31744\n" \
    "buffer_load_dwordx4 v4, s[8:11], s58 offen offset:1024 lds\n" \
    "s_add_u32 s58, s58, 31744\n" \
    "buffer_load_dwordx4 v5, s[8:11], s58 offen offset:2048 lds\n" \
    "s_add_u32 s58, s58, 31744\n" \
    "buffer_load_dwordx4 v5, s[8:11], s58 offen offset:3072 lds\n" \
    "s_xor_b32 m0, s76, 12288\n" \
    "s_xor_b32 s58, s77, 393216\n" \
    "buffer_load_dwordx4 v4, s[8:11], s58 offen offset:0 lds\n" \
    "s_add_u32 s58, s58, 31744\n" \
    "buffer_load_dwordx4 v4, s[8:11], s58 offen offset:1024 lds\n" \
    "s_add_u32 s58, s58, 31744\n" \
    "buffer_load_dwordx4 v5, s[8:11], s58 offen offset:2048 lds\n" \
    "s_add_u32 s58, s58, 31744\n" \
    "buffer_load_dwordx4 v5, s[8:11], s58 offen offset:3072 lds\n" \
    "s_waitcnt vmcnt(16)\n" \
    "s_cmp_eq_u32 s69, 7\n" \
    "s_cbranch_scc0 a2a_st_ret\n" \
    "s_mov_b32 s69, s70\n" \
    "s_mov_b32 s70, 7\n" \
    "s_xor_b32 s72, s72, 16384\n" \
    "s_branch a2a_st_ret\n" \
    "a2a_st_w0:\n" \
    "s_waitcnt vmcnt(0)\n" \
    "a2a_st_ret:\n" \
    "s_cmp_eq_u32 s75, 0\n" \
    "s_cbranch_scc1 a2a_c0\n" \
    "s_branch a2a_c1\n" \
    "a2a_end:\n" \
    "s_waitcnt vmcnt(0) lgkmcnt(0)\n"
#define CLOB_S "s8", "s9", "s10", "s11", "s12", "s13", "s14", "s15", "s16", "s17", "s18", "s19", "s20", "s21", "s22", "s23", "s24", "s25", "s26", "s27", "s28", "s29", "s30", "s31", "s36", "s37", "s38", "s39", "s40", "s41", "s42", "s43", "s44", "s45", "s46", "s47", "s48", "s49", "s50", "s51", "s52", "s53", "s54", "s55", "s56", "s57", "s58", "s59", "s60", "s61", "s62", "s63", "s64", "s65", "s66", "s67", "s68", "s69", "s70", "s71", "s72", "s73", "s74", "s75", "s76", "s77", "s78", "s79", "s80", "s81", "s82", "s83", "s84", "s85", "s86", "s87", "s88", "s89", "s90"

#define R10(r, x) #r #x "0", #r #x "1", #r #x "2", #r #x "3", #r #x "4", #r #x "5", #r #x "6", #r #x "7", #r #x "8", #r #x "9"
#define R3_24(r) R10(r, 3), R10(r, 4), R10(r, 5), R10(r, 6), R10(r, 7), R10(r, 8), R10(r, 9), R10(r, 10), R10(r, 11), R10(r, 12), \
    R10(r, 13), R10(r, 14), R10(r, 15), R10(r, 16), R10(r, 17), R10(r, 18), R10(r, 19), R10(r, 20), R10(r, 21), R10(r, 22), \
    R10(r, 23), R10(r, 24), #r "250", #r "251", #r "252", #r "253", #r "254", #r "255"
#define CLOB_V "v1", "v2", "v3", "v4", "v5", "v6", "v7", "v8", "v9", R10(v, 1), "v20", "v21", "v22", "v23", "v24", "v25", "v28", "v29", R3_24(v)
#define CLOB_A R10(a, ), R10(a, 1), R10(a, 2), R3_24(a)

__global__ __launch_bounds__(256) void a2a_gemm_kernel(Params p) {
    __shared__ __attribute__((aligned(16384))) char smem[LDS_B];
    asm volatile(KERNEL_BODY :
                 : [ka] "s"(__builtin_amdgcn_kernarg_segment_ptr()), [bid] "s"(blockIdx.x), [tid] "v"(threadIdx.x),
                   [lds] "s"((unsigned)(size_t)(LAS char*)smem)
                 : "memory", "scc", "vcc", CLOB_S, CLOB_V, CLOB_A);
}

extern "C" void kernel_launch(int rank, int nranks, ncclComm_t comm, hipStream_t stream,
                              void* const* d_in, const int* in_sizes, int n_in,
                              void* d_out, int out_size, const int* in_dtypes, int out_dtype,
                              void* const* d_out_all, void* const* d_ws_all, size_t ws_bytes) {
    (void)comm; (void)in_dtypes; (void)out_dtype; (void)d_out_all;
    if (nranks != NR || (unsigned)rank >= (unsigned)NR || n_in != 4 || ws_bytes < WS_NEED + 4096 ||
        in_sizes[0] != NN * KC || in_sizes[1] != NN * NN || out_size != KC * NN) {
        fprintf(stderr, "unexpected problem shape\n");
        return;
    }
    Params p = {};
    for (int t = 0; t < 7; ++t)
        for (int r = 0; r < 7; ++r) p.srcs[t] |= (unsigned)((rank + kTake[t][r]) & 7) << (4 * r);
    for (int rel = 1; rel < NR; ++rel) {
        int pos = 0;
        for (int r = 0; r < 7; ++r) {
            const int t = kSend[rel - 1][r], b0 = t <= 3 ? 4 * t : 5 * t - 3;
            for (int k = 0; k < (t < 3 ? 4 : 5); ++k, ++pos) p.tab[rel][pos % 3][pos / 18] |= (unsigned)(b0 + k) << (5 * (pos / 3 % 6));
            p.tab[rel][3][0] |= (unsigned)t << (4 * r);
        }
    }
    for (int rel = 0; rel < NR; ++rel) {
        const int d = (rank - rel) & 7;
        p.peer[rel][0] = (u64)d_in[0] + (size_t)d * KC * KC;
        p.peer[rel][1] = (u64)d_ws_all[d];
    }
    auto desc = [](unsigned* r, u64 a, unsigned bytes) {
        r[0] = (unsigned)a; r[1] = (unsigned)(a >> 32) & 0xffffu; r[2] = bytes; r[3] = 0x00020000u;
    };
    desc(p.rw, (u64)d_in[1], NN * NN);
    desc(p.rx, p.peer[0][0], KC * KC);
    desc(p.ra, p.peer[0][1], 2 * SLOT_B);
    p.rank = rank;
    p.pad = 0;
    p.out = (u64)d_out;
    p.sx = (u64)d_in[2];
    p.sw = (u64)d_in[3];
    a2a_gemm_kernel<<<256, 256, 0, stream>>>(p);
    HIP_CHECK(hipGetLastError());
}
```
